```python
import numpy as np

kernel_name = "gpu_a2a_gemm_m4096_k4096_n16384_bf16_relu"


ABSMAX_RTOL = 0.008

COMPUTE_DTYPE = "bfloat16"
EPILOGUE = "relu"
IN_QUANT = None
OUT_QUANT = None
IN_DTYPES = {"x": "bfloat16", "w_mat": "bfloat16"}
WORLD_SIZE = 8

def _to_bf16(x):
    u = np.ascontiguousarray(x, dtype=np.float32).view(np.uint32)
    return ((u + 0x7FFF + ((u >> 16) & 1)) & 0xFFFF0000).view(np.float32)

def setup_inputs():

    x = _to_bf16(np.random.randn(4096, 4096).astype(np.float32))
    w_mat = _to_bf16((np.random.randn(4096, 16384) / np.sqrt(4096)).astype(np.float32))
    return {"x": x, "w_mat": w_mat}

def _epilogue(y):
    return np.maximum(y, 0.0).astype(np.float32)

def reference(x, w_mat):
    y = (x @ w_mat).astype(np.float32)
    return _epilogue(y)
```

```cpp
#include <hip/hip_runtime.h>
#include <rccl/rccl.h>
#include <cstdio>
#include <cstdint>

typedef __bf16 bf16x8 __attribute__((ext_vector_type(8)));
typedef __bf16 bf16x2 __attribute__((ext_vector_type(2)));
typedef short s16x4 __attribute__((ext_vector_type(4)));
typedef float f32x4 __attribute__((ext_vector_type(4)));
typedef float f32x2 __attribute__((ext_vector_type(2)));
typedef unsigned u32x4 __attribute__((ext_vector_type(4)));
typedef int i32x4 __attribute__((ext_vector_type(4)));
#define LAS __attribute__((address_space(3)))
#define DI __device__ __forceinline__
#define SB() __builtin_amdgcn_sched_barrier(0)
#define BAR() asm volatile("s_barrier" ::: "memory")

constexpr int NR = 8, KTOT = 4096, NTOT = 16384;
constexpr unsigned A_RING = 0, A_ST = 32768, B_RING = 3 * 32768, B_ST = 16384, LDS_BYTES = 163840;
constexpr unsigned WS_FLAGS = 0;
constexpr unsigned WS_TMO = 4096;
constexpr unsigned WS_CTR = 8192;
constexpr unsigned WS_X = 1u << 20;
constexpr unsigned BLK = 524288;
constexpr unsigned XPAR = 8 * BLK;
constexpr size_t WS_NEED = (size_t)WS_X + 2 * (size_t)XPAR;
constexpr unsigned long long DEADLINE_TICKS = 100000000ull;

struct Params { const char* x; const char* w; char* out; char* ws[NR]; int rank; int pad; };

DI unsigned rfl(unsigned x) { return __builtin_amdgcn_readfirstlane(x); }
DI unsigned cvtpk(float lo, float hi) { f32x2 v = {lo, hi}; bf16x2 b = __builtin_convertvector(v, bf16x2); return __builtin_bit_cast(unsigned, b); }
DI i32x4 make_srd(const void* p, unsigned bytes) {
  unsigned long long a = (unsigned long long)p;
  i32x4 r; r[0] = rfl((unsigned)a); r[1] = rfl((unsigned)(a >> 32)) & 0xffff; r[2] = rfl(bytes); r[3] = 0x00020000; return r;
}
template <int SC, int PAD> DI void dma4(unsigned m0v, unsigned v0, unsigned v1, unsigned v2, unsigned v3, i32x4 srd, unsigned so) {
  if constexpr (SC)
    asm volatile("s_mov_b32 m0, %0\n\ts_nop %7\n\t"
                 "buffer_load_dwordx4 %1, %5, %6 offen sc0 sc1 lds\n\t"
                 "buffer_load_dwordx4 %2, %5, %6 offen offset:1024 sc0 sc1 lds\n\t"
                 "buffer_load_dwordx4 %3, %5, %6 offen offset:2048 sc0 sc1 lds\n\t"
                 "buffer_load_dwordx4 %4, %5, %6 offen offset:3072 sc0 sc1 lds"
                 :: "s"(m0v), "v"(v0), "v"(v1), "v"(v2), "v"(v3), "s"(srd), "s"(so), "i"(PAD) : "memory");
  else
    asm volatile("s_mov_b32 m0, %0\n\ts_nop %7\n\t"
                 "buffer_load_dwordx4 %1, %5, %6 offen lds\n\t"
                 "buffer_load_dwordx4 %2, %5, %6 offen offset:1024 lds\n\t"
                 "buffer_load_dwordx4 %3, %5, %6 offen offset:2048 lds\n\t"
                 "buffer_load_dwordx4 %4, %5, %6 offen offset:3072 lds"
                 :: "s"(m0v), "v"(v0), "v"(v1), "v"(v2), "v"(v3), "s"(srd), "s"(so), "i"(PAD) : "memory");
}
DI s16x4 trd(const LAS char* p) { return __builtin_amdgcn_ds_read_tr16_b64_v4i16((LAS s16x4*)p); }
DI unsigned ld_agent(const char* p) { return __hip_atomic_load((const unsigned*)p, __ATOMIC_RELAXED, __HIP_MEMORY_SCOPE_AGENT); }

__global__ __launch_bounds__(512, 2) void a2a_gemm_relu(const Params P) {
  extern __shared__ __attribute__((aligned(16))) char smem[];
  LAS char* lds = (LAS char*)smem;
  const int tid = threadIdx.x, lane = tid & 63;
  const unsigned wid = rfl(tid >> 6);
  const unsigned b = blockIdx.x, xcd = b & 7, idx = b >> 3, tn = xcd * 16 + (idx >> 1), tm = idx & 1;
  const unsigned rank = (unsigned)P.rank & 7u;
  char* const myws = P.ws[rank];

  if (wid == 7) {
    if (b >= 224) return;
    const unsigned kt = b / 28, r = b - kt * 28, s = (r >> 2) + 1, qd = r & 3, dest = (rank - s) & 7u;
    const unsigned callv = ld_agent(myws + WS_CTR + b * 128) + 1;
    const char* src = P.x + (size_t)dest * BLK + (size_t)(qd * 128 + (lane >> 3)) * 1024 + kt * 128 + (lane & 7) * 16;
    u32x4 d[16];
    #pragma unroll
    for (int i = 0; i < 16; ++i) d[i] = *(const u32x4*)(src + i * 8192);
    __builtin_amdgcn_s_waitcnt(0x0F70);
    const unsigned call = rfl(callv);
    char* const pws = P.ws[dest];
    char* dst = pws + WS_X + (call & 1) * XPAR + rank * BLK + kt * 65536 + qd * 16384 + lane * 16;
    const int su = kt == 0 ? -1 : (int)(2 * kt - 2 + (qd >> 1));
    for (int i = -1; i < su; ++i) BAR();
    #pragma unroll
    for (int i = 0; i < 16; ++i)
      asm volatile("global_store_dwordx4 %0, %1, off sc0 sc1\n\ts_nop 1" :: "v"(dst + i * 1024), "v"(d[i]) : "memory");
    BAR(); BAR();
    if (kt != 0) BAR();
    asm volatile("s_waitcnt vmcnt(0)" ::: "memory");
    if (lane == 0)
      __hip_atomic_store((unsigned*)(pws + WS_FLAGS + qd * 256 + (8 + kt * 7 + s - 1) * 4), call, __ATOMIC_RELAXED, __HIP_MEMORY_SCOPE_SYSTEM);
    return;
  }
  if (wid >= 4) {
    unsigned tabA, tabB;
    { const unsigned u = lane; unsigned src, kt;
      if (u < 8) { src = rank; kt = u; tabA = kt * 128; }
      else { const unsigned j = u - 8; kt = (j * 37) >> 8; src = (rank + (j - kt * 7 + 1)) & 7u; tabA = src * BLK + kt * 65536u; }
      tabB = (src * 512u + kt * 64u) * 32768u; }
    if (wid < 6) {
      const unsigned h = wid - 4;
      unsigned callv;
      asm volatile("global_load_dword %0, %1, off sc1" : "=v"(callv) : "v"(myws + WS_CTR + b * 128) : "memory");
      const i32x4 srdL = make_srd(P.x + (size_t)rank * BLK, BLK);
      const unsigned arow = tm * 256 + h * 128 + (lane >> 3), ach = ((lane & 7) ^ (lane >> 3)) * 16;
      unsigned vL[4];
      #pragma unroll
      for (int i = 0; i < 4; ++i) vL[i] = (arow + 8 * i) * 1024 + ach - i * 1024;
      const unsigned vR = arow * 128 + ach, m0A = A_RING + h * 16384;
      auto issueL = [&](int t, unsigned slot) {
        const unsigned so = t * 128, m = m0A + slot * A_ST;
        #pragma unroll
        for (int j = 0; j < 4; ++j) dma4<1, 0>(m + j * 4096, vL[0], vL[1], vL[2], vL[3], srdL, so + j * 32768);
      };
      issueL(0, 0); issueL(1, 1); issueL(2, 2);
      asm volatile("s_waitcnt vmcnt(32)" : "+v"(callv) :: "memory");
      BAR();
      const unsigned call = rfl(callv) + 1;
      i32x4 srdR = make_srd(myws + WS_X + (call & 1) * XPAR, XPAR);
      const char* fl = myws + WS_FLAGS + (tm * 2 + h) * 256 + lane * 4;
      const char* tmo = myws + WS_TMO;
      #define SAMPLE() asm volatile("global_load_dword v200, %0, off sc0 sc1" :: "v"(fl) : "memory", "v200")
      unsigned long long ready = 0xffull;
      unsigned slot = 0;
      for (int u = 0; u < 5; ++u) {
        asm volatile("s_waitcnt vmcnt(16)" ::: "memory"); BAR();
        if (u == 4) SAMPLE();
        issueL(u + 3, slot);
        slot = slot == 2 ? 0 : slot + 1;
      }
      for (int u = 5; u < 64; ++u) {
        const int t = u + 3;
        if (ready != ~0ull) {
          unsigned long long mk;
          asm volatile("s_waitcnt vmcnt(16)\n\ts_barrier\n\tv_cmp_ge_u32 %0, v200, %1" : "=s"(mk) : "v"(call) : "memory", "v200");
          ready |= mk;
          if (!((ready >> t) & 1)) {
            const unsigned long long t0 = __builtin_amdgcn_s_memrealtime();
            for (;;) {
              unsigned fv, tv;
              asm volatile("global_load_dword %0, %2, off sc0 sc1\n\tglobal_load_dword %1, %3, off sc1\n\ts_waitcnt vmcnt(0)"
                           : "=&v"(fv), "=&v"(tv) : "v"(fl), "v"(tmo) : "memory");
              ready |= __ballot(fv >= call);
              if ((ready >> t) & 1) break;
              if (rfl(tv) == call || __builtin_amdgcn_s_memrealtime() - t0 > DEADLINE_TICKS) {
                if (lane == 0) __hip_atomic_store((unsigned*)tmo, call, __ATOMIC_RELAXED, __HIP_MEMORY_SCOPE_AGENT);
                asm volatile("s_waitcnt vmcnt(0)" ::: "memory");
                ready = ~0ull; srdR[2] = 0;
                break;
              }
              __builtin_amdgcn_s_sleep(2);
            }
          }
          ready |= (2ull << t) - 1;
          if (ready != ~0ull) SAMPLE();
        } else {
          asm volatile("s_waitcnt vmcnt(16)\n\ts_barrier" ::: "memory");
        }
        unsigned so = __builtin_amdgcn_readlane(tabA, t & 63);
        if (t >= 64) { srdR[2] = 0; so = 0; }
        const unsigned m = m0A + slot * A_ST;
        dma4<1, 3>(m, vR, vR, vR, vR, srdR, so);
        #pragma unroll
        for (int j = 1; j < 4; ++j) dma4<1, 0>(m + j * 4096, vR, vR, vR, vR, srdR, so + j * 4096);
        slot = slot == 2 ? 0 : slot + 1;
      }
    } else {
      i32x4 srdW = make_srd(P.w, (unsigned)KTOT * NTOT * 2u);
      unsigned vB[4];
      #pragma unroll
      for (int i = 0; i < 4; ++i) vB[i] = (4 * i + (lane >> 4)) * 32768 + tn * 256 + (((lane & 15) ^ (((lane >> 4) << 2) | (i >> 1))) * 16) - i * 1024;
      auto issue = [&](int t, unsigned slot) {
        const unsigned so = __builtin_amdgcn_readlane(tabB, t & 63), m = B_RING + slot * B_ST;
        if (t >= 64) srdW[2] = 0;
        dma4<0, 3>(m, vB[0], vB[1], vB[2], vB[3], srdW, so);
        #pragma unroll
        for (int j = 1; j < 4; ++j) dma4<0, 0>(m + j * 4096, vB[0], vB[1], vB[2], vB[3], srdW, so + j * 16 * 32768);
      };
      issue(0, 0); issue(1, 1); issue(2, 2);
      asm volatile("s_waitcnt vmcnt(32)" ::: "memory");
      issue(3, 3);
      BAR();
      for (int u = 0; u < 64; ++u) {
        asm volatile("s_waitcnt vmcnt(32)" ::: "memory"); BAR();
        issue(u + 4, u & 3);
      }
    }
    return;
  }
  const unsigned wm = wid >> 1, wn = wid & 1, fr = lane & 15, fq = lane >> 4, q = (lane & 15) >> 2, p = lane & 3;
  const unsigned ra0 = A_RING + (wm * 128 + fr) * 128 + ((fq ^ (fr & 7)) * 16);
  const unsigned rb00 = B_RING + (8 * fq + q) * 256 + (((wn * 8 + p) ^ ((q << 2) | (fq & 1))) * 16) + (p & 1) * 8;
  const unsigned call = rfl(ld_agent(myws + WS_CTR + b * 128)) + 1;
  unsigned tmov = 0;
  int ulast = 63; asm volatile("" : "+s"(ulast));
  f32x4 acc[8][4];
  #pragma unroll
  for (int m = 0; m < 8; ++m)
    #pragma unroll
    for (int n = 0; n < 4; ++n) acc[m][n] = (f32x4){0.f, 0.f, 0.f, 0.f};
  bf16x8 af[8]; s16x4 b0l[4], b0h[4], b1l[4], b1h[4];
  #define RDA(dst, base, m) dst = *(const LAS bf16x8*)(lds + (base) + (m) * 2048)
  #define RDB(L, H, base, n, ks) do { const LAS char* pb_ = lds + ((base) ^ (((n) >> 1) * 64 + ((n) & 1) * 8)); L[n] = trd(pb_ + (ks) * 8192); H[n] = trd(pb_ + (ks) * 8192 + 1024); } while (0)
  #define MF(m, n, L, H) do { acc[m][n] = __builtin_amdgcn_mfma_f32_16x16x32_bf16(__builtin_bit_cast(bf16x8, __builtin_shufflevector(L[n], H[n], 0, 1, 2, 3, 4, 5, 6, 7)), af[m], acc[m][n], 0, 0, 0); \
                              asm volatile("" : "+v"(acc[m][n])); } while (0)
  BAR();
  #pragma unroll
  for (int n = 0; n < 4; ++n) RDB(b0l, b0h, rb00, n, 0);
  #pragma unroll
  for (int m = 0; m < 7; ++m) RDA(af[m], ra0, m);
  unsigned sa = 0, sb = 0;
  for (int u = 0; u < 64; ++u) {
    const unsigned a0 = ra0 + sa * A_ST, a1 = a0 ^ 64, bb = rb00 + sb * B_ST;
    sa = sa == 2 ? 0 : sa + 1; sb = (sb + 1) & 3;
    const unsigned a0n = ra0 + sa * A_ST, bbn = rb00 + sb * B_ST;
    if (u == ulast) tmov = ld_agent(myws + WS_TMO);
    SB();
    MF(0, 0, b0l, b0h); SB(); RDA(af[7], a0, 7); SB(); MF(0, 1, b0l, b0h); SB(); RDB(b1l, b1h, bb, 0, 1); SB(); MF(0, 2, b0l, b0h); MF(0, 3, b0l, b0h); SB();
    #pragma unroll
    for (int g = 1; g < 8; ++g) {
      MF(g, 0, b0l, b0h); SB(); RDA(af[g - 1], a1, g - 1); SB(); MF(g, 1, b0l, b0h); SB();
      if (g < 4) RDB(b1l, b1h, bb, g, 1);
      SB(); MF(g, 2, b0l, b0h); MF(g, 3, b0l, b0h); SB();
    }
    MF(0, 0, b1l, b1h); SB(); RDA(af[7], a1, 7); SB(); MF(0, 1, b1l, b1h); MF(0, 2, b1l, b1h); MF(0, 3, b1l, b1h); SB();
    MF(1, 0, b1l, b1h); MF(1, 1, b1l, b1h); MF(1, 2, b1l, b1h); MF(1, 3, b1l, b1h); SB();
    asm volatile("s_waitcnt lgkmcnt(0)\n\ts_barrier" ::: "memory"); SB();
    MF(2, 0, b1l, b1h); SB(); RDA(af[0], a0n, 0); SB(); MF(2, 1, b1l, b1h); SB(); RDB(b0l, b0h, bbn, 0, 0); SB(); MF(2, 2, b1l, b1h); SB(); RDA(af[1], a0n, 1); SB(); MF(2, 3, b1l, b1h); SB();
    #pragma unroll
    for (int g = 3; g < 8; ++g) {
      MF(g, 0, b1l, b1h); SB(); RDA(af[g - 1], a0n, g - 1); SB(); MF(g, 1, b1l, b1h); SB();
      if (g < 6) RDB(b0l, b0h, bbn, g - 2, 0);
      SB(); MF(g, 2, b1l, b1h); MF(g, 3, b1l, b1h); SB();
    }
  }
  const bool bad = tmov == call;
  char* orow = P.out + ((size_t)(tm * 256 + wm * 128 + fr) * NTOT + tn * 128 + wn * 64 + fq * 8) * 2;
  const bool odd = fq & 1;
  #pragma unroll
  for (int m = 0; m < 8; ++m)
    #pragma unroll
    for (int pr = 0; pr < 2; ++pr) {
      const f32x4 a = acc[m][2 * pr], c = acc[m][2 * pr + 1];
      const unsigned a0 = cvtpk(fmaxf(a[0], 0.f), fmaxf(a[1], 0.f)), a1 = cvtpk(fmaxf(a[2], 0.f), fmaxf(a[3], 0.f));
      const unsigned c0 = cvtpk(fmaxf(c[0], 0.f), fmaxf(c[1], 0.f)), c1 = cvtpk(fmaxf(c[2], 0.f), fmaxf(c[3], 0.f));
      u32x4 v; v[0] = odd ? c0 : a0; v[1] = odd ? c1 : a1; v[2] = odd ? a0 : c0; v[3] = odd ? a1 : c1;
      if (bad) v = (u32x4){0x7fc07fc0u, 0x7fc07fc0u, 0x7fc07fc0u, 0x7fc07fc0u};
      char* pp = orow + (size_t)m * 16 * NTOT * 2 + pr * 64;
      asm volatile("global_store_dwordx4 %0, %1, off sc1\n\ts_nop 1" :: "v"(pp), "v"(v) : "memory");
    }
  if (tid == 0) __hip_atomic_store((unsigned*)(myws + WS_CTR + b * 128), call, __ATOMIC_RELAXED, __HIP_MEMORY_SCOPE_AGENT);
}

extern "C" void kernel_launch(int rank, int nranks, ncclComm_t comm, hipStream_t stream,
                              void* const* d_in, const int* in_sizes, int n_in,
                              void* d_out, int out_size, const int* in_dtypes, int out_dtype,
                              void* const* d_out_all, void* const* d_ws_all, size_t ws_bytes) {
  (void)comm; (void)in_sizes; (void)n_in; (void)out_size; (void)in_dtypes; (void)out_dtype; (void)d_out_all;
  if (nranks != NR || (unsigned)rank >= (unsigned)NR || ws_bytes < WS_NEED) { fprintf(stderr, "unexpected launch geometry\n"); return; }
  static bool init = false;
  if (!init) {
    hipError_t e = hipFuncSetAttribute((const void*)a2a_gemm_relu, hipFuncAttributeMaxDynamicSharedMemorySize, (int)LDS_BYTES);
    if (e != hipSuccess) { fprintf(stderr, "hipFuncSetAttribute: %s\n", hipGetErrorString(e)); return; }
    init = true;
  }
  Params p;
  p.x = (const char*)d_in[0]; p.w = (const char*)d_in[1]; p.out = (char*)d_out;
  for (int i = 0; i < NR; ++i) p.ws[i] = (char*)d_ws_all[i];
  p.rank = rank; p.pad = 0;
  hipLaunchKernelGGL(a2a_gemm_relu, dim3(256), dim3(512), LDS_BYTES, stream, p);
  hipError_t e = hipGetLastError();
  if (e != hipSuccess) fprintf(stderr, "launch: %s\n", hipGetErrorString(e));
}
```

```cpp
#include <hip/hip_runtime.h>
#include <rccl/rccl.h>
#include <cstdio>
#include <cstdint>

typedef __bf16 bf16x8 __attribute__((ext_vector_type(8)));
typedef __bf16 bf16x2 __attribute__((ext_vector_type(2)));
typedef short s16x4 __attribute__((ext_vector_type(4)));
typedef short s16x2 __attribute__((ext_vector_type(2)));
typedef float f32x4 __attribute__((ext_vector_type(4)));
typedef float f32x2 __attribute__((ext_vector_type(2)));
typedef unsigned u32x4 __attribute__((ext_vector_type(4)));
typedef int i32x4 __attribute__((ext_vector_type(4)));
#define LAS __attribute__((address_space(3)))
#define DI __device__ __forceinline__
#define SB() __builtin_amdgcn_sched_barrier(0)
#define BAR() asm volatile("s_barrier" ::: "memory")

#define PFK 3
#define PFD 6
#define XPF_REMOTE 1
constexpr int NR = 8, KTOT = 4096, NTOT = 16384;
constexpr unsigned A_RING = 0, A_ST = 32768, B_RING = 3 * 32768, B_ST = 16384, LDS_BYTES = 163840;
constexpr unsigned WS_FLAGS = 0;
constexpr unsigned WS_TMO = 4096;
constexpr unsigned WS_CTR = 8192;
constexpr unsigned WS_X = 1u << 20;
constexpr unsigned BLK = 524288;
constexpr unsigned XPAR = 8 * BLK;
constexpr size_t WS_NEED = (size_t)WS_X + 2 * (size_t)XPAR;
constexpr unsigned long long DEADLINE_TICKS = 100000000ull;

struct Params { const char* x; const char* w; char* out; char* ws[NR]; int rank; int pad; };

DI unsigned rfl(unsigned x) { return __builtin_amdgcn_readfirstlane(x); }
DI unsigned cvtpk(float lo, float hi) { f32x2 v = {lo, hi}; bf16x2 b = __builtin_convertvector(v, bf16x2); return __builtin_bit_cast(unsigned, b); }
DI unsigned relu2(unsigned x) { const s16x2 v = __builtin_bit_cast(s16x2, x), z = {0, 0}; return __builtin_bit_cast(unsigned, __builtin_elementwise_max(v, z)); }
DI i32x4 make_srd(const void* p, unsigned bytes) {
  unsigned long long a = (unsigned long long)p;
  i32x4 r; r[0] = rfl((unsigned)a); r[1] = rfl((unsigned)(a >> 32)) & 0xffff; r[2] = rfl(bytes); r[3] = 0x00020000; return r;
}
template <int SC, int PAD> DI void dma4(unsigned m0v, unsigned v0, unsigned v1, unsigned v2, unsigned v3, i32x4 srd, unsigned so) {
  if constexpr (SC)
    asm volatile("s_mov_b32 m0, %0\n\ts_nop %7\n\t"
                 "buffer_load_dwordx4 %1, %5, %6 offen sc0 sc1 lds\n\t"
                 "buffer_load_dwordx4 %2, %5, %6 offen offset:1024 sc0 sc1 lds\n\t"
                 "buffer_load_dwordx4 %3, %5, %6 offen offset:2048 sc0 sc1 lds\n\t"
                 "buffer_load_dwordx4 %4, %5, %6 offen offset:3072 sc0 sc1 lds"
                 :: "s"(m0v), "v"(v0), "v"(v1), "v"(v2), "v"(v3), "s"(srd), "s"(so), "i"(PAD) : "memory");
  else
    asm volatile("s_mov_b32 m0, %0\n\ts_nop %7\n\t"
                 "buffer_load_dwordx4 %1, %5, %6 offen lds\n\t"
                 "buffer_load_dwordx4 %2, %5, %6 offen offset:1024 lds\n\t"
                 "buffer_load_dwordx4 %3, %5, %6 offen offset:2048 lds\n\t"
                 "buffer_load_dwordx4 %4, %5, %6 offen offset:3072 lds"
                 :: "s"(m0v), "v"(v0), "v"(v1), "v"(v2), "v"(v3), "s"(srd), "s"(so), "i"(PAD) : "memory");
}
DI s16x4 trd(unsigned a) { return __builtin_amdgcn_ds_read_tr16_b64_v4i16((LAS s16x4*)(size_t)a); }
DI bf16x8 rda(unsigned a) { return *(const LAS bf16x8*)(size_t)a; }
DI unsigned ld_agent(const char* p) { return __hip_atomic_load((const unsigned*)p, __ATOMIC_RELAXED, __HIP_MEMORY_SCOPE_AGENT); }

__global__ __launch_bounds__(512, 2) void a2a_gemm_relu(const Params P) {
  const int tid = threadIdx.x, lane = tid & 63;
  const unsigned wid = rfl(tid >> 6);
  const unsigned b = blockIdx.x, xcd = b & 7, idx = b >> 3, tn = xcd * 16 + (idx >> 1), tm = idx & 1;
  const unsigned rank = (unsigned)P.rank & 7u;
  char* const myws = P.ws[rank];

  if (wid == 7) {
    if (b >= 224) return;
    const unsigned kt = b / 28, r = b - kt * 28, s = (r >> 2) + 1, qd = r & 3, dest = (rank - s) & 7u;
    const unsigned callv = ld_agent(myws + WS_CTR + b * 128) + 1;
    const char* src = P.x + (size_t)dest * BLK + (size_t)(qd * 128 + (lane >> 3)) * 1024 + kt * 128 + (lane & 7) * 16;
    u32x4 d[16];
    #pragma unroll
    for (int i = 0; i < 16; ++i) d[i] = *(const u32x4*)(src + i * 8192);
    __builtin_amdgcn_s_waitcnt(0x0F70);
    const unsigned call = rfl(callv);
    char* const pws = P.ws[dest];
    char* dst = pws + WS_X + (call & 1) * XPAR + rank * BLK + kt * 65536 + qd * 16384 + lane * 16;
    const int su = kt == 0 ? -1 : (int)(2 * kt - 2 + (qd >> 1));
    for (int i = -1; i < su; ++i) BAR();
    #pragma unroll
    for (int i = 0; i < 16; ++i)
      asm volatile("global_store_dwordx4 %0, %1, off sc0 sc1\n\ts_nop 1" :: "v"(dst + i * 1024), "v"(d[i]) : "memory");
    BAR(); BAR();
    if (kt != 0) BAR();
    asm volatile("s_waitcnt vmcnt(0)" ::: "memory");
    if (lane == 0)
      __hip_atomic_store((unsigned*)(pws + WS_FLAGS + qd * 256 + (8 + kt * 7 + s - 1) * 4), call, __ATOMIC_RELAXED, __HIP_MEMORY_SCOPE_SYSTEM);
    return;
  }
  if (wid >= 4) {
    unsigned tabA, tabB;
    { const unsigned u = lane; unsigned src, kt;
      if (u < 8) { src = rank; kt = u; tabA = kt * 128; }
      else { const unsigned j = u - 8; kt = (j * 37) >> 8; src = (rank + (j - kt * 7 + 1)) & 7u; tabA = src * BLK + kt * 65536u; }
      tabB = (src * 512u + kt * 64u) * 32768u; }
    if (wid < 6) {
      const unsigned h = wid - 4;
      unsigned callv;
      asm volatile("global_load_dword %0, %1, off sc1" : "=v"(callv) : "v"(myws + WS_CTR + b * 128) : "memory");
      const i32x4 srdL = make_srd(P.x + (size_t)rank * BLK, BLK);
      const unsigned arow = tm * 256 + h * 128 + (lane >> 3), ach = ((lane & 7) ^ (lane >> 3)) * 16;
      unsigned vL[4];
      #pragma unroll
      for (int i = 0; i < 4; ++i) vL[i] = (arow + 8 * i) * 1024 + ach - i * 1024;
      const unsigned vR = arow * 128 + ach, m0A = A_RING + h * 16384;
      auto issueL = [&](int t, unsigned slot) {
        const unsigned so = t * 128, m = m0A + slot * A_ST;
        #pragma unroll
        for (int j = 0; j < 4; ++j) dma4<1, 0>(m + j * 4096, vL[0], vL[1], vL[2], vL[3], srdL, so + j * 32768);
      };
      issueL(0, 0); issueL(1, 1); issueL(2, 2);
      asm volatile("s_waitcnt vmcnt(32)" : "+v"(callv) :: "memory");
      BAR();
      const unsigned call = rfl(callv) + 1;
      i32x4 srdR = make_srd(myws + WS_X + (call & 1) * XPAR, XPAR);
      const char* fl = myws + WS_FLAGS + (tm * 2 + h) * 256 + lane * 4;
      const char* tmo = myws + WS_TMO;
      #define SAMPLE() asm volatile("global_load_dword v200, %0, off sc0 sc1" :: "v"(fl) : "memory", "v200")
      unsigned long long ready = 0xffull;
      unsigned slot = 0;
      for (int u = 0; u < 5; ++u) {
        asm volatile("s_waitcnt vmcnt(16)" ::: "memory"); BAR();
        if (u == 4) SAMPLE();
        issueL(u + 3, slot);
        slot = slot == 2 ? 0 : slot + 1;
      }
      for (int u = 5; u < 64; ++u) {
        const int t = u + 3;
        if (ready != ~0ull) {
          unsigned long long mk;
          asm volatile("s_waitcnt vmcnt(16)\n\ts_barrier\n\tv_cmp_ge_u32 %0, v200, %1" : "=s"(mk) : "v"(call) : "memory", "v200");
          ready |= mk;
          if (!((ready >> t) & 1)) {
            const unsigned long long t0 = __builtin_amdgcn_s_memrealtime();
            for (;;) {
              unsigned fv, tv;
              asm volatile("global_load_dword %0, %2, off sc0 sc1\n\tglobal_load_dword %1, %3, off sc1\n\ts_waitcnt vmcnt(0)"
                           : "=&v"(fv), "=&v"(tv) : "v"(fl), "v"(tmo) : "memory");
              ready |= __ballot(fv >= call);
              if ((ready >> t) & 1) break;
              if (rfl(tv) == call || __builtin_amdgcn_s_memrealtime() - t0 > DEADLINE_TICKS) {
                if (lane == 0) __hip_atomic_store((unsigned*)tmo, call, __ATOMIC_RELAXED, __HIP_MEMORY_SCOPE_AGENT);
                asm volatile("s_waitcnt vmcnt(0)" ::: "memory");
                ready = ~0ull; srdR[2] = 0;
                break;
              }
              __builtin_amdgcn_s_sleep(2);
            }
          }
          ready |= (2ull << t) - 1;
          if (ready != ~0ull) SAMPLE();
        } else {
          asm volatile("s_waitcnt vmcnt(16)\n\ts_barrier" ::: "memory");
        }
        unsigned so = __builtin_amdgcn_readlane(tabA, t & 63);
        if (t >= 64) { srdR[2] = 0; so = 0; }
        const unsigned m = m0A + slot * A_ST;
        dma4<1, 3>(m, vR, vR, vR, vR, srdR, so);
        #pragma unroll
        for (int j = 1; j < 4; ++j) dma4<1, 0>(m + j * 4096, vR, vR, vR, vR, srdR, so + j * 4096);
        slot = slot == 2 ? 0 : slot + 1;
      }
    } else {
      i32x4 srdW = make_srd(P.w, (unsigned)KTOT * NTOT * 2u);
      unsigned vB[4];
      #pragma unroll
      for (int i = 0; i < 4; ++i) vB[i] = (4 * i + (lane >> 4)) * 32768 + tn * 256 + (((lane & 15) ^ (((lane >> 4) << 2) | (i >> 1))) * 16) - i * 1024;
      auto issue = [&](int t, unsigned slot) {
        const unsigned so = __builtin_amdgcn_readlane(tabB, t & 63), m = B_RING + slot * B_ST;
        if (t >= 64) srdW[2] = 0;
        dma4<0, 3>(m, vB[0], vB[1], vB[2], vB[3], srdW, so);
        #pragma unroll
        for (int j = 1; j < 4; ++j) dma4<0, 0>(m + j * 4096, vB[0], vB[1], vB[2], vB[3], srdW, so + j * 16 * 32768);
      };
      unsigned callv, dummy = 0;
      asm volatile("global_load_dword %0, %1, off sc1" : "=v"(callv) : "v"(myws + WS_CTR + b * 128) : "memory");
      i32x4 srdWp = make_srd(P.w, (unsigned)KTOT * NTOT * 2u);
      const i32x4 srdXL = make_srd(P.x + (size_t)rank * BLK, BLK);
      const unsigned vW = (idx * 2 + (lane >> 5)) * 32768 + xcd * 4096 + (lane & 31) * 128;
      const unsigned vX = (idx * 16 + (lane & 15)) * 128, vXL = (idx * 16 + (lane & 15)) * 1024;
      issue(0, 0); issue(1, 1); issue(2, 2);
      asm volatile("s_waitcnt vmcnt(32)" : "+v"(callv) :: "memory");
      issue(3, 3);
      BAR();
      i32x4 srdX = make_srd(myws + WS_X + ((rfl(callv) + 1) & 1) * XPAR, XPF_REMOTE ? XPAR : 0u);
      auto pf = [&](int t) {
        if (t < 64) {
          const unsigned so = __builtin_amdgcn_readlane(tabA, t), sw = __builtin_amdgcn_readlane(tabB, t);
          asm volatile("s_nop 4\n\tbuffer_load_dword %0, %1, %2, %3 offen" : "+v"(dummy) : "v"(vW), "s"(srdWp), "s"(sw) : "memory");
          if (lane < 16) {
            if (t < 8) asm volatile("buffer_load_dword %0, %1, %2, %3 offen sc0 sc1" : "+v"(dummy) : "v"(vXL), "s"(srdXL), "s"(so) : "memory");
            else if (XPF_REMOTE) asm volatile("buffer_load_dword %0, %1, %2, %3 offen sc0 sc1" : "+v"(dummy) : "v"(vX), "s"(srdX), "s"(so) : "memory");
          }
        } };
      int nxt = (int)(idx % PFK);
      for (int t = 4; t < PFD + nxt; ++t) pf(t);
      for (int u = 0; u < 64; ++u) {
        asm volatile("s_waitcnt vmcnt(32)" ::: "memory"); BAR();
        issue(u + 4, u & 3);
        if (u == nxt) {
          nxt += PFK;
          #pragma unroll
          for (int j = 0; j < PFK; ++j) pf(u + PFD + j);
        }
      }
      asm volatile("s_waitcnt vmcnt(0)" :: "v"(dummy) : "memory");
    }
    return;
  }
  const unsigned wm = wid >> 1, wn = wid & 1, fr = lane & 15, fq = lane >> 4, q = (lane & 15) >> 2, p = lane & 3;
  const unsigned ra0 = A_RING + (wm * 128 + fr) * 128 + ((fq ^ (fr & 7)) * 16);
  const unsigned rb00 = B_RING + (8 * fq + q) * 256 + (((wn * 8 + p) ^ ((q << 2) | (fq & 1))) * 16) + (p & 1) * 8;
  const unsigned call = rfl(ld_agent(myws + WS_CTR + b * 128)) + 1;
  unsigned tmov = 0;
  int ulast = 62; asm volatile("" : "+s"(ulast));
  f32x4 acc[8][4];
  #pragma unroll
  for (int m = 0; m < 8; ++m)
    #pragma unroll
    for (int n = 0; n < 4; ++n) acc[m][n] = (f32x4){0.f, 0.f, 0.f, 0.f};
  bf16x8 af[8]; s16x4 b0l[4], b0h[4], b1l[4], b1h[4];
  #define MF(m, n, L, H) do { acc[m][n] = __builtin_amdgcn_mfma_f32_16x16x32_bf16(__builtin_bit_cast(bf16x8, __builtin_shufflevector(L[n], H[n], 0, 1, 2, 3, 4, 5, 6, 7)), af[m], acc[m][n], 0, 0, 0); \
                              asm volatile("" : "+v"(acc[m][n])); SB(); } while (0)
  #define RA(dst, base, m) do { dst = rda((base) + (m) * 2048); SB(); } while (0)
  #define RL(L, n, ks) do { L[n] = trd(bb[n] + (ks) * 8192); SB(); } while (0)
  #define RH(H, n, ks) do { H[n] = trd(bb[n] + (ks) * 8192 + 1024); SB(); } while (0)
  #define PINV(x) do { asm volatile("" : "+v"(x)); SB(); } while (0)
  #define PINS(x) do { asm volatile("" : "+s"(x)); SB(); } while (0)
  unsigned bB[4], bb[4];
  #pragma unroll
  for (int n = 0; n < 4; ++n) { bB[n] = rb00 ^ ((n >> 1) * 64 + (n & 1) * 8); bb[n] = bB[n]; }
  unsigned a0 = ra0, a1 = ra0 ^ 64;
  BAR();
  #pragma unroll
  for (int n = 0; n < 4; ++n) { RL(b0l, n, 0); RH(b0h, n, 0); }
  #pragma unroll
  for (int m = 0; m < 7; ++m) RA(af[m], a0, m);
  unsigned sa = 0, sb = 0;
  for (int u = 0; u < 64; ++u) {
    MF(0, 0, b0l, b0h); RA(af[7], a0, 7); MF(0, 1, b0l, b0h); RL(b1l, 0, 1); MF(0, 2, b0l, b0h); RH(b1h, 0, 1); MF(0, 3, b0l, b0h);
    #pragma unroll
    for (int g = 1; g < 4; ++g) {
      MF(g, 0, b0l, b0h); RA(af[g - 1], a1, g - 1); MF(g, 1, b0l, b0h); RL(b1l, g, 1); MF(g, 2, b0l, b0h); RH(b1h, g, 1); MF(g, 3, b0l, b0h);
      if (g == 1) { sb = sb + B_ST; PINS(sb); }
      if (g == 2) { sb = sb & (4 * B_ST - 1); PINS(sb); }
    }
    #pragma unroll
    for (int g = 4; g < 8; ++g) {
      MF(g, 0, b0l, b0h); RA(af[g - 1], a1, g - 1); MF(g, 1, b0l, b0h); bb[g - 4] = bB[g - 4] + sb; PINV(bb[g - 4]); MF(g, 2, b0l, b0h);
      if (g == 4) { sa = sa == 2 * A_ST ? 0 : sa + A_ST; PINS(sa); }
      if (g == 5) { a0 = ra0 + sa; PINV(a0); }
      MF(g, 3, b0l, b0h);
    }
    MF(0, 0, b1l, b1h); RA(af[7], a1, 7); MF(0, 1, b1l, b1h); MF(0, 2, b1l, b1h); a1 = a0 ^ 64; PINV(a1); MF(0, 3, b1l, b1h);
    MF(1, 0, b1l, b1h); MF(1, 1, b1l, b1h);
    if (u == ulast) tmov = ld_agent(myws + WS_TMO);
    SB();
    MF(1, 2, b1l, b1h); MF(1, 3, b1l, b1h);
    asm volatile("s_waitcnt lgkmcnt(0)\n\ts_barrier" ::: "memory"); SB();
    MF(2, 0, b1l, b1h); RA(af[0], a0, 0); MF(2, 1, b1l, b1h); RL(b0l, 0, 0); MF(2, 2, b1l, b1h); RH(b0h, 0, 0); MF(2, 3, b1l, b1h); RA(af[1], a0, 1);
    #pragma unroll
    for (int g = 3; g < 8; ++g) {
      MF(g, 0, b1l, b1h); RA(af[g - 1], a0, g - 1); MF(g, 1, b1l, b1h); if (g < 6) RL(b0l, g - 2, 0); MF(g, 2, b1l, b1h); if (g < 6) RH(b0h, g - 2, 0); MF(g, 3, b1l, b1h);
    }
  }
  const bool bad = tmov == call;
  char* orow = P.out + ((size_t)(tm * 256 + wm * 128 + fr) * NTOT + tn * 128 + wn * 64 + fq * 8) * 2;
  const bool odd = fq & 1;
  #pragma unroll
  for (int m = 0; m < 8; ++m)
    #pragma unroll
    for (int pr = 0; pr < 2; ++pr) {
      const f32x4 a = acc[m][2 * pr], c = acc[m][2 * pr + 1];
      const unsigned a0 = relu2(cvtpk(a[0], a[1])), a1 = relu2(cvtpk(a[2], a[3])), c0 = relu2(cvtpk(c[0], c[1])), c1 = relu2(cvtpk(c[2], c[3]));
      u32x4 v; v[0] = odd ? c0 : a0; v[1] = odd ? c1 : a1; v[2] = odd ? a0 : c0; v[3] = odd ? a1 : c1;
      if (bad) v = (u32x4){0x7fc07fc0u, 0x7fc07fc0u, 0x7fc07fc0u, 0x7fc07fc0u};
      char* pp = orow + (size_t)m * 16 * NTOT * 2 + pr * 64;
      asm volatile("global_store_dwordx4 %0, %1, off sc1\n\ts_nop 1" :: "v"(pp), "v"(v) : "memory");
    }
  if (tid == 0) __hip_atomic_store((unsigned*)(myws + WS_CTR + b * 128), call, __ATOMIC_RELAXED, __HIP_MEMORY_SCOPE_AGENT);
}

extern "C" void kernel_launch(int rank, int nranks, ncclComm_t comm, hipStream_t stream,
                              void* const* d_in, const int* in_sizes, int n_in,
                              void* d_out, int out_size, const int* in_dtypes, int out_dtype,
                              void* const* d_out_all, void* const* d_ws_all, size_t ws_bytes) {
  (void)comm; (void)in_sizes; (void)n_in; (void)out_size; (void)in_dtypes; (void)out_dtype; (void)d_out_all;
  if (nranks != NR || (unsigned)rank >= (unsigned)NR || ws_bytes < WS_NEED) { fprintf(stderr, "unexpected launch geometry\n"); return; }
  static bool init = false;
  if (!init) {
    hipError_t e = hipFuncSetAttribute((const void*)a2a_gemm_relu, hipFuncAttributeMaxDynamicSharedMemorySize, (int)LDS_BYTES);
    if (e != hipSuccess) { fprintf(stderr, "hipFuncSetAttribute: %s\n", hipGetErrorString(e)); return; }
    init = true;
  }
  Params p;
  p.x = (const char*)d_in[0]; p.w = (const char*)d_in[1]; p.out = (char*)d_out;
  for (int i = 0; i < NR; ++i) p.ws[i] = (char*)d_ws_all[i];
  p.rank = rank; p.pad = 0;
  hipLaunchKernelGGL(a2a_gemm_relu, dim3(256), dim3(512), LDS_BYTES, stream, p);
  hipError_t e = hipGetLastError();
  if (e != hipSuccess) fprintf(stderr, "launch: %s\n", hipGetErrorString(e));
}
```

```cpp
#include <hip/hip_runtime.h>
#include <rccl/rccl.h>
#include <cstdio>
#include <cstdint>

typedef __bf16 bf16x8 __attribute__((ext_vector_type(8)));
typedef __bf16 bf16x2 __attribute__((ext_vector_type(2)));
typedef short s16x4 __attribute__((ext_vector_type(4)));
typedef short s16x2 __attribute__((ext_vector_type(2)));
typedef float f32x4 __attribute__((ext_vector_type(4)));
typedef float f32x2 __attribute__((ext_vector_type(2)));
typedef unsigned u32x4 __attribute__((ext_vector_type(4)));
typedef int i32x4 __attribute__((ext_vector_type(4)));
#define LAS __attribute__((address_space(3)))
#define DI __device__ __forceinline__
#define SB() __builtin_amdgcn_sched_barrier(0)
#define BAR() asm volatile("s_barrier" ::: "memory")

#define XPD 6
#define XPK 1
#define XPF_REMOTE 0
#define BSL 9
#define PT0 170
#define PT1 30
#define PSU 0
#define SHIFT 0
constexpr int NR = 8, KTOT = 4096, NTOT = 16384;
constexpr unsigned A_RING = 0, A_ST = 32768, B_RING = 3 * 32768, B_ST = 16384, LDS_BYTES = 163840;
constexpr unsigned WS_FLAGS = 0;
constexpr unsigned WS_TMO = 4096;
constexpr unsigned WS_CTR = 8192;
constexpr unsigned WS_X = 1u << 20;
constexpr unsigned BLK = 524288;
constexpr unsigned XPAR = 8 * BLK;
constexpr size_t WS_NEED = (size_t)WS_X + 2 * (size_t)XPAR;
constexpr unsigned long long DEADLINE_TICKS = 100000000ull;

#define STAMP(i, v) do {} while (0)
struct Params { const char* x; const char* w; char* out; char* ws[NR]; int rank; int pad; };

DI unsigned rfl(unsigned x) { return __builtin_amdgcn_readfirstlane(x); }
DI unsigned cvtpk(float lo, float hi) { f32x2 v = {lo, hi}; bf16x2 b = __builtin_convertvector(v, bf16x2); return __builtin_bit_cast(unsigned, b); }
DI unsigned relu2(unsigned x) { const s16x2 v = __builtin_bit_cast(s16x2, x), z = {0, 0}; return __builtin_bit_cast(unsigned, __builtin_elementwise_max(v, z)); }
DI i32x4 make_srd(const void* p, unsigned bytes) {
  unsigned long long a = (unsigned long long)p;
  i32x4 r; r[0] = rfl((unsigned)a); r[1] = rfl((unsigned)(a >> 32)) & 0xffff; r[2] = rfl(bytes); r[3] = 0x00020000; return r;
}
template <int SC, int PAD> DI void dma4(unsigned m0v, unsigned v0, unsigned v1, unsigned v2, unsigned v3, i32x4 srd, unsigned so) {
  if constexpr (SC)
    asm volatile("s_mov_b32 m0, %0\n\ts_nop %7\n\t"
                 "buffer_load_dwordx4 %1, %5, %6 offen sc0 sc1 lds\n\t"
                 "buffer_load_dwordx4 %2, %5, %6 offen offset:1024 sc0 sc1 lds\n\t"
                 "buffer_load_dwordx4 %3, %5, %6 offen offset:2048 sc0 sc1 lds\n\t"
                 "buffer_load_dwordx4 %4, %5, %6 offen offset:3072 sc0 sc1 lds"
                 :: "s"(m0v), "v"(v0), "v"(v1), "v"(v2), "v"(v3), "s"(srd), "s"(so), "i"(PAD) : "memory");
  else
    asm volatile("s_mov_b32 m0, %0\n\ts_nop %7\n\t"
                 "buffer_load_dwordx4 %1, %5, %6 offen lds\n\t"
                 "buffer_load_dwordx4 %2, %5, %6 offen offset:1024 lds\n\t"
                 "buffer_load_dwordx4 %3, %5, %6 offen offset:2048 lds\n\t"
                 "buffer_load_dwordx4 %4, %5, %6 offen offset:3072 lds"
                 :: "s"(m0v), "v"(v0), "v"(v1), "v"(v2), "v"(v3), "s"(srd), "s"(so), "i"(PAD) : "memory");
}
DI s16x4 trd(unsigned a) { return __builtin_amdgcn_ds_read_tr16_b64_v4i16((LAS s16x4*)(size_t)a); }
DI bf16x8 rda(unsigned a) { return *(const LAS bf16x8*)(size_t)a; }
DI unsigned ld_agent(const char* p) { return __hip_atomic_load((const unsigned*)p, __ATOMIC_RELAXED, __HIP_MEMORY_SCOPE_AGENT); }

__global__ __launch_bounds__(512, 2) void a2a_gemm_relu(const Params P) {
  const int tid = threadIdx.x, lane = tid & 63;
  const unsigned wid = rfl(tid >> 6);
  const unsigned b = blockIdx.x, xcd = b & 7, idx = b >> 3, tn = xcd * 16 + (idx >> 1), tm = idx & 1;
  const unsigned rank = (unsigned)P.rank & 7u;
  char* const myws = P.ws[rank];
  const unsigned colb = (tn >> 2) * 1024 + (tn & 3) * 128;

  if (wid == 7) {
    if (b >= 224) return;
    int nbar = 0;
    unsigned long long tb = __builtin_amdgcn_s_memrealtime() + (PT0 - PT1);
    auto drain = [&]() {
      while ((__builtin_amdgcn_s_getreg(7 | (31 << 11)) & 0xC0000Fu) != 0 && nbar < 60) {
        if ((long long)(__builtin_amdgcn_s_memrealtime() - tb) > PT1) { BAR(); ++nbar; tb = __builtin_amdgcn_s_memrealtime(); }
        else __builtin_amdgcn_s_sleep(1);
      }
      asm volatile("s_waitcnt vmcnt(0)" ::: "memory");
    };
    unsigned callv;
    u32x4 d[16];
    {
      const unsigned s = (b >> 5) + 1, row = (b & 31) * 16, dest = (rank - s) & 7u;
      const char* src = P.x + (size_t)dest * BLK + (size_t)(row + (lane >> 3)) * 1024 + (lane & 7) * 16;
      asm volatile("global_load_dword %0, %3, off sc1\n\tglobal_load_dwordx4 %1, %4, off\n\tglobal_load_dwordx4 %2, %5, off"
                   : "=&v"(callv), "=&v"(d[0]), "=&v"(d[1]) : "v"(myws + WS_CTR + b * 128), "v"(src), "v"(src + 8192) : "memory");
      char* const pws = P.ws[dest];
      drain();
      asm volatile("" : "+v"(callv), "+v"(d[0]), "+v"(d[1]));
      char* dst = pws + WS_X + ((rfl(callv) + 1) & 1) * XPAR + rank * BLK + row * 128 + lane * 16;
      asm volatile("global_store_dwordx4 %0, %1, off sc0 sc1\n\tglobal_store_dwordx4 %0, %2, off offset:1024 sc0 sc1\n\ts_nop 1" :: "v"(dst), "v"(d[0]), "v"(d[1]) : "memory");
      drain();
      if (lane == 0) __hip_atomic_fetch_add((unsigned*)(pws + WS_FLAGS + (row >> 7) * 256 + (8 + s - 1) * 4), 1u, __ATOMIC_RELAXED, __HIP_MEMORY_SCOPE_SYSTEM);
      STAMP(8, rp0); STAMP(9, RT()); STAMP(10, RT());
    }
    if (b >= 196) return;
    const unsigned call = rfl(callv) + 1;
    const unsigned kt = b / 28 + 1, r = b % 28, s = (r >> 2) + 1, qd = r & 3, dest = (rank - s) & 7u;
    const char* src = P.x + (size_t)dest * BLK + (size_t)(qd * 128 + (lane >> 3)) * 1024 + kt * 128 + (lane & 7) * 16;
    #pragma unroll
    for (int i = 0; i < 16; i += 4)
      asm volatile("global_load_dwordx4 %0, %4, off\n\tglobal_load_dwordx4 %1, %5, off\n\tglobal_load_dwordx4 %2, %6, off\n\tglobal_load_dwordx4 %3, %7, off"
                   : "=&v"(d[i]), "=&v"(d[i + 1]), "=&v"(d[i + 2]), "=&v"(d[i + 3])
                   : "v"(src + i * 8192), "v"(src + (i + 1) * 8192), "v"(src + (i + 2) * 8192), "v"(src + (i + 3) * 8192) : "memory");
    char* const pws = P.ws[dest];
    char* dst = pws + WS_X + (call & 1) * XPAR + rank * BLK + kt * 65536 + qd * 16384 + lane * 16;
    const int su = (int)(2 * kt + (qd >> 1)) + PSU;
    drain();
    #pragma unroll
    for (int i = 0; i < 16; i += 4) asm volatile("" : "+v"(d[i]), "+v"(d[i + 1]), "+v"(d[i + 2]), "+v"(d[i + 3]));
    for (; nbar < su; ++nbar) BAR();
    #pragma unroll
    for (int i = 0; i < 16; ++i)
      asm volatile("global_store_dwordx4 %0, %1, off sc0 sc1\n\ts_nop 1" :: "v"(dst + i * 1024), "v"(d[i]) : "memory");
    tb = __builtin_amdgcn_s_memrealtime() - PT1;
    drain();
    if (lane == 0)
      __hip_atomic_store((unsigned*)(pws + WS_FLAGS + qd * 256 + (8 + kt * 7 + s - 1) * 4), call * 8u, __ATOMIC_RELAXED, __HIP_MEMORY_SCOPE_SYSTEM);
    return;
  }
  if (wid >= 4) {
    unsigned tabA, tabB;
    unsigned pos = lane;
    if (tm) {
      if (SHIFT == 2 || (SHIFT == 1 && (tn & 1))) { if ((lane >= 2 && lane < 6) || (lane >= 10 && lane < 62)) pos = ((lane - 2) ^ 2) + 2; }
      else pos = lane ^ 2;
    }
    { const unsigned u = pos; unsigned src, kt;
      if (u < 8) { src = rank; kt = u; tabA = kt * 128; }
      else { const unsigned j = u - 8; kt = (j * 37) >> 8; src = (rank + (j - kt * 7 + 1)) & 7u; tabA = src * BLK + kt * 65536u; }
      tabB = (src * 512u + kt * 64u) * 32768u; }
    if (wid < 6) {
      const unsigned h = wid - 4;
      unsigned callv;
      asm volatile("global_load_dword %0, %1, off sc1" : "=v"(callv) : "v"(myws + WS_CTR + b * 128) : "memory");
      const i32x4 srdL = make_srd(P.x + (size_t)rank * BLK, BLK);
      const unsigned arow = tm * 256 + h * 128 + (lane >> 3), ach = ((lane & 7) ^ (lane >> 3)) * 16;
      unsigned vL[4];
      #pragma unroll
      for (int i = 0; i < 4; ++i) vL[i] = (arow + 8 * i) * 1024 + ach - i * 1024;
      const unsigned vR = arow * 128 + ach, m0A = A_RING + h * 16384;
      auto issueL = [&](int t, unsigned slot) {
        const unsigned so = __builtin_amdgcn_readlane(tabA, t), m = m0A + slot * A_ST;
        dma4<1, 3>(m, vL[0], vL[1], vL[2], vL[3], srdL, so);
        #pragma unroll
        for (int j = 1; j < 4; ++j) dma4<1, 0>(m + j * 4096, vL[0], vL[1], vL[2], vL[3], srdL, so + j * 32768);
      };
      issueL(0, 0); issueL(1, 1); issueL(2, 2);
      asm volatile("s_waitcnt vmcnt(32)" : "+v"(callv) :: "memory");
      BAR();
      const unsigned call = rfl(callv) + 1;
      i32x4 srdR = make_srd(myws + WS_X + (call & 1) * XPAR, XPAR);
      const char* fl = myws + WS_FLAGS + (tm * 2 + h) * 256 + pos * 4;
      const char* tmo = myws + WS_TMO;
      #define SAMPLE() asm volatile("global_load_dword v200, %0, off sc0 sc1" :: "v"(fl) : "memory", "v200")
      unsigned long long ready = 0xffull;
      unsigned slot = 0;
      for (int u = 0; u < 5; ++u) {
        asm volatile("s_waitcnt vmcnt(16)" ::: "memory"); BAR();
        if (u == 4) SAMPLE();
        issueL(u + 3, slot);
        slot = slot == 2 ? 0 : slot + 1;
      }
      for (int u = 5; u < 64; ++u) {
        const int t = u + 3;
        if (ready != ~0ull) {
          unsigned long long mk;
          asm volatile("s_waitcnt vmcnt(16)\n\ts_barrier\n\tv_cmp_ge_u32 %0, v200, %1" : "=s"(mk) : "v"(call * 8u) : "memory", "v200");
          ready |= mk;
          if (!((ready >> t) & 1)) {
            const unsigned long long t0 = __builtin_amdgcn_s_memrealtime();
            for (;;) {
              unsigned fv, tv;
              asm volatile("global_load_dword %0, %2, off sc0 sc1\n\tglobal_load_dword %1, %3, off sc1\n\ts_waitcnt vmcnt(0)"
                           : "=&v"(fv), "=&v"(tv) : "v"(fl), "v"(tmo) : "memory");
              ready |= __ballot(fv >= call * 8u);
              if ((ready >> t) & 1) break;
              if (rfl(tv) == call || __builtin_amdgcn_s_memrealtime() - t0 > DEADLINE_TICKS) {
                if (lane == 0) __hip_atomic_store((unsigned*)tmo, call, __ATOMIC_RELAXED, __HIP_MEMORY_SCOPE_AGENT);
                asm volatile("s_waitcnt vmcnt(0)" ::: "memory");
                ready = ~0ull; srdR[2] = 0;
                break;
              }
              __builtin_amdgcn_s_sleep(2);
            }
          }
          ready |= (2ull << t) - 1;
          if (ready != ~0ull) SAMPLE();
        } else {
          asm volatile("s_waitcnt vmcnt(16)\n\ts_barrier" ::: "memory");
        }
        unsigned so = __builtin_amdgcn_readlane(tabA, t & 63);
        if (t >= 64) { srdR[2] = 0; so = 0; }
        const unsigned m = m0A + slot * A_ST;
        dma4<1, 3>(m, vR, vR, vR, vR, srdR, so);
        #pragma unroll
        for (int j = 1; j < 4; ++j) dma4<1, 0>(m + j * 4096, vR, vR, vR, vR, srdR, so + j * 4096);
        slot = slot == 2 ? 0 : slot + 1;
      }
      STAMP(5 + 6 * h, spin); STAMP(6 + 6 * h, nspin); STAMP(7 + 6 * h, firstspin);
    } else {
      i32x4 srdW = make_srd(P.w, (unsigned)KTOT * NTOT * 2u);
      unsigned vB[4];
      #pragma unroll
      for (int i = 0; i < 4; ++i) {
        const unsigned ch = (lane & 15) ^ (((lane >> 4) << 2) | (i >> 1));
        vB[i] = (4 * i + (lane >> 4)) * 32768 + colb + (ch >> 3) * 512 + (ch & 7) * 16 - i * 1024;
      }
      auto issue = [&](int t, unsigned slot) {
        const unsigned so = __builtin_amdgcn_readlane(tabB, t & 63), m = B_RING + slot * B_ST;
        if (t >= 64) srdW[2] = 0;
        dma4<0, 3>(m, vB[0], vB[1], vB[2], vB[3], srdW, so);
        #pragma unroll
        for (int j = 1; j < 4; ++j) dma4<0, 0>(m + j * 4096, vB[0], vB[1], vB[2], vB[3], srdW, so + j * 16 * 32768);
      };
      unsigned callv, dummy = 0;
      asm volatile("global_load_dword %0, %1, off sc1" : "=v"(callv) : "v"(myws + WS_CTR + b * 128) : "memory");
      const i32x4 srdXL = make_srd(P.x + (size_t)rank * BLK, BLK);
      const unsigned xr = tm * 256 + (idx >> 1) * 16 + (lane & 15), vX = xr * 128, vXL = xr * 1024;
      issue(0, 0); issue(1, 1); issue(2, 2);
      asm volatile("s_waitcnt vmcnt(32)" : "+v"(callv) :: "memory");
      issue(3, 3);
      BAR();
      const i32x4 srdX = make_srd(myws + WS_X + ((rfl(callv) + 1) & 1) * XPAR, XPAR);
      auto pf = [&](int t) {
        if (t < 64 && lane < 16) {
          const unsigned so = __builtin_amdgcn_readlane(tabA, t);
          if (t < 8) asm volatile("s_nop 4\n\tbuffer_load_dword %0, %1, %2, %3 offen sc0 sc1" : "+v"(dummy) : "v"(vXL), "s"(srdXL), "s"(so) : "memory");
          else if (XPF_REMOTE) asm volatile("s_nop 4\n\tbuffer_load_dword %0, %1, %2, %3 offen sc0 sc1" : "+v"(dummy) : "v"(vX), "s"(srdX), "s"(so) : "memory");
        } };
      const int ph = (SHIFT && (tn & 1) && XPK > 2) ? 2 : 0;
      for (int t = 3; t < XPD + ph; ++t) pf(t);
      for (int u = 0; u < 64; ++u) {
        asm volatile("s_waitcnt vmcnt(32)" ::: "memory"); BAR();
        if (BSL) __builtin_amdgcn_s_sleep(BSL);
        if (((u - ph) & (XPK - 1)) == 0 && u >= ph) {
          #pragma unroll
          for (int j = 0; j < XPK; ++j) pf(u + XPD + j);
        }
        issue(u + 4, u & 3);
      }
      asm volatile("s_waitcnt vmcnt(0)" :: "v"(dummy) : "memory");
    }
    return;
  }
  const unsigned wm = wid >> 1, wn = wid & 1, fr = lane & 15, fq = lane >> 4, q = (lane & 15) >> 2, p = lane & 3;
  const unsigned ra0 = A_RING + (wm * 128 + fr) * 128 + ((fq ^ (fr & 7)) * 16);
  const unsigned rb00 = B_RING + (8 * fq + q) * 256 + (((wn * 8 + p) ^ ((q << 2) | (fq & 1))) * 16) + (p & 1) * 8;
  const unsigned call = rfl(ld_agent(myws + WS_CTR + b * 128)) + 1;
  unsigned tmov = 0;
  int ulast = 62; asm volatile("" : "+s"(ulast));
  f32x4 acc[8][4];
  #pragma unroll
  for (int m = 0; m < 8; ++m)
    #pragma unroll
    for (int n = 0; n < 4; ++n) acc[m][n] = (f32x4){0.f, 0.f, 0.f, 0.f};
  bf16x8 af[8]; s16x4 b0l[4], b0h[4], b1l[4], b1h[4];
  #define MF(m, n_, L, H) MF2(m, (((m) & 1) ? 3 - (n_) : (n_)), L, H)
  #define MF2(m, n, L, H) do { acc[m][n] = __builtin_amdgcn_mfma_f32_16x16x32_bf16(__builtin_bit_cast(bf16x8, __builtin_shufflevector(L[n], H[n], 0, 1, 2, 3, 4, 5, 6, 7)), af[m], acc[m][n], 0, 0, 0); \
                              asm volatile("" : "+v"(acc[m][n])); SB(); } while (0)
  #define RA(dst, base, m) do { dst = rda((base) + (m) * 2048); SB(); } while (0)
  #define RL(L, n, ks) do { L[n] = trd(bb[n] + (ks) * 8192); SB(); } while (0)
  #define RH(H, n, ks) do { H[n] = trd(bb[n] + (ks) * 8192 + 1024); SB(); } while (0)
  #define PINV(x) do { asm volatile("" : "+v"(x)); SB(); } while (0)
  #define PINS(x) do { asm volatile("" : "+s"(x)); SB(); } while (0)
  unsigned bB[4], bb[4];
  #pragma unroll
  for (int n = 0; n < 4; ++n) { bB[n] = rb00 ^ ((n >> 1) * 64 + (n & 1) * 8); bb[n] = bB[n]; }
  unsigned a0 = ra0, a1 = ra0 ^ 64;
  BAR();
  #pragma unroll
  for (int n = 0; n < 4; ++n) { RL(b0l, n, 0); RH(b0h, n, 0); }
  #pragma unroll
  for (int m = 0; m < 7; ++m) RA(af[m], a0, m);
  unsigned sa = 0, sb = 0;
  for (int u = 0; u < 64; ++u) {
    MF(0, 0, b0l, b0h); RA(af[7], a0, 7); MF(0, 1, b0l, b0h); RL(b1l, 0, 1); MF(0, 2, b0l, b0h); RH(b1h, 0, 1); MF(0, 3, b0l, b0h);
    #pragma unroll
    for (int g = 1; g < 4; ++g) {
      MF(g, 0, b0l, b0h); RA(af[g - 1], a1, g - 1); MF(g, 1, b0l, b0h); RL(b1l, g, 1); MF(g, 2, b0l, b0h); RH(b1h, g, 1); MF(g, 3, b0l, b0h);
      if (g == 1) { sb = sb + B_ST; PINS(sb); }
      if (g == 2) { sb = sb & (4 * B_ST - 1); PINS(sb); }
    }
    #pragma unroll
    for (int g = 4; g < 8; ++g) {
      MF(g, 0, b0l, b0h); RA(af[g - 1], a1, g - 1); MF(g, 1, b0l, b0h); bb[g - 4] = bB[g - 4] + sb; PINV(bb[g - 4]); MF(g, 2, b0l, b0h);
      if (g == 4) { sa = sa == 2 * A_ST ? 0 : sa + A_ST; PINS(sa); }
      if (g == 5) { a0 = ra0 + sa; PINV(a0); }
      MF(g, 3, b0l, b0h);
    }
    MF(0, 0, b1l, b1h); RA(af[7], a1, 7); MF(0, 1, b1l, b1h); MF(0, 2, b1l, b1h); a1 = a0 ^ 64; PINV(a1); MF(0, 3, b1l, b1h);
    MF(1, 0, b1l, b1h); MF(1, 1, b1l, b1h);
    if (u == ulast) tmov = ld_agent(myws + WS_TMO);
    SB();
    MF(1, 2, b1l, b1h); MF(1, 3, b1l, b1h);
    asm volatile("s_waitcnt lgkmcnt(0)\n\ts_barrier" ::: "memory"); SB();
    MF(2, 0, b1l, b1h); RA(af[0], a0, 0); MF(2, 1, b1l, b1h); RL(b0l, 0, 0); MF(2, 2, b1l, b1h); RH(b0h, 0, 0); MF(2, 3, b1l, b1h); RA(af[1], a0, 1);
    #pragma unroll
    for (int g = 3; g < 8; ++g) {
      MF(g, 0, b1l, b1h); RA(af[g - 1], a0, g - 1); MF(g, 1, b1l, b1h); if (g < 6) RL(b0l, g - 2, 0); MF(g, 2, b1l, b1h); if (g < 6) RH(b0h, g - 2, 0); MF(g, 3, b1l, b1h);
    }
  }
  const bool bad = tmov == call;
  char* orow = P.out + (size_t)(tm * 256 + wm * 128 + fr) * NTOT * 2 + colb + wn * 512 + fq * 16;
  const bool odd = fq & 1;
  #pragma unroll
  for (int m = 0; m < 8; ++m)
    #pragma unroll
    for (int pr = 0; pr < 2; ++pr) {
      const f32x4 a = acc[m][2 * pr], c = acc[m][2 * pr + 1];
      const unsigned a0 = relu2(cvtpk(a[0], a[1])), a1 = relu2(cvtpk(a[2], a[3])), c0 = relu2(cvtpk(c[0], c[1])), c1 = relu2(cvtpk(c[2], c[3]));
      u32x4 v; v[0] = odd ? c0 : a0; v[1] = odd ? c1 : a1; v[2] = odd ? a0 : c0; v[3] = odd ? a1 : c1;
      if (bad) v = (u32x4){0x7fc07fc0u, 0x7fc07fc0u, 0x7fc07fc0u, 0x7fc07fc0u};
      char* pp = orow + (size_t)m * 16 * NTOT * 2 + pr * 64;
      asm volatile("global_store_dwordx4 %0, %1, off sc1\n\ts_nop 1" :: "v"(pp), "v"(v) : "memory");
    }
  if (tid == 0) __hip_atomic_store((unsigned*)(myws + WS_CTR + b * 128), call, __ATOMIC_RELAXED, __HIP_MEMORY_SCOPE_AGENT);
}

extern "C" void kernel_launch(int rank, int nranks, ncclComm_t comm, hipStream_t stream,
                              void* const* d_in, const int* in_sizes, int n_in,
                              void* d_out, int out_size, const int* in_dtypes, int out_dtype,
                              void* const* d_out_all, void* const* d_ws_all, size_t ws_bytes) {
  (void)comm; (void)in_sizes; (void)n_in; (void)out_size; (void)in_dtypes; (void)out_dtype; (void)d_out_all;
  if (nranks != NR || (unsigned)rank >= (unsigned)NR || ws_bytes < WS_NEED) { fprintf(stderr, "unexpected launch geometry\n"); return; }
  static bool init = false;
  if (!init) {
    hipError_t e = hipFuncSetAttribute((const void*)a2a_gemm_relu, hipFuncAttributeMaxDynamicSharedMemorySize, (int)LDS_BYTES);
    if (e != hipSuccess) { fprintf(stderr, "hipFuncSetAttribute: %s\n", hipGetErrorString(e)); return; }
    init = true;
  }
  Params p;
  p.x = (const char*)d_in[0]; p.w = (const char*)d_in[1]; p.out = (char*)d_out;
  for (int i = 0; i < NR; ++i) p.ws[i] = (char*)d_ws_all[i];
  p.rank = rank; p.pad = 0;
  hipLaunchKernelGGL(a2a_gemm_relu, dim3(256), dim3(512), LDS_BYTES, stream, p);
  hipError_t e = hipGetLastError();
  if (e != hipSuccess) fprintf(stderr, "launch: %s\n", hipGetErrorString(e));
}
```

```cpp
#include <hip/hip_runtime.h>
#include <rccl/rccl.h>
#include <cstdio>
#include <cstdint>

typedef __bf16 bf16x8 __attribute__((ext_vector_type(8)));
typedef __bf16 bf16x2 __attribute__((ext_vector_type(2)));
typedef short s16x4 __attribute__((ext_vector_type(4)));
typedef short s16x2 __attribute__((ext_vector_type(2)));
typedef float f32x4 __attribute__((ext_vector_type(4)));
typedef float f32x2 __attribute__((ext_vector_type(2)));
typedef unsigned u32x4 __attribute__((ext_vector_type(4)));
typedef int i32x4 __attribute__((ext_vector_type(4)));
#define LAS __attribute__((address_space(3)))
#define DI __device__ __forceinline__
#define SB() __builtin_amdgcn_sched_barrier(0)
#define BAR() asm volatile("s_barrier" ::: "memory")

#define XPD 6
#define XPK 1
#define XPF_REMOTE 0
#define BSL 9
#define PT0 120
#define PT1 30
#define PSU 0
#define PSP 4
#define LOOK 3
#define WPF 0
#define PERM 11
#define FULL_LINE 1
#define OUT_POL "sc1"
#define W_POL ""
constexpr int NR = 8, KTOT = 4096, NTOT = 16384;
constexpr unsigned A_RING = 0, A_ST = 32768, B_RING = 3 * 32768, B_ST = 16384, LDS_BYTES = 163840;
constexpr unsigned WS_FLAGS = 0;
constexpr unsigned WS_TMO = 4096;
constexpr unsigned WS_CTR = 8192;
constexpr unsigned WS_X = 1u << 20;
constexpr unsigned BLK = 524288;
constexpr unsigned XPAR = 8 * BLK;
constexpr size_t WS_NEED = (size_t)WS_X + 2 * (size_t)XPAR;
constexpr unsigned long long DEADLINE_TICKS = 100000000ull;

#define STAMP(i, v) do {} while (0)
struct Params { const char* x; const char* w; char* out; char* ws[NR]; int rank; int pad; };

DI unsigned rfl(unsigned x) { return __builtin_amdgcn_readfirstlane(x); }
DI unsigned cvtpk(float lo, float hi) { f32x2 v = {lo, hi}; bf16x2 b = __builtin_convertvector(v, bf16x2); return __builtin_bit_cast(unsigned, b); }
DI unsigned relu2(unsigned x) { const s16x2 v = __builtin_bit_cast(s16x2, x), z = {0, 0}; return __builtin_bit_cast(unsigned, __builtin_elementwise_max(v, z)); }
DI i32x4 make_srd(const void* p, unsigned bytes) {
  unsigned long long a = (unsigned long long)p;
  i32x4 r; r[0] = rfl((unsigned)a); r[1] = rfl((unsigned)(a >> 32)) & 0xffff; r[2] = rfl(bytes); r[3] = 0x00020000; return r;
}
template <int SC, int PAD> DI void dma4(unsigned m0v, unsigned v0, unsigned v1, unsigned v2, unsigned v3, i32x4 srd, unsigned so) {
  if constexpr (SC)
    asm volatile("s_mov_b32 m0, %0\n\ts_nop %7\n\t"
                 "buffer_load_dwordx4 %1, %5, %6 offen sc0 sc1 lds\n\t"
                 "buffer_load_dwordx4 %2, %5, %6 offen offset:1024 sc0 sc1 lds\n\t"
                 "buffer_load_dwordx4 %3, %5, %6 offen offset:2048 sc0 sc1 lds\n\t"
                 "buffer_load_dwordx4 %4, %5, %6 offen offset:3072 sc0 sc1 lds"
                 :: "s"(m0v), "v"(v0), "v"(v1), "v"(v2), "v"(v3), "s"(srd), "s"(so), "i"(PAD) : "memory");
  else
    asm volatile("s_mov_b32 m0, %0\n\ts_nop %7\n\t"
                 "buffer_load_dwordx4 %1, %5, %6 offen " W_POL " lds\n\t"
                 "buffer_load_dwordx4 %2, %5, %6 offen offset:1024 " W_POL " lds\n\t"
                 "buffer_load_dwordx4 %3, %5, %6 offen offset:2048 " W_POL " lds\n\t"
                 "buffer_load_dwordx4 %4, %5, %6 offen offset:3072 " W_POL " lds"
                 :: "s"(m0v), "v"(v0), "v"(v1), "v"(v2), "v"(v3), "s"(srd), "s"(so), "i"(PAD) : "memory");
}
DI s16x4 trd(unsigned a) { return __builtin_amdgcn_ds_read_tr16_b64_v4i16((LAS s16x4*)(size_t)a); }
DI bf16x8 rda(unsigned a) { return *(const LAS bf16x8*)(size_t)a; }
DI unsigned ld_agent(const char* p) { return __hip_atomic_load((const unsigned*)p, __ATOMIC_RELAXED, __HIP_MEMORY_SCOPE_AGENT); }

__global__ __launch_bounds__(512, 2) void a2a_gemm_relu(const Params P) {
  const int tid = threadIdx.x, lane = tid & 63;
  const unsigned wid = rfl(tid >> 6);
  const unsigned b = blockIdx.x, xcd = b & 7, idx = b >> 3, tn = xcd * 16 + (idx >> 1), tm = idx & 1;
  const unsigned rank = (unsigned)P.rank & 7u;
  char* const myws = P.ws[rank];
  const unsigned colb = (tn >> 2) * 1024 + (tn & 3) * 128;

  if (wid == 7) {
    if (b >= 224) return;
    int nbar = 0;
    unsigned long long tb = __builtin_amdgcn_s_memrealtime() + (PT0 - PT1);
    auto drain = [&]() {
      while ((__builtin_amdgcn_s_getreg(7 | (31 << 11)) & 0xC0000Fu) != 0 && nbar < 60) {
        if ((long long)(__builtin_amdgcn_s_memrealtime() - tb) > PT1) { BAR(); ++nbar; tb = __builtin_amdgcn_s_memrealtime(); }
        else __builtin_amdgcn_s_sleep(1);
      }
      asm volatile("s_waitcnt vmcnt(0)" ::: "memory");
    };
    unsigned callv;
    u32x4 d[16];
    {
      const unsigned s = (b >> 5) + 1, row = (b & 31) * 16, dest = (rank - s) & 7u;
      const char* src = P.x + (size_t)dest * BLK + (size_t)(row + (lane >> 3)) * 1024 + (lane & 7) * 16;
      asm volatile("global_load_dword %0, %3, off sc1\n\tglobal_load_dwordx4 %1, %4, off\n\tglobal_load_dwordx4 %2, %5, off"
                   : "=&v"(callv), "=&v"(d[0]), "=&v"(d[1]) : "v"(myws + WS_CTR + b * 128), "v"(src), "v"(src + 8192) : "memory");
      char* const pws = P.ws[dest];
      drain();
      asm volatile("" : "+v"(callv), "+v"(d[0]), "+v"(d[1]));
      char* dst = pws + WS_X + ((rfl(callv) + 1) & 1) * XPAR + rank * BLK + row * 128 + lane * 16;
      asm volatile("global_store_dwordx4 %0, %1, off sc0 sc1\n\tglobal_store_dwordx4 %0, %2, off offset:1024 sc0 sc1\n\ts_nop 1" :: "v"(dst), "v"(d[0]), "v"(d[1]) : "memory");
      drain();
      if (lane == 0) __hip_atomic_fetch_add((unsigned*)(pws + WS_FLAGS + (row >> 7) * 256 + (8 + s - 1) * 4), 1u, __ATOMIC_RELAXED, __HIP_MEMORY_SCOPE_SYSTEM);
      STAMP(8, rp0); STAMP(9, RT()); STAMP(10, RT());
    }
    if (b >= 196) return;
    const unsigned call = rfl(callv) + 1;
    const unsigned kt = b / 28 + 1, r = b % 28, s = (r >> 2) + 1, qd = r & 3, dest = (rank - s) & 7u;
    const char* src = P.x + (size_t)dest * BLK + (size_t)(qd * 128 + (lane >> 3)) * 1024 + kt * 128 + (lane & 7) * 16;
    #pragma unroll
    for (int i = 0; i < 16; i += 4)
      asm volatile("global_load_dwordx4 %0, %4, off\n\tglobal_load_dwordx4 %1, %5, off\n\tglobal_load_dwordx4 %2, %6, off\n\tglobal_load_dwordx4 %3, %7, off"
                   : "=&v"(d[i]), "=&v"(d[i + 1]), "=&v"(d[i + 2]), "=&v"(d[i + 3])
                   : "v"(src + i * 8192), "v"(src + (i + 1) * 8192), "v"(src + (i + 2) * 8192), "v"(src + (i + 3) * 8192) : "memory");
    char* const pws = P.ws[dest];
    char* dst = pws + WS_X + (call & 1) * XPAR + rank * BLK + kt * 65536 + qd * 16384 + lane * 16;
    const int su = (PSP == 2 ? (int)(2 * kt + (qd >> 1)) : PSP == 4 ? (int)(4 * kt + qd) - 2 : (int)(3 * kt + (qd * 3 >> 2)) - 1) + PSU;
    drain();
    #pragma unroll
    for (int i = 0; i < 16; i += 4) asm volatile("" : "+v"(d[i]), "+v"(d[i + 1]), "+v"(d[i + 2]), "+v"(d[i + 3]));
    for (; nbar < su; ++nbar) BAR();
    #pragma unroll
    for (int i = 0; i < 16; ++i)
      asm volatile("global_store_dwordx4 %0, %1, off sc0 sc1\n\ts_nop 1" :: "v"(dst + i * 1024), "v"(d[i]) : "memory");
    tb = __builtin_amdgcn_s_memrealtime() - PT1;
    drain();
    if (lane == 0)
      __hip_atomic_store((unsigned*)(pws + WS_FLAGS + qd * 256 + (8 + kt * 7 + s - 1) * 4), call * 8u, __ATOMIC_RELAXED, __HIP_MEMORY_SCOPE_SYSTEM);
    return;
  }
  if (wid >= 4) {
    unsigned tabA, tabB;
    unsigned pos = lane;
    if (tm) {
      if (PERM < 8) pos = lane ^ PERM;
      else if (lane & 1) { if (lane >= (lane < 8 ? 0u : 8u) + (PERM - 8)) pos = lane - (PERM - 8); }
      else if (lane + (PERM - 8) < (lane < 8 ? 8u : 64u)) pos = lane + (PERM - 8);
    }
    { const unsigned u = pos; unsigned src, kt;
      if (u < 8) { src = rank; kt = u; tabA = kt * 128; }
      else { const unsigned j = u - 8; kt = (j * 37) >> 8; src = (rank + (j - kt * 7 + 1)) & 7u; tabA = src * BLK + kt * 65536u; }
      tabB = (src * 512u + kt * 64u) * 32768u; }
    if (wid < 6) {
      const unsigned h = wid - 4;
      unsigned callv;
      asm volatile("global_load_dword %0, %1, off sc1" : "=v"(callv) : "v"(myws + WS_CTR + b * 128) : "memory");
      const i32x4 srdL = make_srd(P.x + (size_t)rank * BLK, BLK);
      const unsigned arow = tm * 256 + h * 128 + (lane >> 3), ach = ((lane & 7) ^ (lane >> 3)) * 16;
      unsigned vL[4];
      #pragma unroll
      for (int i = 0; i < 4; ++i) vL[i] = (arow + 8 * i) * 1024 + ach - i * 1024;
      const unsigned vR = arow * 128 + ach, m0A = A_RING + h * 16384;
      auto issueL = [&](int t, unsigned slot) {
        const unsigned so = __builtin_amdgcn_readlane(tabA, t), m = m0A + slot * A_ST;
        dma4<1, 3>(m, vL[0], vL[1], vL[2], vL[3], srdL, so);
        #pragma unroll
        for (int j = 1; j < 4; ++j) dma4<1, 0>(m + j * 4096, vL[0], vL[1], vL[2], vL[3], srdL, so + j * 32768);
      };
      issueL(0, 0);
      issueL(1, 1);
      asm volatile("s_waitcnt vmcnt(16)" : "+v"(callv) :: "memory");
      BAR();
      issueL(2, 2);
      const unsigned call = rfl(callv) + 1;
      i32x4 srdR = make_srd(myws + WS_X + (call & 1) * XPAR, XPAR);
      const char* fl = myws + WS_FLAGS + (tm * 2 + h) * 256 + pos * 4;
      const char* tmo = myws + WS_TMO;
      #define SAMPLE() asm volatile("global_load_dword v200, %0, off sc0 sc1" :: "v"(fl) : "memory", "v200")
      unsigned long long ready = 0xffull;
      bool pend = true;
      unsigned slot = 0;
      for (int u = 0; u < 5; ++u) {
        asm volatile("s_waitcnt vmcnt(16)" ::: "memory"); BAR();
        if (u == 4) SAMPLE();
        issueL(u + 3, slot);
        slot = slot == 2 ? 0 : slot + 1;
      }
      for (int u = 5; u < 64; ++u) {
        const int t = u + 3;
        if (ready != ~0ull) {
          if (pend) {
            unsigned long long mk;
            asm volatile("s_waitcnt vmcnt(16)\n\ts_barrier\n\tv_cmp_ge_u32 %0, v200, %1" : "=s"(mk) : "v"(call * 8u) : "memory", "v200");
            ready |= mk;
          } else {
            asm volatile("s_waitcnt vmcnt(16)\n\ts_barrier" ::: "memory");
          }
          if (!((ready >> t) & 1)) {
            const unsigned long long t0 = __builtin_amdgcn_s_memrealtime();
            for (;;) {
              unsigned fv, tv;
              asm volatile("global_load_dword %0, %2, off sc0 sc1\n\tglobal_load_dword %1, %3, off sc1\n\ts_waitcnt vmcnt(0)"
                           : "=&v"(fv), "=&v"(tv) : "v"(fl), "v"(tmo) : "memory");
              ready |= __ballot(fv >= call * 8u);
              if ((ready >> t) & 1) break;
              if (rfl(tv) == call || __builtin_amdgcn_s_memrealtime() - t0 > DEADLINE_TICKS) {
                if (lane == 0) __hip_atomic_store((unsigned*)tmo, call, __ATOMIC_RELAXED, __HIP_MEMORY_SCOPE_AGENT);
                asm volatile("s_waitcnt vmcnt(0)" ::: "memory");
                ready = ~0ull; srdR[2] = 0;
                break;
              }
              __builtin_amdgcn_s_sleep(2);
            }
          }
          ready |= (2ull << t) - 1;
          pend = ((~ready >> t) & ((2ull << LOOK) - 1)) != 0;
          if (pend) SAMPLE();
        } else {
          asm volatile("s_waitcnt vmcnt(16)\n\ts_barrier" ::: "memory");
        }
        unsigned so = __builtin_amdgcn_readlane(tabA, t & 63);
        if (t >= 64) { srdR[2] = 0; so = 0; }
        const unsigned m = m0A + slot * A_ST;
        dma4<1, 3>(m, vR, vR, vR, vR, srdR, so);
        #pragma unroll
        for (int j = 1; j < 4; ++j) dma4<1, 0>(m + j * 4096, vR, vR, vR, vR, srdR, so + j * 4096);
        slot = slot == 2 ? 0 : slot + 1;
      }
      if (h == 0) STAMP(15, ((q0 - rt0) & 0xffff) | (((q1 - rt0) & 0xffff) << 16) | (((q2 - rt0) & 0xffff) << 32) | ((rt0 & 0xffff) << 48));
      STAMP(5 + 6 * h, spin); STAMP(6 + 6 * h, nspin); STAMP(7 + 6 * h, firstspin);
    } else {
      i32x4 srdW = make_srd(P.w, (unsigned)KTOT * NTOT * 2u);
      unsigned vB[4];
      #pragma unroll
      for (int i = 0; i < 4; ++i) {
        const unsigned ch = (lane & 15) ^ (((lane >> 4) << 2) | (i >> 1));
        vB[i] = (4 * i + (lane >> 4)) * 32768 + colb + (ch >> 3) * 512 + (ch & 7) * 16 - i * 1024;
      }
      auto issue = [&](int t, unsigned slot) {
        const unsigned so = __builtin_amdgcn_readlane(tabB, t & 63), m = B_RING + slot * B_ST;
        if (t >= 64) srdW[2] = 0;
        dma4<0, 3>(m, vB[0], vB[1], vB[2], vB[3], srdW, so);
        #pragma unroll
        for (int j = 1; j < 4; ++j) dma4<0, 0>(m + j * 4096, vB[0], vB[1], vB[2], vB[3], srdW, so + j * 16 * 32768);
      };
      unsigned callv, dummy = 0;
      asm volatile("global_load_dword %0, %1, off sc1" : "=v"(callv) : "v"(myws + WS_CTR + b * 128) : "memory");
      const i32x4 srdXL = make_srd(P.x + (size_t)rank * BLK, BLK);
      const unsigned xr = tm * 256 + (idx >> 1) * 16 + (lane & 15), vX = xr * 128, vXL = xr * 1024;
      issue(0, 0);
      issue(1, 1);
      asm volatile("s_waitcnt vmcnt(16)" : "+v"(callv) :: "memory");
      BAR();
      issue(2, 2); issue(3, 3);
      const i32x4 srdX = make_srd(myws + WS_X + ((rfl(callv) + 1) & 1) * XPAR, XPAR);
      const i32x4 srdWp = make_srd(P.w, (unsigned)KTOT * NTOT * 2u);
      const unsigned vW = (idx * 2 + (lane >> 5)) * 32768 + xcd * 4096 + (lane & 31) * 128;
      auto pf = [&](int t) {
        if (WPF && t < 64 && t >= 4) {
          const unsigned sw = __builtin_amdgcn_readlane(tabB, t);
          asm volatile("s_nop 4\n\tbuffer_load_dword %0, %1, %2, %3 offen" : "+v"(dummy) : "v"(vW), "s"(srdWp), "s"(sw) : "memory");
        }
        if (t < 64 && lane < 16) {
          const unsigned so = __builtin_amdgcn_readlane(tabA, t);
          if (t < 8) asm volatile("s_nop 4\n\tbuffer_load_dword %0, %1, %2, %3 offen sc0 sc1" : "+v"(dummy) : "v"(vXL), "s"(srdXL), "s"(so) : "memory");
          else if (XPF_REMOTE) asm volatile("s_nop 4\n\tbuffer_load_dword %0, %1, %2, %3 offen sc0 sc1" : "+v"(dummy) : "v"(vX), "s"(srdX), "s"(so) : "memory");
        } };
      int nxt = WPF ? (int)(idx % XPK) : 0;
      for (int t = 3; t < XPD + nxt; ++t) pf(t);
      for (int u = 0; u < 64; ++u) {
        asm volatile("s_waitcnt vmcnt(32)" ::: "memory"); BAR();
        if (BSL) __builtin_amdgcn_s_sleep(BSL);
        if (WPF) issue(u + 4, u & 3);
        if (u == nxt) {
          nxt += XPK;
          #pragma unroll
          for (int j = 0; j < XPK; ++j) pf(u + XPD + j);
        }
        if (!WPF) issue(u + 4, u & 3);
      }
      asm volatile("s_waitcnt vmcnt(0)" :: "v"(dummy) : "memory");
      STAMP(14, ((q0 - rt0) & 0xffff) | (((q1 - rt0) & 0xffff) << 16) | (((q2 - rt0) & 0xffff) << 32) | ((rt0 & 0xffff) << 48));
    }
    return;
  }
  const unsigned wm = wid >> 1, wn = wid & 1, fr = lane & 15, fq = lane >> 4, q = (lane & 15) >> 2, p = lane & 3;
  const unsigned ra0 = A_RING + (wm * 128 + fr) * 128 + ((fq ^ (fr & 7)) * 16);
  const unsigned rb00 = B_RING + (8 * fq + q) * 256 + (((wn * 8 + p) ^ ((q << 2) | (fq & 1))) * 16) + (p & 1) * 8;
  const unsigned call = rfl(ld_agent(myws + WS_CTR + b * 128)) + 1;
  unsigned tmov = 0;
  int ulast = 62; asm volatile("" : "+s"(ulast));
  f32x4 acc[8][4];
  #pragma unroll
  for (int m = 0; m < 8; ++m)
    #pragma unroll
    for (int n = 0; n < 4; ++n) acc[m][n] = (f32x4){0.f, 0.f, 0.f, 0.f};
  bf16x8 af[8]; s16x4 b0l[4], b0h[4], b1l[4], b1h[4];
  #define MF(m, n_, L, H) MF2(m, (((m) & 1) ? 3 - (n_) : (n_)), L, H)
  #define MF2(m, n, L, H) do { acc[m][n] = __builtin_amdgcn_mfma_f32_16x16x32_bf16(__builtin_bit_cast(bf16x8, __builtin_shufflevector(L[n], H[n], 0, 1, 2, 3, 4, 5, 6, 7)), af[m], acc[m][n], 0, 0, 0); \
                              asm volatile("" : "+v"(acc[m][n])); SB(); } while (0)
  #define RA(dst, base, m) do { dst = rda((base) + (m) * 2048); SB(); } while (0)
  #define RL(L, n, ks) do { L[n] = trd(bb[n] + (ks) * 8192); SB(); } while (0)
  #define RH(H, n, ks) do { H[n] = trd(bb[n] + (ks) * 8192 + 1024); SB(); } while (0)
  #define PINV(x) do { asm volatile("" : "+v"(x)); SB(); } while (0)
  #define PINS(x) do { asm volatile("" : "+s"(x)); SB(); } while (0)
  unsigned bB[4], bb[4];
  #pragma unroll
  for (int n = 0; n < 4; ++n) { bB[n] = rb00 ^ ((n >> 1) * 64 + (n & 1) * 8); bb[n] = bB[n]; }
  unsigned a0 = ra0, a1 = ra0 ^ 64;
  BAR();
  #pragma unroll
  for (int n = 0; n < 4; ++n) { RL(b0l, n, 0); RH(b0h, n, 0); }
  #pragma unroll
  for (int m = 0; m < 7; ++m) RA(af[m], a0, m);
  unsigned sa = 0, sb = 0;
  for (int u = 0; u < 64; ++u) {
    MF(0, 0, b0l, b0h); RA(af[7], a0, 7); MF(0, 1, b0l, b0h); RL(b1l, 0, 1); MF(0, 2, b0l, b0h); RH(b1h, 0, 1); MF(0, 3, b0l, b0h);
    #pragma unroll
    for (int g = 1; g < 4; ++g) {
      MF(g, 0, b0l, b0h); RA(af[g - 1], a1, g - 1); MF(g, 1, b0l, b0h); RL(b1l, g, 1); MF(g, 2, b0l, b0h); RH(b1h, g, 1); MF(g, 3, b0l, b0h);
      if (g == 1) { sb = sb + B_ST; PINS(sb); }
      if (g == 2) { sb = sb & (4 * B_ST - 1); PINS(sb); }
    }
    #pragma unroll
    for (int g = 4; g < 8; ++g) {
      MF(g, 0, b0l, b0h); RA(af[g - 1], a1, g - 1); MF(g, 1, b0l, b0h); bb[g - 4] = bB[g - 4] + sb; PINV(bb[g - 4]); MF(g, 2, b0l, b0h);
      if (g == 4) { sa = sa == 2 * A_ST ? 0 : sa + A_ST; PINS(sa); }
      if (g == 5) { a0 = ra0 + sa; PINV(a0); }
      MF(g, 3, b0l, b0h);
    }
    MF(0, 0, b1l, b1h); RA(af[7], a1, 7); MF(0, 1, b1l, b1h); MF(0, 2, b1l, b1h); a1 = a0 ^ 64; PINV(a1); MF(0, 3, b1l, b1h);
    MF(1, 0, b1l, b1h); MF(1, 1, b1l, b1h);
    if (u == ulast) tmov = ld_agent(myws + WS_TMO);
    SB();
    MF(1, 2, b1l, b1h); MF(1, 3, b1l, b1h);
    asm volatile("s_waitcnt lgkmcnt(0)\n\ts_barrier" ::: "memory"); SB();
    MF(2, 0, b1l, b1h); RA(af[0], a0, 0); MF(2, 1, b1l, b1h); RL(b0l, 0, 0); MF(2, 2, b1l, b1h); RH(b0h, 0, 0); MF(2, 3, b1l, b1h); RA(af[1], a0, 1);
    #pragma unroll
    for (int g = 3; g < 8; ++g) {
      MF(g, 0, b1l, b1h); RA(af[g - 1], a0, g - 1); MF(g, 1, b1l, b1h); if (g < 6) RL(b0l, g - 2, 0); MF(g, 2, b1l, b1h); if (g < 6) RH(b0h, g - 2, 0); MF(g, 3, b1l, b1h);
    }
  }
  if (rfl(tmov) == call) {
    asm volatile("" ::: "memory");
    const float qn = __builtin_nanf("");
    #pragma unroll
    for (int m = 0; m < 8; ++m)
      #pragma unroll
      for (int n = 0; n < 4; ++n) acc[m][n] = (f32x4){qn, qn, qn, qn};
  }
  char* orow = P.out + (size_t)(tm * 256 + wm * 128 + fr) * NTOT * 2 + colb + wn * 512 + fq * 16;
  const bool odd = fq & 1;
  orow += (fr & 8) ? (long)64 - (long)8 * NTOT * 2 : 0;
  SB();
  #pragma unroll
  for (int m = 0; m < 8; ++m) {
    u32x4 v[2];
    #pragma unroll
    for (int pr = 0; pr < 2; ++pr) {
      const f32x4 a = acc[m][2 * pr], c = acc[m][2 * pr + 1];
      const unsigned a0 = relu2(cvtpk(a[0], a[1])), a1 = relu2(cvtpk(a[2], a[3])), c0 = relu2(cvtpk(c[0], c[1])), c1 = relu2(cvtpk(c[2], c[3]));
      v[pr][0] = odd ? c0 : a0; v[pr][1] = odd ? c1 : a1; v[pr][2] = odd ? a0 : c0; v[pr][3] = odd ? a1 : c1;
    }
    char* pp = orow + (size_t)m * 16 * NTOT * 2;
    u32x4 lo, hi;
    #pragma unroll
    for (int e = 0; e < 4; ++e) {
      lo[e] = __builtin_amdgcn_update_dpp(v[0][e], v[1][e], 0x128, 0xf, 0xc, false);
      hi[e] = __builtin_amdgcn_update_dpp(v[1][e], v[0][e], 0x128, 0xf, 0x3, false);
    }
    asm volatile("global_store_dwordx4 %0, %1, off " OUT_POL "\n\ts_nop 1" :: "v"(pp), "v"(lo) : "memory");
    asm volatile("global_store_dwordx4 %0, %1, off " OUT_POL "\n\ts_nop 1" :: "v"(pp + (size_t)8 * NTOT * 2), "v"(hi) : "memory");
    SB();
  }
  if (tid == 0) __hip_atomic_store((unsigned*)(myws + WS_CTR + b * 128), call, __ATOMIC_RELAXED, __HIP_MEMORY_SCOPE_AGENT);
}

extern "C" void kernel_launch(int rank, int nranks, ncclComm_t comm, hipStream_t stream,
                              void* const* d_in, const int* in_sizes, int n_in,
                              void* d_out, int out_size, const int* in_dtypes, int out_dtype,
                              void* const* d_out_all, void* const* d_ws_all, size_t ws_bytes) {
  (void)comm; (void)in_sizes; (void)n_in; (void)out_size; (void)in_dtypes; (void)out_dtype; (void)d_out_all;
  if (nranks != NR || (unsigned)rank >= (unsigned)NR || ws_bytes < WS_NEED) { fprintf(stderr, "unexpected launch geometry\n"); return; }
  static bool init = false;
  if (!init) {
    hipError_t e = hipFuncSetAttribute((const void*)a2a_gemm_relu, hipFuncAttributeMaxDynamicSharedMemorySize, (int)LDS_BYTES);
    if (e != hipSuccess) { fprintf(stderr, "hipFuncSetAttribute: %s\n", hipGetErrorString(e)); return; }
    init = true;
  }
  Params p;
  p.x = (const char*)d_in[0]; p.w = (const char*)d_in[1]; p.out = (char*)d_out;
  for (int i = 0; i < NR; ++i) p.ws[i] = (char*)d_ws_all[i];
  p.rank = rank; p.pad = 0;
  hipLaunchKernelGGL(a2a_gemm_relu, dim3(256), dim3(512), LDS_BYTES, stream, p);
  hipError_t e = hipGetLastError();
  if (e != hipSuccess) fprintf(stderr, "launch: %s\n", hipGetErrorString(e));
}
```

```cpp
#include <hip/hip_runtime.h>
#include <rccl/rccl.h>
#include <cstdio>
#include <cstdint>

typedef __bf16 bf16x8 __attribute__((ext_vector_type(8)));
typedef __bf16 bf16x2 __attribute__((ext_vector_type(2)));
typedef short s16x4 __attribute__((ext_vector_type(4)));
typedef short s16x2 __attribute__((ext_vector_type(2)));
typedef float f32x4 __attribute__((ext_vector_type(4)));
typedef float f32x2 __attribute__((ext_vector_type(2)));
typedef unsigned u32x4 __attribute__((ext_vector_type(4)));
typedef int i32x4 __attribute__((ext_vector_type(4)));
#define LAS __attribute__((address_space(3)))
#define DI __device__ __forceinline__
#define SB() __builtin_amdgcn_sched_barrier(0)
#define BAR() asm volatile("s_barrier" ::: "memory")

#define XPD 6
#define XPK 1
#define XPF_REMOTE 0
#define BSL 0
#define PT0 120
#define PT1 30
#define PSU 0
#define PSP 4
#define LOOK 3
#define WPF 2
#define PERM 0
#define FULL_LINE 1
#define OUT_POL "sc1"
#define W_POL ""
constexpr int NR = 8, KTOT = 4096, NTOT = 16384;
constexpr unsigned A_RING = 0, A_ST = 32768, B_RING = 3 * 32768, B_ST = 16384, LDS_BYTES = 163840;
constexpr unsigned WS_FLAGS = 0;
constexpr unsigned WS_TMO = 4096;
constexpr unsigned WS_CTR = 8192;
constexpr unsigned WS_X = 1u << 20;
constexpr unsigned BLK = 524288;
constexpr unsigned XPAR = 8 * BLK;
constexpr size_t WS_NEED = (size_t)WS_X + 2 * (size_t)XPAR;
constexpr unsigned long long DEADLINE_TICKS = 100000000ull;

#define STAMP(i, v) do {} while (0)
struct Params { const char* x; const char* w; char* out; char* ws[NR]; int rank; int pad; };

DI unsigned rfl(unsigned x) { return __builtin_amdgcn_readfirstlane(x); }
DI unsigned cvtpk(float lo, float hi) { f32x2 v = {lo, hi}; bf16x2 b = __builtin_convertvector(v, bf16x2); return __builtin_bit_cast(unsigned, b); }
DI unsigned relu2(unsigned x) { const s16x2 v = __builtin_bit_cast(s16x2, x), z = {0, 0}; return __builtin_bit_cast(unsigned, __builtin_elementwise_max(v, z)); }
DI i32x4 make_srd(const void* p, unsigned bytes) {
  unsigned long long a = (unsigned long long)p;
  i32x4 r; r[0] = rfl((unsigned)a); r[1] = rfl((unsigned)(a >> 32)) & 0xffff; r[2] = rfl(bytes); r[3] = 0x00020000; return r;
}
template <int SC, int PAD> DI void dma4(unsigned m0v, unsigned v0, unsigned v1, unsigned v2, unsigned v3, i32x4 srd, unsigned so) {
  if constexpr (SC)
    asm volatile("s_mov_b32 m0, %0\n\ts_nop %7\n\t"
                 "buffer_load_dwordx4 %1, %5, %6 offen sc0 sc1 lds\n\t"
                 "buffer_load_dwordx4 %2, %5, %6 offen offset:1024 sc0 sc1 lds\n\t"
                 "buffer_load_dwordx4 %3, %5, %6 offen offset:2048 sc0 sc1 lds\n\t"
                 "buffer_load_dwordx4 %4, %5, %6 offen offset:3072 sc0 sc1 lds"
                 :: "s"(m0v), "v"(v0), "v"(v1), "v"(v2), "v"(v3), "s"(srd), "s"(so), "i"(PAD) : "memory");
  else
    asm volatile("s_mov_b32 m0, %0\n\ts_nop %7\n\t"
                 "buffer_load_dwordx4 %1, %5, %6 offen " W_POL " lds\n\t"
                 "buffer_load_dwordx4 %2, %5, %6 offen offset:1024 " W_POL " lds\n\t"
                 "buffer_load_dwordx4 %3, %5, %6 offen offset:2048 " W_POL " lds\n\t"
                 "buffer_load_dwordx4 %4, %5, %6 offen offset:3072 " W_POL " lds"
                 :: "s"(m0v), "v"(v0), "v"(v1), "v"(v2), "v"(v3), "s"(srd), "s"(so), "i"(PAD) : "memory");
}
DI s16x4 trd(unsigned a) { return __builtin_amdgcn_ds_read_tr16_b64_v4i16((LAS s16x4*)(size_t)a); }
DI bf16x8 rda(unsigned a) { return *(const LAS bf16x8*)(size_t)a; }
DI unsigned ld_agent(const char* p) { return __hip_atomic_load((const unsigned*)p, __ATOMIC_RELAXED, __HIP_MEMORY_SCOPE_AGENT); }

__global__ __launch_bounds__(512, 2) void a2a_gemm_relu(const Params P) {
  const int tid = threadIdx.x, lane = tid & 63;
  const unsigned wid = rfl(tid >> 6);
  const unsigned b = blockIdx.x, xcd = b & 7, idx = b >> 3, tn = xcd * 16 + (idx >> 1), tm = idx & 1;
  const unsigned rank = (unsigned)P.rank & 7u;
  char* const myws = P.ws[rank];
  const unsigned colb = (tn >> 2) * 1024 + (tn & 3) * 128;

  if (wid == 7) {
    if (b >= 224) return;
    int nbar = 0;
    unsigned long long tb = __builtin_amdgcn_s_memrealtime() + (PT0 - PT1);
    auto drain = [&]() {
      while ((__builtin_amdgcn_s_getreg(7 | (31 << 11)) & 0xC0000Fu) != 0 && nbar < 60) {
        if ((long long)(__builtin_amdgcn_s_memrealtime() - tb) > PT1) { BAR(); ++nbar; tb = __builtin_amdgcn_s_memrealtime(); }
        else __builtin_amdgcn_s_sleep(1);
      }
      asm volatile("s_waitcnt vmcnt(0)" ::: "memory");
    };
    unsigned callv;
    u32x4 d[16];
    {
      const unsigned s = (b >> 5) + 1, row = (b & 31) * 16, dest = (rank - s) & 7u;
      const char* src = P.x + (size_t)dest * BLK + (size_t)(row + (lane >> 3)) * 1024 + (lane & 7) * 16;
      asm volatile("global_load_dword %0, %3, off sc1\n\tglobal_load_dwordx4 %1, %4, off\n\tglobal_load_dwordx4 %2, %5, off"
                   : "=&v"(callv), "=&v"(d[0]), "=&v"(d[1]) : "v"(myws + WS_CTR + b * 128), "v"(src), "v"(src + 8192) : "memory");
      char* const pws = P.ws[dest];
      drain();
      asm volatile("" : "+v"(callv), "+v"(d[0]), "+v"(d[1]));
      char* dst = pws + WS_X + ((rfl(callv) + 1) & 1) * XPAR + rank * BLK + row * 128 + lane * 16;
      asm volatile("global_store_dwordx4 %0, %1, off sc0 sc1\n\tglobal_store_dwordx4 %0, %2, off offset:1024 sc0 sc1\n\ts_nop 1" :: "v"(dst), "v"(d[0]), "v"(d[1]) : "memory");
      drain();
      if (lane == 0) __hip_atomic_fetch_add((unsigned*)(pws + WS_FLAGS + (row >> 7) * 256 + (8 + s - 1) * 4), 1u, __ATOMIC_RELAXED, __HIP_MEMORY_SCOPE_SYSTEM);
      STAMP(8, rp0); STAMP(9, RT()); STAMP(10, RT());
    }
    if (b >= 196) return;
    const unsigned call = rfl(callv) + 1;
    const unsigned kt = b / 28 + 1, r = b % 28, s = (r >> 2) + 1, qd = r & 3, dest = (rank - s) & 7u;
    const char* src = P.x + (size_t)dest * BLK + (size_t)(qd * 128 + (lane >> 3)) * 1024 + kt * 128 + (lane & 7) * 16;
    #pragma unroll
    for (int i = 0; i < 16; i += 4)
      asm volatile("global_load_dwordx4 %0, %4, off\n\tglobal_load_dwordx4 %1, %5, off\n\tglobal_load_dwordx4 %2, %6, off\n\tglobal_load_dwordx4 %3, %7, off"
                   : "=&v"(d[i]), "=&v"(d[i + 1]), "=&v"(d[i + 2]), "=&v"(d[i + 3])
                   : "v"(src + i * 8192), "v"(src + (i + 1) * 8192), "v"(src + (i + 2) * 8192), "v"(src + (i + 3) * 8192) : "memory");
    char* const pws = P.ws[dest];
    char* dst = pws + WS_X + (call & 1) * XPAR + rank * BLK + kt * 65536 + qd * 16384 + lane * 16;
    const int su = (PSP == 2 ? (int)(2 * kt + (qd >> 1)) : PSP == 4 ? (int)(4 * kt + qd) - 2 : (int)(3 * kt + (qd * 3 >> 2)) - 1) + PSU;
    drain();
    #pragma unroll
    for (int i = 0; i < 16; i += 4) asm volatile("" : "+v"(d[i]), "+v"(d[i + 1]), "+v"(d[i + 2]), "+v"(d[i + 3]));
    for (; nbar < su; ++nbar) BAR();
    #pragma unroll
    for (int i = 0; i < 16; ++i)
      asm volatile("global_store_dwordx4 %0, %1, off sc0 sc1\n\ts_nop 1" :: "v"(dst + i * 1024), "v"(d[i]) : "memory");
    tb = __builtin_amdgcn_s_memrealtime() - PT1;
    drain();
    if (lane == 0)
      __hip_atomic_store((unsigned*)(pws + WS_FLAGS + qd * 256 + (8 + kt * 7 + s - 1) * 4), call * 8u, __ATOMIC_RELAXED, __HIP_MEMORY_SCOPE_SYSTEM);
    return;
  }
  if (wid >= 4) {
    unsigned tabA, tabB;
    unsigned pos = lane;
    if (tm) {
      if (PERM < 8) pos = lane ^ PERM;
      else if (lane & 1) { if (lane >= (lane < 8 ? 0u : 8u) + (PERM - 8)) pos = lane - (PERM - 8); }
      else if (lane + (PERM - 8) < (lane < 8 ? 8u : 64u)) pos = lane + (PERM - 8);
    }
    { const unsigned u = pos; unsigned src, kt;
      if (u < 8) { src = rank; kt = u; tabA = kt * 128; }
      else { const unsigned j = u - 8; kt = (j * 37) >> 8; src = (rank + (j - kt * 7 + 1)) & 7u; tabA = src * BLK + kt * 65536u; }
      tabB = (src * 512u + kt * 64u) * 32768u; }
    if (wid < 6) {
      const unsigned h = wid - 4;
      unsigned callv;
      asm volatile("global_load_dword %0, %1, off sc1" : "=v"(callv) : "v"(myws + WS_CTR + b * 128) : "memory");
      const i32x4 srdL = make_srd(P.x + (size_t)rank * BLK, BLK);
      const unsigned arow = tm * 256 + h * 128 + (lane >> 3), ach = ((lane & 7) ^ (lane >> 3)) * 16;
      unsigned vL[4];
      #pragma unroll
      for (int i = 0; i < 4; ++i) vL[i] = (arow + 8 * i) * 1024 + ach - i * 1024;
      const unsigned vR = arow * 128 + ach, m0A = A_RING + h * 16384;
      auto issueL = [&](int t, unsigned slot) {
        const unsigned so = __builtin_amdgcn_readlane(tabA, t), m = m0A + slot * A_ST;
        dma4<1, 3>(m, vL[0], vL[1], vL[2], vL[3], srdL, so);
        #pragma unroll
        for (int j = 1; j < 4; ++j) dma4<1, 0>(m + j * 4096, vL[0], vL[1], vL[2], vL[3], srdL, so + j * 32768);
      };
      issueL(0, 0);
      issueL(1, 1);
      asm volatile("s_waitcnt vmcnt(16)" : "+v"(callv) :: "memory");
      BAR();
      issueL(2, 2);
      const unsigned call = rfl(callv) + 1;
      i32x4 srdR = make_srd(myws + WS_X + (call & 1) * XPAR, XPAR);
      const char* fl = myws + WS_FLAGS + (tm * 2 + h) * 256 + pos * 4;
      const char* tmo = myws + WS_TMO;
      #define SAMPLE() asm volatile("global_load_dword v200, %0, off sc0 sc1" :: "v"(fl) : "memory", "v200")
      unsigned long long ready = 0xffull;
      bool pend = true;
      unsigned slot = 0;
      for (int u = 0; u < 5; ++u) {
        asm volatile("s_waitcnt vmcnt(16)" ::: "memory"); BAR();
        if (u == 4) SAMPLE();
        issueL(u + 3, slot);
        slot = slot == 2 ? 0 : slot + 1;
      }
      for (int u = 5; u < 64; ++u) {
        const int t = u + 3;
        if (ready != ~0ull) {
          if (pend) {
            unsigned long long mk;
            asm volatile("s_waitcnt vmcnt(16)\n\ts_barrier\n\tv_cmp_ge_u32 %0, v200, %1" : "=s"(mk) : "v"(call * 8u) : "memory", "v200");
            ready |= mk;
          } else {
            asm volatile("s_waitcnt vmcnt(16)\n\ts_barrier" ::: "memory");
          }
          if (!((ready >> t) & 1)) {
            const unsigned long long t0 = __builtin_amdgcn_s_memrealtime();
            for (;;) {
              unsigned fv, tv;
              asm volatile("global_load_dword %0, %2, off sc0 sc1\n\tglobal_load_dword %1, %3, off sc1\n\ts_waitcnt vmcnt(0)"
                           : "=&v"(fv), "=&v"(tv) : "v"(fl), "v"(tmo) : "memory");
              ready |= __ballot(fv >= call * 8u);
              if ((ready >> t) & 1) break;
              if (rfl(tv) == call || __builtin_amdgcn_s_memrealtime() - t0 > DEADLINE_TICKS) {
                if (lane == 0) __hip_atomic_store((unsigned*)tmo, call, __ATOMIC_RELAXED, __HIP_MEMORY_SCOPE_AGENT);
                asm volatile("s_waitcnt vmcnt(0)" ::: "memory");
                ready = ~0ull; srdR[2] = 0;
                break;
              }
              __builtin_amdgcn_s_sleep(2);
            }
          }
          ready |= (2ull << t) - 1;
          pend = ((~ready >> t) & ((2ull << LOOK) - 1)) != 0;
          if (pend) SAMPLE();
        } else {
          asm volatile("s_waitcnt vmcnt(16)\n\ts_barrier" ::: "memory");
        }
        unsigned so = __builtin_amdgcn_readlane(tabA, t & 63);
        if (t >= 64) { srdR[2] = 0; so = 0; }
        const unsigned m = m0A + slot * A_ST;
        dma4<1, 3>(m, vR, vR, vR, vR, srdR, so);
        #pragma unroll
        for (int j = 1; j < 4; ++j) dma4<1, 0>(m + j * 4096, vR, vR, vR, vR, srdR, so + j * 4096);
        slot = slot == 2 ? 0 : slot + 1;
      }
      if (h == 0) STAMP(15, ((q0 - rt0) & 0xffff) | (((q1 - rt0) & 0xffff) << 16) | (((q2 - rt0) & 0xffff) << 32) | ((rt0 & 0xffff) << 48));
      STAMP(5 + 6 * h, spin); STAMP(6 + 6 * h, nspin); STAMP(7 + 6 * h, firstspin);
    } else {
      i32x4 srdW = make_srd(P.w, (unsigned)KTOT * NTOT * 2u);
      unsigned vB[4];
      #pragma unroll
      for (int i = 0; i < 4; ++i) {
        const unsigned ch = (lane & 15) ^ (((lane >> 4) << 2) | (i >> 1));
        vB[i] = (4 * i + (lane >> 4)) * 32768 + colb + (ch >> 3) * 512 + (ch & 7) * 16 - i * 1024;
      }
      auto issue = [&](int t, unsigned slot) {
        const unsigned so = __builtin_amdgcn_readlane(tabB, t & 63), m = B_RING + slot * B_ST;
        if (t >= 64) srdW[2] = 0;
        dma4<0, 3>(m, vB[0], vB[1], vB[2], vB[3], srdW, so);
        #pragma unroll
        for (int j = 1; j < 4; ++j) dma4<0, 0>(m + j * 4096, vB[0], vB[1], vB[2], vB[3], srdW, so + j * 16 * 32768);
      };
      unsigned callv, dummy = 0;
      asm volatile("global_load_dword %0, %1, off sc1" : "=v"(callv) : "v"(myws + WS_CTR + b * 128) : "memory");
      const i32x4 srdXL = make_srd(P.x + (size_t)rank * BLK, BLK);
      const unsigned xr = tm * 256 + (idx >> 1) * 16 + (lane & 15), vX = xr * 128, vXL = xr * 1024;
      issue(0, 0);
      issue(1, 1);
      asm volatile("s_waitcnt vmcnt(16)" : "+v"(callv) :: "memory");
      BAR();
      issue(2, 2); issue(3, 3);
      const i32x4 srdX = make_srd(myws + WS_X + ((rfl(callv) + 1) & 1) * XPAR, XPAR);
      const i32x4 srdWp = make_srd(P.w, (unsigned)KTOT * NTOT * 2u);
      const unsigned vW = WPF == 1 ? (idx * 2 + (lane >> 5)) * 32768 + xcd * 4096 + (lane & 31) * 128
                                   : ((idx & 7) * 8 + (lane >> 3)) * 32768 + (tn >> 2) * 1024 + (lane & 7) * 128;
      auto pf = [&](int t) {
        if (WPF && t < 64 && t >= 4) {
          const unsigned sw = __builtin_amdgcn_readlane(tabB, t);
          asm volatile("s_nop 4\n\tbuffer_load_dword %0, %1, %2, %3 offen" : "+v"(dummy) : "v"(vW), "s"(srdWp), "s"(sw) : "memory");
        }
        if (t < 64 && lane < 16) {
          const unsigned so = __builtin_amdgcn_readlane(tabA, t);
          if (t < 8) asm volatile("s_nop 4\n\tbuffer_load_dword %0, %1, %2, %3 offen sc0 sc1" : "+v"(dummy) : "v"(vXL), "s"(srdXL), "s"(so) : "memory");
          else if (XPF_REMOTE) asm volatile("s_nop 4\n\tbuffer_load_dword %0, %1, %2, %3 offen sc0 sc1" : "+v"(dummy) : "v"(vX), "s"(srdX), "s"(so) : "memory");
        } };
      int nxt = WPF ? (int)(idx % XPK) : 0;
      for (int t = 3; t < XPD + nxt; ++t) pf(t);
      for (int u = 0; u < 64; ++u) {
        asm volatile("s_waitcnt vmcnt(32)" ::: "memory"); BAR();
        if (BSL) __builtin_amdgcn_s_sleep(BSL);
        if (WPF) issue(u + 4, u & 3);
        if (u == nxt) {
          nxt += XPK;
          #pragma unroll
          for (int j = 0; j < XPK; ++j) pf(u + XPD + j);
        }
        if (!WPF) issue(u + 4, u & 3);
      }
      asm volatile("s_waitcnt vmcnt(0)" :: "v"(dummy) : "memory");
      STAMP(14, ((q0 - rt0) & 0xffff) | (((q1 - rt0) & 0xffff) << 16) | (((q2 - rt0) & 0xffff) << 32) | ((rt0 & 0xffff) << 48));
    }
    return;
  }
  const unsigned wm = wid >> 1, wn = wid & 1, fr = lane & 15, fq = lane >> 4, q = (lane & 15) >> 2, p = lane & 3;
  const unsigned ra0 = A_RING + (wm * 128 + fr) * 128 + ((fq ^ (fr & 7)) * 16);
  const unsigned rb00 = B_RING + (8 * fq + q) * 256 + (((wn * 8 + p) ^ ((q << 2) | (fq & 1))) * 16) + (p & 1) * 8;
  const unsigned call = rfl(ld_agent(myws + WS_CTR + b * 128)) + 1;
  unsigned tmov = 0;
  int ulast = 62; asm volatile("" : "+s"(ulast));
  f32x4 acc[8][4];
  #pragma unroll
  for (int m = 0; m < 8; ++m)
    #pragma unroll
    for (int n = 0; n < 4; ++n) acc[m][n] = (f32x4){0.f, 0.f, 0.f, 0.f};
  bf16x8 af[8]; s16x4 b0l[4], b0h[4], b1l[4], b1h[4];
  #define MF(m, n_, L, H) MF2(m, (((m) & 1) ? 3 - (n_) : (n_)), L, H)
  #define MF2(m, n, L, H) do { acc[m][n] = __builtin_amdgcn_mfma_f32_16x16x32_bf16(__builtin_bit_cast(bf16x8, __builtin_shufflevector(L[n], H[n], 0, 1, 2, 3, 4, 5, 6, 7)), af[m], acc[m][n], 0, 0, 0); \
                              asm volatile("" : "+v"(acc[m][n])); SB(); } while (0)
  #define RA(dst, base, m) do { dst = rda((base) + (m) * 2048); SB(); } while (0)
  #define RL(L, n, ks) do { L[n] = trd(bb[n] + (ks) * 8192); SB(); } while (0)
  #define RH(H, n, ks) do { H[n] = trd(bb[n] + (ks) * 8192 + 1024); SB(); } while (0)
  #define PINV(x) do { asm volatile("" : "+v"(x)); SB(); } while (0)
  #define PINS(x) do { asm volatile("" : "+s"(x)); SB(); } while (0)
  unsigned bB[4], bb[4];
  #pragma unroll
  for (int n = 0; n < 4; ++n) { bB[n] = rb00 ^ ((n >> 1) * 64 + (n & 1) * 8); bb[n] = bB[n]; }
  unsigned a0 = ra0, a1 = ra0 ^ 64;
  BAR();
  #pragma unroll
  for (int n = 0; n < 4; ++n) { RL(b0l, n, 0); RH(b0h, n, 0); }
  #pragma unroll
  for (int m = 0; m < 7; ++m) RA(af[m], a0, m);
  unsigned sa = 0, sb = 0;
  for (int u = 0; u < 64; ++u) {
    MF(0, 0, b0l, b0h); RA(af[7], a0, 7); MF(0, 1, b0l, b0h); RL(b1l, 0, 1); MF(0, 2, b0l, b0h); RH(b1h, 0, 1); MF(0, 3, b0l, b0h);
    #pragma unroll
    for (int g = 1; g < 4; ++g) {
      MF(g, 0, b0l, b0h); RA(af[g - 1], a1, g - 1); MF(g, 1, b0l, b0h); RL(b1l, g, 1); MF(g, 2, b0l, b0h); RH(b1h, g, 1); MF(g, 3, b0l, b0h);
      if (g == 1) { sb = sb + B_ST; PINS(sb); }
      if (g == 2) { sb = sb & (4 * B_ST - 1); PINS(sb); }
    }
    #pragma unroll
    for (int g = 4; g < 8; ++g) {
      MF(g, 0, b0l, b0h); RA(af[g - 1], a1, g - 1); MF(g, 1, b0l, b0h); bb[g - 4] = bB[g - 4] + sb; PINV(bb[g - 4]); MF(g, 2, b0l, b0h);
      if (g == 4) { sa = sa == 2 * A_ST ? 0 : sa + A_ST; PINS(sa); }
      if (g == 5) { a0 = ra0 + sa; PINV(a0); }
      MF(g, 3, b0l, b0h);
    }
    MF(0, 0, b1l, b1h); RA(af[7], a1, 7); MF(0, 1, b1l, b1h); MF(0, 2, b1l, b1h); a1 = a0 ^ 64; PINV(a1); MF(0, 3, b1l, b1h);
    MF(1, 0, b1l, b1h); MF(1, 1, b1l, b1h);
    if (u == ulast) tmov = ld_agent(myws + WS_TMO);
    SB();
    MF(1, 2, b1l, b1h); MF(1, 3, b1l, b1h);
    asm volatile("s_waitcnt lgkmcnt(0)\n\ts_barrier" ::: "memory"); SB();
    MF(2, 0, b1l, b1h); RA(af[0], a0, 0); MF(2, 1, b1l, b1h); RL(b0l, 0, 0); MF(2, 2, b1l, b1h); RH(b0h, 0, 0); MF(2, 3, b1l, b1h); RA(af[1], a0, 1);
    #pragma unroll
    for (int g = 3; g < 8; ++g) {
      MF(g, 0, b1l, b1h); RA(af[g - 1], a0, g - 1); MF(g, 1, b1l, b1h); if (g < 6) RL(b0l, g - 2, 0); MF(g, 2, b1l, b1h); if (g < 6) RH(b0h, g - 2, 0); MF(g, 3, b1l, b1h);
    }
  }
  if (rfl(tmov) == call) {
    asm volatile("" ::: "memory");
    const float qn = __builtin_nanf("");
    #pragma unroll
    for (int m = 0; m < 8; ++m)
      #pragma unroll
      for (int n = 0; n < 4; ++n) acc[m][n] = (f32x4){qn, qn, qn, qn};
  }
  char* orow = P.out + (size_t)(tm * 256 + wm * 128 + fr) * NTOT * 2 + colb + wn * 512 + fq * 16;
  const bool odd = fq & 1;
  orow += (fr & 8) ? (long)64 - (long)8 * NTOT * 2 : 0;
  SB();
  #pragma unroll
  for (int m = 0; m < 8; ++m) {
    u32x4 v[2];
    #pragma unroll
    for (int pr = 0; pr < 2; ++pr) {
      const f32x4 a = acc[m][2 * pr], c = acc[m][2 * pr + 1];
      const unsigned a0 = relu2(cvtpk(a[0], a[1])), a1 = relu2(cvtpk(a[2], a[3])), c0 = relu2(cvtpk(c[0], c[1])), c1 = relu2(cvtpk(c[2], c[3]));
      v[pr][0] = odd ? c0 : a0; v[pr][1] = odd ? c1 : a1; v[pr][2] = odd ? a0 : c0; v[pr][3] = odd ? a1 : c1;
    }
    char* pp = orow + (size_t)m * 16 * NTOT * 2;
    u32x4 lo, hi;
    #pragma unroll
    for (int e = 0; e < 4; ++e) {
      lo[e] = __builtin_amdgcn_update_dpp(v[0][e], v[1][e], 0x128, 0xf, 0xc, false);
      hi[e] = __builtin_amdgcn_update_dpp(v[1][e], v[0][e], 0x128, 0xf, 0x3, false);
    }
    asm volatile("global_store_dwordx4 %0, %1, off " OUT_POL "\n\ts_nop 1" :: "v"(pp), "v"(lo) : "memory");
    asm volatile("global_store_dwordx4 %0, %1, off " OUT_POL "\n\ts_nop 1" :: "v"(pp + (size_t)8 * NTOT * 2), "v"(hi) : "memory");
    SB();
  }
  if (tid == 0) __hip_atomic_store((unsigned*)(myws + WS_CTR + b * 128), call, __ATOMIC_RELAXED, __HIP_MEMORY_SCOPE_AGENT);
}

extern "C" void kernel_launch(int rank, int nranks, ncclComm_t comm, hipStream_t stream,
                              void* const* d_in, const int* in_sizes, int n_in,
                              void* d_out, int out_size, const int* in_dtypes, int out_dtype,
                              void* const* d_out_all, void* const* d_ws_all, size_t ws_bytes) {
  (void)comm; (void)in_sizes; (void)n_in; (void)out_size; (void)in_dtypes; (void)out_dtype; (void)d_out_all;
  if (nranks != NR || (unsigned)rank >= (unsigned)NR || ws_bytes < WS_NEED) { fprintf(stderr, "unexpected launch geometry\n"); return; }
  static bool init = false;
  if (!init) {
    hipError_t e = hipFuncSetAttribute((const void*)a2a_gemm_relu, hipFuncAttributeMaxDynamicSharedMemorySize, (int)LDS_BYTES);
    if (e != hipSuccess) { fprintf(stderr, "hipFuncSetAttribute: %s\n", hipGetErrorString(e)); return; }
    init = true;
  }
  Params p;
  p.x = (const char*)d_in[0]; p.w = (const char*)d_in[1]; p.out = (char*)d_out;
  for (int i = 0; i < NR; ++i) p.ws[i] = (char*)d_ws_all[i];
  p.rank = rank; p.pad = 0;
  hipLaunchKernelGGL(a2a_gemm_relu, dim3(256), dim3(512), LDS_BYTES, stream, p);
  hipError_t e = hipGetLastError();
  if (e != hipSuccess) fprintf(stderr, "launch: %s\n", hipGetErrorString(e));
}
```

```cpp
#include <hip/hip_runtime.h>
#include <rccl/rccl.h>
#include <cstdio>
#include <cstdint>

typedef __bf16 bf16x8 __attribute__((ext_vector_type(8)));
typedef __bf16 bf16x2 __attribute__((ext_vector_type(2)));
typedef short s16x4 __attribute__((ext_vector_type(4)));
typedef short s16x2 __attribute__((ext_vector_type(2)));
typedef float f32x4 __attribute__((ext_vector_type(4)));
typedef float f32x2 __attribute__((ext_vector_type(2)));
typedef unsigned u32x4 __attribute__((ext_vector_type(4)));
typedef int i32x4 __attribute__((ext_vector_type(4)));
#define LAS __attribute__((address_space(3)))
#define DI __device__ __forceinline__
#define SB() __builtin_amdgcn_sched_barrier(0)
#define BAR() asm volatile("s_barrier" ::: "memory")

#define XPD 6
#define XPK 1
#define XPF_REMOTE 0
#define BSL 0
#define PT0 120
#define PT1 30
#define PSU 2
#define PSP 4
#define STG 0
#define STGS 2
#define DEFL 1
#define TLBW 0
#define PSL 0
#define LOOK 3
#define WPF 2
#define PERM 0
#define PF0 0
#define PF0S 0
#define CSP 512
#define FULL_LINE 1
#define OUT_POL "sc1"
#define PF_POL ""
#define PFFIRST 0
#define W_POL ""
constexpr int NR = 8, KTOT = 4096, NTOT = 16384;
constexpr unsigned A_RING = 0, A_ST = 32768, B_RING = 3 * 32768, B_ST = 16384, LDS_BYTES = 163840;
constexpr unsigned WS_FLAGS = 0;
constexpr unsigned WS_TMO = 4096;
constexpr unsigned WS_CTR = 8192;
constexpr unsigned WS_X = 1u << 20;
constexpr unsigned BLK = 524288;
constexpr unsigned XPAR = 8 * BLK;
constexpr size_t WS_NEED = (size_t)WS_X + 2 * (size_t)XPAR;
constexpr unsigned long long DEADLINE_TICKS = 100000000ull;

#define STAMP(i, v) do {} while (0)
struct Params { const char* x; const char* w; char* out; char* ws[NR]; int rank; int pad; };

DI unsigned rfl(unsigned x) { return __builtin_amdgcn_readfirstlane(x); }
DI unsigned cvtpk(float lo, float hi) { f32x2 v = {lo, hi}; bf16x2 b = __builtin_convertvector(v, bf16x2); return __builtin_bit_cast(unsigned, b); }
DI unsigned relu2(unsigned x) { const s16x2 v = __builtin_bit_cast(s16x2, x), z = {0, 0}; return __builtin_bit_cast(unsigned, __builtin_elementwise_max(v, z)); }
DI i32x4 make_srd(const void* p, unsigned bytes) {
  unsigned long long a = (unsigned long long)p;
  i32x4 r; r[0] = rfl((unsigned)a); r[1] = rfl((unsigned)(a >> 32)) & 0xffff; r[2] = rfl(bytes); r[3] = 0x00020000; return r;
}
template <int SC, int PAD> DI void dma4(unsigned m0v, unsigned v0, unsigned v1, unsigned v2, unsigned v3, i32x4 srd, unsigned so) {
  if constexpr (SC)
    asm volatile("s_mov_b32 m0, %0\n\ts_nop %7\n\t"
                 "buffer_load_dwordx4 %1, %5, %6 offen sc0 sc1 lds\n\t"
                 "buffer_load_dwordx4 %2, %5, %6 offen offset:1024 sc0 sc1 lds\n\t"
                 "buffer_load_dwordx4 %3, %5, %6 offen offset:2048 sc0 sc1 lds\n\t"
                 "buffer_load_dwordx4 %4, %5, %6 offen offset:3072 sc0 sc1 lds"
                 :: "s"(m0v), "v"(v0), "v"(v1), "v"(v2), "v"(v3), "s"(srd), "s"(so), "i"(PAD) : "memory");
  else
    asm volatile("s_mov_b32 m0, %0\n\ts_nop %7\n\t"
                 "buffer_load_dwordx4 %1, %5, %6 offen " W_POL " lds\n\t"
                 "buffer_load_dwordx4 %2, %5, %6 offen offset:1024 " W_POL " lds\n\t"
                 "buffer_load_dwordx4 %3, %5, %6 offen offset:2048 " W_POL " lds\n\t"
                 "buffer_load_dwordx4 %4, %5, %6 offen offset:3072 " W_POL " lds"
                 :: "s"(m0v), "v"(v0), "v"(v1), "v"(v2), "v"(v3), "s"(srd), "s"(so), "i"(PAD) : "memory");
}
DI s16x4 trd(unsigned a) { return __builtin_amdgcn_ds_read_tr16_b64_v4i16((LAS s16x4*)(size_t)a); }
DI bf16x8 rda(unsigned a) { return *(const LAS bf16x8*)(size_t)a; }
DI unsigned ld_agent(const char* p) { return __hip_atomic_load((const unsigned*)p, __ATOMIC_RELAXED, __HIP_MEMORY_SCOPE_AGENT); }

__global__ __launch_bounds__(512, 2) void a2a_gemm_relu(const Params P) {
  const int tid = threadIdx.x, lane = tid & 63;
  const unsigned wid = rfl(tid >> 6);
  const unsigned b = blockIdx.x, xcd = b & 7, idx = b >> 3, tn = xcd * 16 + (idx >> 1), tm = idx & 1;
  const unsigned rank = (unsigned)P.rank & 7u;
  char* const myws = P.ws[rank];
  const unsigned colb = CSP == 512 ? (tn >> 2) * 1024 + (tn & 3) * 128 : tn * 256;

  if (wid == 7) {
    if (b >= 224) {
      return;
    }
    int nbar = 0;
    unsigned long long tb = __builtin_amdgcn_s_memrealtime() + (PT0 - PT1);
    auto drain = [&]() {
      while ((__builtin_amdgcn_s_getreg(7 | (31 << 11)) & 0xC0000Fu) != 0 && nbar < 60) {
        if ((long long)(__builtin_amdgcn_s_memrealtime() - tb) > PT1) { BAR(); ++nbar; tb = __builtin_amdgcn_s_memrealtime(); }
        else __builtin_amdgcn_s_sleep(1);
      }
      asm volatile("s_waitcnt vmcnt(0)" ::: "memory");
    };
    unsigned callv;
    u32x4 d[16];
    {
      const unsigned s = (b >> 5) + 1, row = (b & 31) * 16, dest = (rank - s) & 7u;
      const char* src = P.x + (size_t)dest * BLK + (size_t)(row + (lane >> 3)) * 1024 + (lane & 7) * 16;
      asm volatile("global_load_dword %0, %3, off sc1\n\tglobal_load_dwordx4 %1, %4, off\n\tglobal_load_dwordx4 %2, %5, off"
                   : "=&v"(callv), "=&v"(d[0]), "=&v"(d[1]) : "v"(myws + WS_CTR + b * 128), "v"(src), "v"(src + 8192) : "memory");
      char* const pws = P.ws[dest];
      drain();
      asm volatile("" : "+v"(callv), "+v"(d[0]), "+v"(d[1]));
      if (STG) {
        const int w = (int)((b & 31) % STG + ((s - 1) * STGS >> 2));
        if (nbar < w) { for (; nbar < w; ++nbar) BAR(); tb = __builtin_amdgcn_s_memrealtime(); }
      }
      char* dst = pws + WS_X + ((rfl(callv) + 1) & 1) * XPAR + rank * BLK + row * 128 + lane * 16;
      asm volatile("global_store_dwordx4 %0, %1, off sc0 sc1\n\tglobal_store_dwordx4 %0, %2, off offset:1024 sc0 sc1\n\ts_nop 1" :: "v"(dst), "v"(d[0]), "v"(d[1]) : "memory");
      drain();
      if (lane == 0) __hip_atomic_fetch_add((unsigned*)(pws + WS_FLAGS + (row >> 7) * 256 + (8 + s - 1) * 4), 1u, __ATOMIC_RELAXED, __HIP_MEMORY_SCOPE_SYSTEM);
      STAMP(8, rp0); STAMP(9, RT()); STAMP(10, RT());
    }
    if (b >= 196) return;
    const unsigned call = rfl(callv) + 1;
    const unsigned kt = b / 28 + 1, r = b % 28, s = (r >> 2) + 1, qd = r & 3, dest = (rank - s) & 7u;
    const char* src = P.x + (size_t)dest * BLK + (size_t)(qd * 128 + (lane >> 3)) * 1024 + kt * 128 + (lane & 7) * 16;
    const int su = (PSP == 2 ? (int)(2 * kt + (qd >> 1)) : PSP == 4 ? (int)(4 * kt + qd) - 2 : (int)(3 * kt + (qd * 3 >> 2)) - 1) + PSU;
    if (DEFL && nbar < su - 2) { for (; nbar < su - 2; ++nbar) BAR(); tb = __builtin_amdgcn_s_memrealtime(); }
    #pragma unroll
    for (int i = 0; i < 16; i += 4)
      asm volatile("global_load_dwordx4 %0, %4, off\n\tglobal_load_dwordx4 %1, %5, off\n\tglobal_load_dwordx4 %2, %6, off\n\tglobal_load_dwordx4 %3, %7, off"
                   : "=&v"(d[i]), "=&v"(d[i + 1]), "=&v"(d[i + 2]), "=&v"(d[i + 3])
                   : "v"(src + i * 8192), "v"(src + (i + 1) * 8192), "v"(src + (i + 2) * 8192), "v"(src + (i + 3) * 8192) : "memory");
    char* const pws = P.ws[dest];
    char* dst = pws + WS_X + (call & 1) * XPAR + rank * BLK + kt * 65536 + qd * 16384 + lane * 16;
    drain();
    #pragma unroll
    for (int i = 0; i < 16; i += 4) asm volatile("" : "+v"(d[i]), "+v"(d[i + 1]), "+v"(d[i + 2]), "+v"(d[i + 3]));
    for (; nbar < su; ++nbar) BAR();
    if (PSL) __builtin_amdgcn_s_sleep(PSL);
    #pragma unroll
    for (int i = 0; i < 16; ++i)
      asm volatile("global_store_dwordx4 %0, %1, off sc0 sc1\n\ts_nop 1" :: "v"(dst + i * 1024), "v"(d[i]) : "memory");
    tb = __builtin_amdgcn_s_memrealtime() - PT1;
    drain();
    if (lane == 0)
      __hip_atomic_store((unsigned*)(pws + WS_FLAGS + qd * 256 + (8 + kt * 7 + s - 1) * 4), call * 8u, __ATOMIC_RELAXED, __HIP_MEMORY_SCOPE_SYSTEM);
    return;
  }
  if (wid >= 4) {
    unsigned tabA, tabB;
    unsigned pos = lane;
    if (tm) {
      if (PERM < 8) pos = lane ^ PERM;
      else if (lane & 1) { if (lane >= (lane < 8 ? 0u : 8u) + (PERM - 8)) pos = lane - (PERM - 8); }
      else if (lane + (PERM - 8) < (lane < 8 ? 8u : 64u)) pos = lane + (PERM - 8);
    }
    { const unsigned u = pos; unsigned src, kt;
      if (u < 8) { src = rank; kt = u; tabA = kt * 128; }
      else { const unsigned j = u - 8; kt = (j * 37) >> 8; src = (rank + (j - kt * 7 + 1)) & 7u; tabA = src * BLK + kt * 65536u; }
      tabB = (src * 512u + kt * 64u) * 32768u; }
    if (wid < 6) {
      const unsigned h = wid - 4;
      unsigned callv;
      asm volatile("global_load_dword %0, %1, off sc1" : "=v"(callv) : "v"(myws + WS_CTR + b * 128) : "memory");
      const i32x4 srdL = make_srd(P.x + (size_t)rank * BLK, BLK);
      const unsigned arow = tm * 256 + h * 128 + (lane >> 3), ach = ((lane & 7) ^ (lane >> 3)) * 16;
      unsigned vL[4];
      #pragma unroll
      for (int i = 0; i < 4; ++i) vL[i] = (arow + 8 * i) * 1024 + ach - i * 1024;
      const unsigned vR = arow * 128 + ach, m0A = A_RING + h * 16384;
      auto issueL = [&](int t, unsigned slot) {
        const unsigned so = __builtin_amdgcn_readlane(tabA, t), m = m0A + slot * A_ST;
        dma4<1, 3>(m, vL[0], vL[1], vL[2], vL[3], srdL, so);
        #pragma unroll
        for (int j = 1; j < 4; ++j) dma4<1, 0>(m + j * 4096, vL[0], vL[1], vL[2], vL[3], srdL, so + j * 32768);
      };
      issueL(0, 0);
      issueL(1, 1);
      asm volatile("s_waitcnt vmcnt(16)" : "+v"(callv) :: "memory");
      BAR();
      issueL(2, 2);
      const unsigned call = rfl(callv) + 1;
      i32x4 srdR = make_srd(myws + WS_X + (call & 1) * XPAR, XPAR);
      const char* fl = myws + WS_FLAGS + (tm * 2 + h) * 256 + pos * 4;
      const char* tmo = myws + WS_TMO;
      #define SAMPLE() asm volatile("global_load_dword v200, %0, off sc0 sc1" :: "v"(fl) : "memory", "v200")
      unsigned long long ready = 0xffull;
      bool pend = true;
      unsigned slot = 0;
      for (int u = 0; u < 5; ++u) {
        asm volatile("s_waitcnt vmcnt(16)" ::: "memory"); BAR();
        if (u == 4) SAMPLE();
        issueL(u + 3, slot);
        slot = slot == 2 ? 0 : slot + 1;
      }
      for (int u = 5; u < 64; ++u) {
        const int t = u + 3;
        if (ready != ~0ull) {
          if (pend) {
            unsigned long long mk;
            asm volatile("s_waitcnt vmcnt(16)\n\ts_barrier\n\tv_cmp_ge_u32 %0, v200, %1" : "=s"(mk) : "v"(call * 8u) : "memory", "v200");
            ready |= mk;
          } else {
            asm volatile("s_waitcnt vmcnt(16)\n\ts_barrier" ::: "memory");
          }
          if (!((ready >> t) & 1)) {
            const unsigned long long t0 = __builtin_amdgcn_s_memrealtime();
            for (;;) {
              unsigned fv, tv;
              asm volatile("global_load_dword %0, %2, off sc0 sc1\n\tglobal_load_dword %1, %3, off sc1\n\ts_waitcnt vmcnt(0)"
                           : "=&v"(fv), "=&v"(tv) : "v"(fl), "v"(tmo) : "memory");
              ready |= __ballot(fv >= call * 8u);
              if ((ready >> t) & 1) break;
              if (rfl(tv) == call || __builtin_amdgcn_s_memrealtime() - t0 > DEADLINE_TICKS) {
                if (lane == 0) __hip_atomic_store((unsigned*)tmo, call, __ATOMIC_RELAXED, __HIP_MEMORY_SCOPE_AGENT);
                asm volatile("s_waitcnt vmcnt(0)" ::: "memory");
                ready = ~0ull; srdR[2] = 0;
                break;
              }
              __builtin_amdgcn_s_sleep(2);
            }
          }
          ready |= (2ull << t) - 1;
          pend = ((~ready >> t) & ((2ull << LOOK) - 1)) != 0;
          if (pend) SAMPLE();
        } else {
          asm volatile("s_waitcnt vmcnt(16)\n\ts_barrier" ::: "memory");
        }
        unsigned so = __builtin_amdgcn_readlane(tabA, t & 63);
        if (t >= 64) { srdR[2] = 0; so = 0; }
        const unsigned m = m0A + slot * A_ST;
        dma4<1, 3>(m, vR, vR, vR, vR, srdR, so);
        #pragma unroll
        for (int j = 1; j < 4; ++j) dma4<1, 0>(m + j * 4096, vR, vR, vR, vR, srdR, so + j * 4096);
        slot = slot == 2 ? 0 : slot + 1;
      }
      if (h == 0) STAMP(15, ((q0 - rt0) & 0xffff) | (((q1 - rt0) & 0xffff) << 16) | (((q2 - rt0) & 0xffff) << 32) | ((rt0 & 0xffff) << 48));
      STAMP(5 + 6 * h, spin); STAMP(6 + 6 * h, nspin); STAMP(7 + 6 * h, firstspin);
    } else {
      i32x4 srdW = make_srd(P.w, (unsigned)KTOT * NTOT * 2u);
      unsigned vB[4];
      #pragma unroll
      for (int i = 0; i < 4; ++i) {
        const unsigned ch = (lane & 15) ^ (((lane >> 4) << 2) | (i >> 1));
        vB[i] = (4 * i + (lane >> 4)) * 32768 + colb + (ch >> 3) * CSP + (ch & 7) * 16 - i * 1024;
      }
      auto issue = [&](int t, unsigned slot) {
        const unsigned so = __builtin_amdgcn_readlane(tabB, t & 63), m = B_RING + slot * B_ST;
        if (t >= 64) srdW[2] = 0;
        dma4<0, 3>(m, vB[0], vB[1], vB[2], vB[3], srdW, so);
        #pragma unroll
        for (int j = 1; j < 4; ++j) dma4<0, 0>(m + j * 4096, vB[0], vB[1], vB[2], vB[3], srdW, so + j * 16 * 32768);
      };
      unsigned callv, dummy = 0;
      asm volatile("global_load_dword %0, %1, off sc1" : "=v"(callv) : "v"(myws + WS_CTR + b * 128) : "memory");
      const i32x4 srdXL = make_srd(P.x + (size_t)rank * BLK, BLK);
      const unsigned xr = tm * 256 + (idx >> 1) * 16 + (lane & 15), vX = xr * 128, vXL = xr * 1024;
      const i32x4 srdWp = make_srd(P.w, (unsigned)KTOT * NTOT * 2u);
      const unsigned vW = WPF == 1 ? (idx * 2 + (lane >> 5)) * 32768 + xcd * 4096 + (lane & 31) * 128
                        : WPF == 2 ? ((idx & 7) * 8 + (lane >> 3)) * 32768 + (tn >> 2) * 1024 + (lane & 7) * 128
                                   : ((idx >> 4) * 32 + (lane >> 1)) * 32768 + xcd * 4096 + (((idx & 15) ^ (lane >> 2)) * 256) + (lane & 1) * 128;
      auto pfw = [&](int t) {
        const unsigned sw = __builtin_amdgcn_readlane(tabB, t);
        asm volatile("s_nop 4\n\tbuffer_load_dword %0, %1, %2, %3 offen" : "+v"(dummy) : "v"(vW), "s"(srdWp), "s"(sw) : "memory");
      };
      if (PF0 >= 1) pfw(0);
      if (PF0 >= 2) pfw(1);
      issue(0, 0);
      if (PF0 == 1) pfw(1);
      if (PF0) pfw(2);
      issue(1, 1);
      if (PF0) { pfw(3); asm volatile("s_waitcnt vmcnt(18)" : "+v"(callv) :: "memory"); }
      else asm volatile("s_waitcnt vmcnt(16)" : "+v"(callv) :: "memory");
      BAR();
      issue(2, 2);
      if (PF0) pfw(4);
      issue(3, 3);
      const i32x4 srdX = make_srd(myws + WS_X + ((rfl(callv) + 1) & 1) * XPAR, XPAR);
      auto pf = [&](int t) {
        if (WPF && t < 64 && t >= (PF0 ? 5 : 4)) {
          const unsigned sw = __builtin_amdgcn_readlane(tabB, t);
          asm volatile("s_nop 4\n\tbuffer_load_dword %0, %1, %2, %3 offen " PF_POL : "+v"(dummy) : "v"(vW), "s"(srdWp), "s"(sw) : "memory");
        }
        if (t < 64 && lane < 16) {
          const unsigned so = __builtin_amdgcn_readlane(tabA, t);
          if (t < 8) { if (!PF0) asm volatile("s_nop 4\n\tbuffer_load_dword %0, %1, %2, %3 offen sc0 sc1" : "+v"(dummy) : "v"(vXL), "s"(srdXL), "s"(so) : "memory"); }
          else if (XPF_REMOTE) asm volatile("s_nop 4\n\tbuffer_load_dword %0, %1, %2, %3 offen sc0 sc1" : "+v"(dummy) : "v"(vX), "s"(srdX), "s"(so) : "memory");
        } };
      int nxt = WPF ? (int)(idx % XPK) : 0;
      for (int t = 3; t < XPD + nxt; ++t) pf(t);
      for (int u = 0; u < 64; ++u) {
        asm volatile("s_waitcnt vmcnt(32)" ::: "memory"); BAR();
        if (BSL) __builtin_amdgcn_s_sleep(BSL);
        if (WPF && !PFFIRST) issue(u + 4, u & 3);
        if (u == nxt) {
          nxt += XPK;
          #pragma unroll
          for (int j = 0; j < XPK; ++j) pf(u + XPD + j);
        }
        if (!WPF || PFFIRST) issue(u + 4, u & 3);
      }
      asm volatile("s_waitcnt vmcnt(0)" :: "v"(dummy) : "memory");
      STAMP(14, ((q0 - rt0) & 0xffff) | (((q1 - rt0) & 0xffff) << 16) | (((q2 - rt0) & 0xffff) << 32) | ((rt0 & 0xffff) << 48));
    }
    return;
  }
  const unsigned wm = wid >> 1, wn = wid & 1, fr = lane & 15, fq = lane >> 4, q = (lane & 15) >> 2, p = lane & 3;
  const unsigned ra0 = A_RING + (wm * 128 + fr) * 128 + ((fq ^ (fr & 7)) * 16);
  const unsigned rb00 = B_RING + (8 * fq + q) * 256 + (((wn * 8 + p) ^ ((q << 2) | (fq & 1))) * 16) + (p & 1) * 8;
  const unsigned call = rfl(ld_agent(myws + WS_CTR + b * 128)) + 1;
  unsigned tmov = 0;
  int ulast = 62; asm volatile("" : "+s"(ulast));
  f32x4 acc[8][4];
  #pragma unroll
  for (int m = 0; m < 8; ++m)
    #pragma unroll
    for (int n = 0; n < 4; ++n) acc[m][n] = (f32x4){0.f, 0.f, 0.f, 0.f};
  bf16x8 af[8]; s16x4 b0l[4], b0h[4], b1l[4], b1h[4];
  #define MF(m, n_, L, H) MF2(m, (((m) & 1) ? 3 - (n_) : (n_)), L, H)
  #define MF2(m, n, L, H) do { acc[m][n] = __builtin_amdgcn_mfma_f32_16x16x32_bf16(__builtin_bit_cast(bf16x8, __builtin_shufflevector(L[n], H[n], 0, 1, 2, 3, 4, 5, 6, 7)), af[m], acc[m][n], 0, 0, 0); \
                              asm volatile("" : "+v"(acc[m][n])); SB(); } while (0)
  #define RA(dst, base, m) do { dst = rda((base) + (m) * 2048); SB(); } while (0)
  #define RL(L, n, ks) do { L[n] = trd(bb[n] + (ks) * 8192); SB(); } while (0)
  #define RH(H, n, ks) do { H[n] = trd(bb[n] + (ks) * 8192 + 1024); SB(); } while (0)
  #define PINV(x) do { asm volatile("" : "+v"(x)); SB(); } while (0)
  #define PINS(x) do { asm volatile("" : "+s"(x)); SB(); } while (0)
  unsigned bB[4], bb[4];
  #pragma unroll
  for (int n = 0; n < 4; ++n) { bB[n] = rb00 ^ ((n >> 1) * 64 + (n & 1) * 8); bb[n] = bB[n]; }
  unsigned a0 = ra0, a1 = ra0 ^ 64;
  BAR();
  #pragma unroll
  for (int n = 0; n < 4; ++n) { RL(b0l, n, 0); RH(b0h, n, 0); }
  #pragma unroll
  for (int m = 0; m < 7; ++m) RA(af[m], a0, m);
  unsigned sa = 0, sb = 0;
  for (int u = 0; u < 64; ++u) {
    MF(0, 0, b0l, b0h); RA(af[7], a0, 7); MF(0, 1, b0l, b0h); RL(b1l, 0, 1); MF(0, 2, b0l, b0h); RH(b1h, 0, 1); MF(0, 3, b0l, b0h);
    #pragma unroll
    for (int g = 1; g < 4; ++g) {
      MF(g, 0, b0l, b0h); RA(af[g - 1], a1, g - 1); MF(g, 1, b0l, b0h); RL(b1l, g, 1); MF(g, 2, b0l, b0h); RH(b1h, g, 1); MF(g, 3, b0l, b0h);
      if (g == 1) { sb = sb + B_ST; PINS(sb); }
      if (g == 2) { sb = sb & (4 * B_ST - 1); PINS(sb); }
    }
    #pragma unroll
    for (int g = 4; g < 8; ++g) {
      MF(g, 0, b0l, b0h); RA(af[g - 1], a1, g - 1); MF(g, 1, b0l, b0h); bb[g - 4] = bB[g - 4] + sb; PINV(bb[g - 4]); MF(g, 2, b0l, b0h);
      if (g == 4) { sa = sa == 2 * A_ST ? 0 : sa + A_ST; PINS(sa); }
      if (g == 5) { a0 = ra0 + sa; PINV(a0); }
      MF(g, 3, b0l, b0h);
    }
    MF(0, 0, b1l, b1h); RA(af[7], a1, 7); MF(0, 1, b1l, b1h); MF(0, 2, b1l, b1h); a1 = a0 ^ 64; PINV(a1); MF(0, 3, b1l, b1h);
    MF(1, 0, b1l, b1h); MF(1, 1, b1l, b1h);
    if (u == ulast) tmov = ld_agent(myws + WS_TMO);
    SB();
    MF(1, 2, b1l, b1h); MF(1, 3, b1l, b1h);
    asm volatile("s_waitcnt lgkmcnt(0)\n\ts_barrier" ::: "memory"); SB();
    MF(2, 0, b1l, b1h); RA(af[0], a0, 0); MF(2, 1, b1l, b1h); RL(b0l, 0, 0); MF(2, 2, b1l, b1h); RH(b0h, 0, 0); MF(2, 3, b1l, b1h); RA(af[1], a0, 1);
    #pragma unroll
    for (int g = 3; g < 8; ++g) {
      MF(g, 0, b1l, b1h); RA(af[g - 1], a0, g - 1); MF(g, 1, b1l, b1h); if (g < 6) RL(b0l, g - 2, 0); MF(g, 2, b1l, b1h); if (g < 6) RH(b0h, g - 2, 0); MF(g, 3, b1l, b1h);
    }
  }
  if (rfl(tmov) == call) {
    asm volatile("" ::: "memory");
    const float qn = __builtin_nanf("");
    #pragma unroll
    for (int m = 0; m < 8; ++m)
      #pragma unroll
      for (int n = 0; n < 4; ++n) acc[m][n] = (f32x4){qn, qn, qn, qn};
  }
  char* orow = P.out + (size_t)(tm * 256 + wm * 128 + fr) * NTOT * 2 + colb + wn * CSP + fq * 16;
  const bool odd = fq & 1;
  orow += (fr & 8) ? (long)64 - (long)8 * NTOT * 2 : 0;
  SB();
  #pragma unroll
  for (int m = 0; m < 8; ++m) {
    u32x4 v[2];
    #pragma unroll
    for (int pr = 0; pr < 2; ++pr) {
      const f32x4 a = acc[m][2 * pr], c = acc[m][2 * pr + 1];
      const unsigned a0 = relu2(cvtpk(a[0], a[1])), a1 = relu2(cvtpk(a[2], a[3])), c0 = relu2(cvtpk(c[0], c[1])), c1 = relu2(cvtpk(c[2], c[3]));
      v[pr][0] = odd ? c0 : a0; v[pr][1] = odd ? c1 : a1; v[pr][2] = odd ? a0 : c0; v[pr][3] = odd ? a1 : c1;
    }
    char* pp = orow + (size_t)m * 16 * NTOT * 2;
    u32x4 lo, hi;
    #pragma unroll
    for (int e = 0; e < 4; ++e) {
      lo[e] = __builtin_amdgcn_update_dpp(v[0][e], v[1][e], 0x128, 0xf, 0xc, false);
      hi[e] = __builtin_amdgcn_update_dpp(v[1][e], v[0][e], 0x128, 0xf, 0x3, false);
    }
    asm volatile("global_store_dwordx4 %0, %1, off " OUT_POL "\n\ts_nop 1" :: "v"(pp), "v"(lo) : "memory");
    asm volatile("global_store_dwordx4 %0, %1, off " OUT_POL "\n\ts_nop 1" :: "v"(pp + (size_t)8 * NTOT * 2), "v"(hi) : "memory");
    SB();
  }
  if (tid == 0) __hip_atomic_store((unsigned*)(myws + WS_CTR + b * 128), call, __ATOMIC_RELAXED, __HIP_MEMORY_SCOPE_AGENT);
}

extern "C" void kernel_launch(int rank, int nranks, ncclComm_t comm, hipStream_t stream,
                              void* const* d_in, const int* in_sizes, int n_in,
                              void* d_out, int out_size, const int* in_dtypes, int out_dtype,
                              void* const* d_out_all, void* const* d_ws_all, size_t ws_bytes) {
  (void)comm; (void)in_sizes; (void)n_in; (void)out_size; (void)in_dtypes; (void)out_dtype; (void)d_out_all;
  if (nranks != NR || (unsigned)rank >= (unsigned)NR || ws_bytes < WS_NEED) { fprintf(stderr, "unexpected launch geometry\n"); return; }
  static bool init = false;
  if (!init) {
    hipError_t e = hipFuncSetAttribute((const void*)a2a_gemm_relu, hipFuncAttributeMaxDynamicSharedMemorySize, (int)LDS_BYTES);
    if (e != hipSuccess) { fprintf(stderr, "hipFuncSetAttribute: %s\n", hipGetErrorString(e)); return; }
    init = true;
  }
  Params p;
  p.x = (const char*)d_in[0]; p.w = (const char*)d_in[1]; p.out = (char*)d_out;
  for (int i = 0; i < NR; ++i) p.ws[i] = (char*)d_ws_all[i];
  p.rank = rank; p.pad = 0;
  hipLaunchKernelGGL(a2a_gemm_relu, dim3(256), dim3(512), LDS_BYTES, stream, p);
  hipError_t e = hipGetLastError();
  if (e != hipSuccess) fprintf(stderr, "launch: %s\n", hipGetErrorString(e));
}
```

```cpp
#include <hip/hip_runtime.h>
#include <rccl/rccl.h>
#include <cstdio>
#include <cstdint>

typedef __bf16 bf16x8 __attribute__((ext_vector_type(8)));
typedef __bf16 bf16x2 __attribute__((ext_vector_type(2)));
typedef short s16x4 __attribute__((ext_vector_type(4)));
typedef short s16x2 __attribute__((ext_vector_type(2)));
typedef float f32x4 __attribute__((ext_vector_type(4)));
typedef float f32x2 __attribute__((ext_vector_type(2)));
typedef unsigned u32x4 __attribute__((ext_vector_type(4)));
typedef int i32x4 __attribute__((ext_vector_type(4)));
#define LAS __attribute__((address_space(3)))
#define DI __device__ __forceinline__
#define SB() __builtin_amdgcn_sched_barrier(0)
#define BAR() asm volatile("s_barrier" ::: "memory")

#define PFD 6
#define SW 8
#define BSL 9
#define PT0 120
#define PT1 30
#define LOOK 3
constexpr int NR = 8, KTOT = 4096, NTOT = 16384;
constexpr unsigned A_RING = 0, A_ST = 32768, B_RING = 3 * 32768, B_ST = 16384, LDS_BYTES = 163840;
constexpr unsigned WS_FLAGS = 0;
constexpr unsigned WS_TMO = 4096;
constexpr unsigned WS_CTR = 8192;
constexpr unsigned WS_X = 1u << 20;
constexpr unsigned BLK = 524288;
constexpr unsigned XPAR = 8 * BLK;
constexpr size_t WS_NEED = (size_t)WS_X + 2 * (size_t)XPAR;
constexpr unsigned long long DEADLINE_TICKS = 100000000ull;

#define STAMP(i, v) do {} while (0)
struct Params { const char* x; const char* w; char* out; char* ws[NR]; int rank; int pad; };

DI unsigned rfl(unsigned x) { return __builtin_amdgcn_readfirstlane(x); }
DI unsigned cvtpk(float lo, float hi) { f32x2 v = {lo, hi}; bf16x2 b = __builtin_convertvector(v, bf16x2); return __builtin_bit_cast(unsigned, b); }
DI unsigned relu2(unsigned x) { const s16x2 v = __builtin_bit_cast(s16x2, x), z = {0, 0}; return __builtin_bit_cast(unsigned, __builtin_elementwise_max(v, z)); }
DI i32x4 make_srd(const void* p, unsigned bytes) {
  unsigned long long a = (unsigned long long)p;
  i32x4 r; r[0] = rfl((unsigned)a); r[1] = rfl((unsigned)(a >> 32)) & 0xffff; r[2] = rfl(bytes); r[3] = 0x00020000; return r;
}
#define DMA1(v, off, pol) "buffer_load_dwordx4 " v ", %5, %6 offen offset:" off pol " lds\n\t"
#define DMA4_ASM(pol) asm volatile("s_mov_b32 m0, %0\n\ts_nop %7\n\t" DMA1("%1", "0", pol) DMA1("%2", "1024", pol) DMA1("%3", "2048", pol) DMA1("%4", "3072", pol) \
    :: "s"(m0v), "v"(v0), "v"(v1), "v"(v2), "v"(v3), "s"(srd), "s"(so), "i"(PAD) : "memory")
template <int SC, int PAD> DI void dma4(unsigned m0v, unsigned v0, unsigned v1, unsigned v2, unsigned v3, i32x4 srd, unsigned so) {
  if constexpr (SC) DMA4_ASM(" sc0 sc1"); else DMA4_ASM("");
}
DI s16x4 trd(unsigned a) { return __builtin_amdgcn_ds_read_tr16_b64_v4i16((LAS s16x4*)(size_t)a); }
DI bf16x8 rda(unsigned a) { return *(const LAS bf16x8*)(size_t)a; }
DI unsigned ld_agent(const char* p) { return __hip_atomic_load((const unsigned*)p, __ATOMIC_RELAXED, __HIP_MEMORY_SCOPE_AGENT); }

__global__ __launch_bounds__(512, 2) void a2a_gemm_relu(const Params P) {
  const int tid = threadIdx.x, lane = tid & 63;
  const unsigned wid = rfl(tid >> 6);
  const unsigned b = blockIdx.x, xcd = b & 7, idx = b >> 3, tn = xcd * 16 + (idx >> 1), tm = idx & 1;
  const unsigned rank = (unsigned)P.rank & 7u;
  char* const myws = P.ws[rank];
  const unsigned colb = (tn >> 2) * 1024 + (tn & 3) * 128;

  if (wid >= 4) {
    unsigned pos = lane, tabA, tabB;
    if (tm && lane < SW) {
      if (lane & 1) { if (lane >= (lane < 8 ? 3u : 11u)) pos = lane - 3; }
      else if (lane + 3 < (lane < 8 ? 8u : (unsigned)SW)) pos = lane + 3;
    }
    { const unsigned u = pos; unsigned src, kt;
      if (u < 8) { src = rank; kt = u; tabA = kt * 128; }
      else { const unsigned j = u - 8; kt = (j * 37) >> 8; src = (rank + (j - kt * 7 + 1)) & 7u; tabA = src * BLK + kt * 65536u; }
      tabB = (src * 512u + kt * 64u) * 32768u; }
    if (wid < 6) {
      const unsigned h = wid - 4;
      unsigned callv;
      asm volatile("global_load_dword %0, %1, off sc1" : "=v"(callv) : "v"(myws + WS_CTR + b * 128) : "memory");
      const i32x4 srdL = make_srd(P.x + (size_t)rank * BLK, BLK);
      const unsigned arow = tm * 256 + h * 128 + (lane >> 3), ach = ((lane & 7) ^ (lane >> 3)) * 16;
      unsigned vL[4];
      #pragma unroll
      for (int i = 0; i < 4; ++i) vL[i] = (arow + 8 * i) * 1024 + ach - i * 1024;
      const unsigned vR = arow * 128 + ach, m0A = A_RING + h * 16384;
      auto issueL = [&](int t, unsigned slot) {
        const unsigned so = __builtin_amdgcn_readlane(tabA, t), m = m0A + slot * A_ST;
        dma4<1, 3>(m, vL[0], vL[1], vL[2], vL[3], srdL, so);
        #pragma unroll
        for (int j = 1; j < 4; ++j) dma4<1, 0>(m + j * 4096, vL[0], vL[1], vL[2], vL[3], srdL, so + j * 32768);
      };
      issueL(0, 0);
      issueL(1, 1);
      asm volatile("s_waitcnt vmcnt(16)" : "+v"(callv) :: "memory");
      BAR();
      issueL(2, 2);
      const unsigned call = rfl(callv) + 1;
      i32x4 srdR = make_srd(myws + WS_X + (call & 1) * XPAR, XPAR);
      const char* fl = myws + WS_FLAGS + (tm * 2 + h) * 256 + pos * 4;
      const char* tmo = myws + WS_TMO;
      #define SAMPLE() asm volatile("global_load_dword v200, %0, off sc0 sc1" :: "v"(fl) : "memory", "v200")
      unsigned long long ready = 0xffull;
      bool pend = true;
      unsigned slot = 0;
      for (int u = 0; u < 5; ++u) {
        asm volatile("s_waitcnt vmcnt(16)" ::: "memory"); BAR();
        if (u == 4) SAMPLE();
        issueL(u + 3, slot);
        slot = slot == 2 ? 0 : slot + 1;
      }
      for (int u = 5; u < 63; ++u) {
        const int t = u + 3;
        if (ready != ~0ull) {
          if (pend) {
            unsigned long long mk;
            asm volatile("s_waitcnt vmcnt(16)\n\ts_barrier\n\tv_cmp_ge_u32 %0, v200, %1" : "=s"(mk) : "v"(call * 8u) : "memory", "v200");
            ready |= mk;
          } else {
            asm volatile("s_waitcnt vmcnt(16)\n\ts_barrier" ::: "memory");
          }
          if (!((ready >> t) & 1)) {
            const unsigned long long t0 = __builtin_amdgcn_s_memrealtime();
            for (;;) {
              unsigned fv, tv;
              asm volatile("global_load_dword %0, %2, off sc0 sc1\n\tglobal_load_dword %1, %3, off sc1\n\ts_waitcnt vmcnt(0)"
                           : "=&v"(fv), "=&v"(tv) : "v"(fl), "v"(tmo) : "memory");
              ready |= __ballot(fv >= call * 8u);
              if ((ready >> t) & 1) break;
              if (rfl(tv) == call || __builtin_amdgcn_s_memrealtime() - t0 > DEADLINE_TICKS) {
                if (lane == 0) __hip_atomic_store((unsigned*)tmo, call, __ATOMIC_RELAXED, __HIP_MEMORY_SCOPE_AGENT);
                asm volatile("s_waitcnt vmcnt(0)" ::: "memory");
                ready = ~0ull; srdR[2] = 0;
                break;
              }
              __builtin_amdgcn_s_sleep(2);
            }
          }
          ready |= (2ull << t) - 1;
          pend = ((~ready >> t) & ((2ull << LOOK) - 1)) != 0;
          if (pend) SAMPLE();
        } else {
          asm volatile("s_waitcnt vmcnt(16)\n\ts_barrier" ::: "memory");
        }
        unsigned so = __builtin_amdgcn_readlane(tabA, t & 63);
        if (t >= 64) { srdR[2] = 0; so = 0; }
        const unsigned m = m0A + slot * A_ST;
        dma4<1, 3>(m, vR, vR, vR, vR, srdR, so);
        #pragma unroll
        for (int j = 1; j < 4; ++j) dma4<1, 0>(m + j * 4096, vR, vR, vR, vR, srdR, so + j * 4096);
        slot = slot == 2 ? 0 : slot + 1;
      }
      if (h == 0) STAMP(15, ((q0 - rt0) & 0xffff) | (((q1 - rt0) & 0xffff) << 16) | (((q2 - rt0) & 0xffff) << 32) | ((rt0 & 0xffff) << 48));
      STAMP(5 + 6 * h, spin); STAMP(6 + 6 * h, nspin); STAMP(7 + 6 * h, firstspin);
    } else if (wid == 6) {
      i32x4 srdW = make_srd(P.w, (unsigned)KTOT * NTOT * 2u);
      const i32x4 srdWp = srdW, srdXL = make_srd(P.x + (size_t)rank * BLK, BLK);
      unsigned vB[4], dummy = 0;
      #pragma unroll
      for (int i = 0; i < 4; ++i) {
        const unsigned ch = (lane & 15) ^ (((lane >> 4) << 2) | (i >> 1));
        vB[i] = (4 * i + (lane >> 4)) * 32768 + colb + (ch >> 3) * 512 + (ch & 7) * 16 - i * 1024;
      }
      const unsigned vW = ((idx & 7) * 8 + (lane >> 3)) * 32768 + (tn >> 2) * 1024 + (lane & 7) * 128;
      const unsigned vXL = (tm * 256 + (idx >> 1) * 16 + (lane & 15)) * 1024 + (lane >> 4) * 128;
      auto issue = [&](int t, unsigned slot) {
        const unsigned so = __builtin_amdgcn_readlane(tabB, t & 63), m = B_RING + slot * B_ST;
        dma4<0, 3>(m, vB[0], vB[1], vB[2], vB[3], srdW, so);
        #pragma unroll
        for (int j = 1; j < 4; ++j) dma4<0, 0>(m + j * 4096, vB[0], vB[1], vB[2], vB[3], srdW, so + j * 16 * 32768);
      };
      auto pfw = [&](int t) {
        const unsigned sw = __builtin_amdgcn_readlane(tabB, t);
        asm volatile("s_nop 4\n\tbuffer_load_dword %0, %1, %2, %3 offen" : "+v"(dummy) : "v"(vW), "s"(srdWp), "s"(sw) : "memory");
      };
      issue(0, 0);
      issue(1, 1);
      asm volatile("s_waitcnt vmcnt(16)" ::: "memory");
      BAR();
      issue(2, 2); issue(3, 3);
      asm volatile("buffer_load_dword %0, %1, %2, 0 offen offset:512" : "+v"(dummy) : "v"(vXL), "s"(srdXL) : "memory");
      for (int t = 4; t < PFD; ++t) if (t >= SW) pfw(t);
      for (int u = 0; u < 63; ++u) {
        asm volatile("s_waitcnt vmcnt(32)" ::: "memory"); BAR();
        if (u >= 60) srdW[2] = 0;
        if (u + 4 < SW) __builtin_amdgcn_s_sleep(BSL);
        issue(u + 4, u & 3);
        if (u + PFD >= SW && u + PFD < 64) pfw(u + PFD);
      }
      asm volatile("s_waitcnt vmcnt(0)" :: "v"(dummy) : "memory");
      STAMP(14, ((q0 - rt0) & 0xffff) | (((q1 - rt0) & 0xffff) << 16) | (((q2 - rt0) & 0xffff) << 32) | ((rt0 & 0xffff) << 48));
    } else {
      int nbar = 0;
      unsigned long long tb = __builtin_amdgcn_s_memrealtime() + (PT0 - PT1);
      auto bar = [&]() { BAR(); ++nbar; tb = __builtin_amdgcn_s_memrealtime(); };
      auto drain = [&]() {
        while ((__builtin_amdgcn_s_getreg(7 | (31 << 11)) & 0xC0000Fu) != 0 && nbar < 60) {
          if ((long long)(__builtin_amdgcn_s_memrealtime() - tb) > PT1) bar();
          else __builtin_amdgcn_s_sleep(1);
        }
        asm volatile("s_waitcnt vmcnt(0)" ::: "memory");
      };
      if (b < 224) {
        unsigned callv;
        u32x4 d[16];
        {
          const unsigned s = (b >> 5) + 1, row = (b & 31) * 16, dest = (rank - s) & 7u;
          const char* src = P.x + (size_t)dest * BLK + (size_t)(row + (lane >> 3)) * 1024 + (lane & 7) * 16;
          asm volatile("global_load_dword %0, %3, off sc1\n\tglobal_load_dwordx4 %1, %4, off\n\tglobal_load_dwordx4 %2, %5, off"
                       : "=&v"(callv), "=&v"(d[0]), "=&v"(d[1]) : "v"(myws + WS_CTR + b * 128), "v"(src), "v"(src + 8192) : "memory");
          char* const pws = P.ws[dest];
          drain();
          asm volatile("" : "+v"(callv), "+v"(d[0]), "+v"(d[1]));
          char* dst = pws + WS_X + ((rfl(callv) + 1) & 1) * XPAR + rank * BLK + row * 128 + lane * 16;
          asm volatile("global_store_dwordx4 %0, %1, off sc0 sc1\n\tglobal_store_dwordx4 %0, %2, off offset:1024 sc0 sc1\n\ts_nop 1" :: "v"(dst), "v"(d[0]), "v"(d[1]) : "memory");
          drain();
          if (lane == 0) __hip_atomic_fetch_add((unsigned*)(pws + WS_FLAGS + (row >> 7) * 256 + (8 + s - 1) * 4), 1u, __ATOMIC_RELAXED, __HIP_MEMORY_SCOPE_SYSTEM);
          STAMP(8, rp0); STAMP(9, RT()); STAMP(10, RT());
        }
        if (b < 196) {
          const unsigned call = rfl(callv) + 1;
          const unsigned kt = b / 28 + 1, r = b % 28, s = (r >> 2) + 1, qd = r & 3, dest = (rank - s) & 7u;
          const char* src = P.x + (size_t)dest * BLK + (size_t)(qd * 128 + (lane >> 3)) * 1024 + kt * 128 + (lane & 7) * 16;
          const int su = (int)(4 * kt + qd);
          while (nbar < su - 2) bar();
          #pragma unroll
          for (int i = 0; i < 16; i += 4)
            asm volatile("global_load_dwordx4 %0, %4, off\n\tglobal_load_dwordx4 %1, %5, off\n\tglobal_load_dwordx4 %2, %6, off\n\tglobal_load_dwordx4 %3, %7, off"
                         : "=&v"(d[i]), "=&v"(d[i + 1]), "=&v"(d[i + 2]), "=&v"(d[i + 3])
                         : "v"(src + i * 8192), "v"(src + (i + 1) * 8192), "v"(src + (i + 2) * 8192), "v"(src + (i + 3) * 8192) : "memory");
          char* const pws = P.ws[dest];
          char* dst = pws + WS_X + (call & 1) * XPAR + rank * BLK + kt * 65536 + qd * 16384 + lane * 16;
          drain();
          #pragma unroll
          for (int i = 0; i < 16; i += 4) asm volatile("" : "+v"(d[i]), "+v"(d[i + 1]), "+v"(d[i + 2]), "+v"(d[i + 3]));
          while (nbar < su) bar();
          #pragma unroll
          for (int i = 0; i < 16; ++i)
            asm volatile("global_store_dwordx4 %0, %1, off sc0 sc1\n\ts_nop 1" :: "v"(dst + i * 1024), "v"(d[i]) : "memory");
          tb -= PT1;
          drain();
          if (lane == 0)
            __hip_atomic_store((unsigned*)(pws + WS_FLAGS + qd * 256 + (8 + kt * 7 + s - 1) * 4), call * 8u, __ATOMIC_RELAXED, __HIP_MEMORY_SCOPE_SYSTEM);
        }
      }
    }
    return;
  }
  const unsigned wm = wid >> 1, wn = wid & 1, fr = lane & 15, fq = lane >> 4, q = (lane & 15) >> 2, p = lane & 3;
  const unsigned ra0 = A_RING + (wm * 128 + fr) * 128 + ((fq ^ (fr & 7)) * 16);
  const unsigned rb00 = B_RING + (8 * fq + q) * 256 + (((wn * 8 + p) ^ ((q << 2) | (fq & 1))) * 16) + (p & 1) * 8;
  const unsigned call = rfl(ld_agent(myws + WS_CTR + b * 128)) + 1;
  unsigned tmov = 0;
  int ulast = 61; asm volatile("" : "+s"(ulast));
  f32x4 acc[8][4];
  #pragma unroll
  for (int m = 0; m < 8; ++m)
    #pragma unroll
    for (int n = 0; n < 4; ++n) acc[m][n] = (f32x4){0.f, 0.f, 0.f, 0.f};
  bf16x8 af[8]; s16x4 b0l[4], b0h[4], b1l[4], b1h[4];
  #define MF(m, n_, L, H) MF2(m, (((m) & 1) ? 3 - (n_) : (n_)), L, H)
  #define MF2(m, n, L, H) do { acc[m][n] = __builtin_amdgcn_mfma_f32_16x16x32_bf16(__builtin_bit_cast(bf16x8, __builtin_shufflevector(L[n], H[n], 0, 1, 2, 3, 4, 5, 6, 7)), af[m], acc[m][n], 0, 0, 0); \
                              asm volatile("" : "+v"(acc[m][n])); SB(); } while (0)
  #define RA(dst, base, m) do { dst = rda((base) + (m) * 2048); SB(); } while (0)
  #define RL(L, n, ks) do { L[n] = trd(bb[n] + (ks) * 8192); SB(); } while (0)
  #define RH(H, n, ks) do { H[n] = trd(bb[n] + (ks) * 8192 + 1024); SB(); } while (0)
  #define PINV(x) do { asm volatile("" : "+v"(x)); SB(); } while (0)
  #define PINS(x) do { asm volatile("" : "+s"(x)); SB(); } while (0)
  unsigned bB[4], bb[4];
  #pragma unroll
  for (int n = 0; n < 4; ++n) { bB[n] = rb00 ^ ((n >> 1) * 64 + (n & 1) * 8); bb[n] = bB[n]; }
  unsigned a0 = ra0, a1 = ra0 ^ 64;
  BAR();
  #pragma unroll
  for (int n = 0; n < 4; ++n) { RL(b0l, n, 0); RH(b0h, n, 0); }
  #pragma unroll
  for (int m = 0; m < 7; ++m) RA(af[m], a0, m);
  unsigned sa = 0, sb = 0;
  for (int u = 0; u < 63; ++u) {
    MF(0, 0, b0l, b0h); RA(af[7], a0, 7); MF(0, 1, b0l, b0h); RL(b1l, 0, 1); MF(0, 2, b0l, b0h); RH(b1h, 0, 1); MF(0, 3, b0l, b0h);
    #pragma unroll
    for (int g = 1; g < 4; ++g) {
      MF(g, 0, b0l, b0h); RA(af[g - 1], a1, g - 1); MF(g, 1, b0l, b0h); RL(b1l, g, 1); MF(g, 2, b0l, b0h); RH(b1h, g, 1); MF(g, 3, b0l, b0h);
      if (g == 1) { sb = sb + B_ST; PINS(sb); }
      if (g == 2) { sb = sb & (4 * B_ST - 1); PINS(sb); }
    }
    #pragma unroll
    for (int g = 4; g < 8; ++g) {
      MF(g, 0, b0l, b0h); RA(af[g - 1], a1, g - 1); MF(g, 1, b0l, b0h); bb[g - 4] = bB[g - 4] + sb; PINV(bb[g - 4]); MF(g, 2, b0l, b0h);
      if (g == 4) { sa = sa == 2 * A_ST ? 0 : sa + A_ST; PINS(sa); }
      if (g == 5) { a0 = ra0 + sa; PINV(a0); }
      MF(g, 3, b0l, b0h);
    }
    MF(0, 0, b1l, b1h); RA(af[7], a1, 7); MF(0, 1, b1l, b1h); MF(0, 2, b1l, b1h); a1 = a0 ^ 64; PINV(a1); MF(0, 3, b1l, b1h);
    MF(1, 0, b1l, b1h); MF(1, 1, b1l, b1h);
    MF(1, 2, b1l, b1h); MF(1, 3, b1l, b1h);
    asm volatile("s_waitcnt lgkmcnt(0)\n\ts_barrier" ::: "memory"); SB();
    if (u == ulast) tmov = ld_agent(myws + WS_TMO);
    SB();
    MF(2, 0, b1l, b1h); RA(af[0], a0, 0); MF(2, 1, b1l, b1h); RL(b0l, 0, 0); MF(2, 2, b1l, b1h); RH(b0h, 0, 0); MF(2, 3, b1l, b1h); RA(af[1], a0, 1);
    #pragma unroll
    for (int g = 3; g < 8; ++g) {
      MF(g, 0, b1l, b1h); RA(af[g - 1], a0, g - 1); MF(g, 1, b1l, b1h); if (g < 6) RL(b0l, g - 2, 0); MF(g, 2, b1l, b1h); if (g < 6) RH(b0h, g - 2, 0); MF(g, 3, b1l, b1h);
    }
  }
  bf16x8 ag[8];
  #pragma unroll
  for (int n = 0; n < 4; ++n) { RL(b1l, n, 1); RH(b1h, n, 1); }
  RA(ag[0], a1, 0); RA(ag[1], a1, 1); RA(af[7], a0, 7);
  #pragma unroll
  for (int m = 2; m < 8; ++m) RA(ag[m], a1, m);
  const bool bad = rfl(tmov) == call;
  char* orow = P.out + (size_t)(tm * 256 + wm * 128 + (fr & 7)) * NTOT * 2 + colb + wn * 512 + (fr & 8) * 8 + fq * 16;
  const bool odd = fq & 1;
  auto epi = [&](const f32x4 (&ac)[4], int m) {
    u32x4 v[2], lo, hi;
    #pragma unroll
    for (int pr = 0; pr < 2; ++pr) {
      const f32x4 a = ac[2 * pr], c = ac[2 * pr + 1];
      const unsigned a0 = relu2(cvtpk(a[0], a[1])), a1 = relu2(cvtpk(a[2], a[3])), c0 = relu2(cvtpk(c[0], c[1])), c1 = relu2(cvtpk(c[2], c[3]));
      v[pr][0] = odd ? c0 : a0; v[pr][1] = odd ? c1 : a1; v[pr][2] = odd ? a0 : c0; v[pr][3] = odd ? a1 : c1;
    }
    #pragma unroll
    for (int e = 0; e < 4; ++e) {
      lo[e] = __builtin_amdgcn_update_dpp(v[0][e], v[1][e], 0x128, 0xf, 0xc, false);
      hi[e] = __builtin_amdgcn_update_dpp(v[1][e], v[0][e], 0x128, 0xf, 0x3, false);
    }
    char* pp = orow + (size_t)m * 16 * NTOT * 2;
    asm volatile("global_store_dwordx4 %0, %1, off sc1\n\ts_nop 1" :: "v"(pp), "v"(lo) : "memory");
    asm volatile("global_store_dwordx4 %0, %1, off sc1\n\ts_nop 1" :: "v"(pp + (size_t)8 * NTOT * 2), "v"(hi) : "memory");
  };
  if (bad) {
    const u32x4 qn = {0x7fc07fc0u, 0x7fc07fc0u, 0x7fc07fc0u, 0x7fc07fc0u};
    for (int i = 0; i < 16; ++i) *(u32x4*)(orow + (size_t)i * 8 * NTOT * 2) = qn;
  } else {
    SB();
    #pragma unroll
    for (int m = 0; m < 8; ++m) {
      #pragma unroll
      for (int n = 0; n < 4; ++n) {
        acc[m][n] = __builtin_amdgcn_mfma_f32_16x16x32_bf16(__builtin_bit_cast(bf16x8, __builtin_shufflevector(b0l[n], b0h[n], 0, 1, 2, 3, 4, 5, 6, 7)), af[m], acc[m][n], 0, 0, 0);
        asm volatile("" : "+v"(acc[m][n]));
      }
      #pragma unroll
      for (int n = 0; n < 4; ++n) {
        acc[m][n] = __builtin_amdgcn_mfma_f32_16x16x32_bf16(__builtin_bit_cast(bf16x8, __builtin_shufflevector(b1l[n], b1h[n], 0, 1, 2, 3, 4, 5, 6, 7)), ag[m], acc[m][n], 0, 0, 0);
        asm volatile("" : "+v"(acc[m][n]));
      }
      if (m > 0) {
        epi(acc[m - 1], m - 1);
        #pragma unroll
        for (int k = 0; k < 8; ++k) { __builtin_amdgcn_sched_group_barrier(0x8, 1, 0); __builtin_amdgcn_sched_group_barrier(0x2, 5, 0); }
      }
      SB();
    }
    epi(acc[7], 7);
  }
  if (tid == 0) __hip_atomic_store((unsigned*)(myws + WS_CTR + b * 128), call, __ATOMIC_RELAXED, __HIP_MEMORY_SCOPE_AGENT);
}

extern "C" void kernel_launch(int rank, int nranks, ncclComm_t comm, hipStream_t stream,
                              void* const* d_in, const int* in_sizes, int n_in,
                              void* d_out, int out_size, const int* in_dtypes, int out_dtype,
                              void* const* d_out_all, void* const* d_ws_all, size_t ws_bytes) {
  (void)comm; (void)in_sizes; (void)n_in; (void)out_size; (void)in_dtypes; (void)out_dtype; (void)d_out_all;
  if (nranks != NR || (unsigned)rank >= (unsigned)NR || ws_bytes < WS_NEED) { fprintf(stderr, "unexpected launch geometry\n"); return; }
  static bool init = false;
  if (!init) {
    hipError_t e = hipFuncSetAttribute((const void*)a2a_gemm_relu, hipFuncAttributeMaxDynamicSharedMemorySize, (int)LDS_BYTES);
    if (e != hipSuccess) { fprintf(stderr, "hipFuncSetAttribute: %s\n", hipGetErrorString(e)); return; }
    init = true;
  }
  Params p;
  p.x = (const char*)d_in[0]; p.w = (const char*)d_in[1]; p.out = (char*)d_out;
  for (int i = 0; i < NR; ++i) p.ws[i] = (char*)d_ws_all[i];
  p.rank = rank; p.pad = 0;
  hipLaunchKernelGGL(a2a_gemm_relu, dim3(256), dim3(512), LDS_BYTES, stream, p);
  hipError_t e = hipGetLastError();
  if (e != hipSuccess) fprintf(stderr, "launch: %s\n", hipGetErrorString(e));
}
```

```cpp
#include <hip/hip_runtime.h>
#include <rccl/rccl.h>
#include <cstdio>
#include <cstdint>

typedef __bf16 bf16x8 __attribute__((ext_vector_type(8)));
typedef __bf16 bf16x2 __attribute__((ext_vector_type(2)));
typedef short s16x4 __attribute__((ext_vector_type(4)));
typedef short s16x2 __attribute__((ext_vector_type(2)));
typedef float f32x4 __attribute__((ext_vector_type(4)));
typedef float f32x2 __attribute__((ext_vector_type(2)));
typedef unsigned u32x4 __attribute__((ext_vector_type(4)));
typedef int i32x4 __attribute__((ext_vector_type(4)));
#define LAS __attribute__((address_space(3)))
#define DI __device__ __forceinline__
#define SB() __builtin_amdgcn_sched_barrier(0)
#define BAR() asm volatile("s_barrier" ::: "memory")

#define PFD 6
#define PFX 6
#define RM0 8
#define RMP 2
#define PPER 1000
#define PLEN 0
#define PACE 84
#define PACEN 38
#define PJX 0
#define PJW 0
#define PACE0 330
#define SW 64
#define BSL 9
#define PT0 120
#define PT1 30
#define LOOK 3
constexpr int NR = 8, KTOT = 4096, NTOT = 16384;
constexpr unsigned A_RING = 0, A_ST = 32768, B_RING = 3 * 32768, B_ST = 16384, LDS_BYTES = 163840;
constexpr unsigned WS_FLAGS = 0;
constexpr unsigned WS_TMO = 4096;
constexpr unsigned WS_CTR = 8192;
constexpr unsigned WS_X = 1u << 20;
constexpr unsigned BLK = 524288;
constexpr unsigned XPAR = 8 * BLK;
constexpr size_t WS_NEED = (size_t)WS_X + 2 * (size_t)XPAR;
constexpr unsigned long long DEADLINE_TICKS = 100000000ull;

#define STAMP(i, v) do {} while (0)
struct Params { const char* x; const char* w; char* out; char* ws[NR]; int rank; int pad; };

DI unsigned rfl(unsigned x) { return __builtin_amdgcn_readfirstlane(x); }
DI unsigned cvtpk(float lo, float hi) { f32x2 v = {lo, hi}; bf16x2 b = __builtin_convertvector(v, bf16x2); return __builtin_bit_cast(unsigned, b); }
DI unsigned relu2(unsigned x) { const s16x2 v = __builtin_bit_cast(s16x2, x), z = {0, 0}; return __builtin_bit_cast(unsigned, __builtin_elementwise_max(v, z)); }
DI i32x4 make_srd(const void* p, unsigned bytes) {
  unsigned long long a = (unsigned long long)p;
  i32x4 r; r[0] = rfl((unsigned)a); r[1] = rfl((unsigned)(a >> 32)) & 0xffff; r[2] = rfl(bytes); r[3] = 0x00020000; return r;
}
#define DMA1(v, off, pol) "buffer_load_dwordx4 " v ", %5, %6 offen offset:" off pol " lds\n\t"
#define DMA4_ASM(pol) asm volatile("s_mov_b32 m0, %0\n\ts_nop %7\n\t" DMA1("%1", "0", pol) DMA1("%2", "1024", pol) DMA1("%3", "2048", pol) DMA1("%4", "3072", pol) \
    :: "s"(m0v), "v"(v0), "v"(v1), "v"(v2), "v"(v3), "s"(srd), "s"(so), "i"(PAD) : "memory")
template <int SC, int PAD> DI void dma4(unsigned m0v, unsigned v0, unsigned v1, unsigned v2, unsigned v3, i32x4 srd, unsigned so) {
  if constexpr (SC) DMA4_ASM(" sc0 sc1"); else DMA4_ASM("");
}
DI s16x4 trd(unsigned a) { return __builtin_amdgcn_ds_read_tr16_b64_v4i16((LAS s16x4*)(size_t)a); }
DI bf16x8 rda(unsigned a) { return *(const LAS bf16x8*)(size_t)a; }
DI unsigned ld_agent(const char* p) { return __hip_atomic_load((const unsigned*)p, __ATOMIC_RELAXED, __HIP_MEMORY_SCOPE_AGENT); }

__global__ __launch_bounds__(512, 2) void a2a_gemm_relu(const Params P) {
  const int tid = threadIdx.x, lane = tid & 63;
  const unsigned wid = rfl(tid >> 6);
  const unsigned b = blockIdx.x, xcd = b & 7, idx = b >> 3, tn = xcd * 16 + (idx >> 1), tm = idx & 1;
  const unsigned rank = (unsigned)P.rank & 7u;
  char* const myws = P.ws[rank];
  const unsigned colb = (tn >> 2) * 1024 + (tn & 3) * 128;

  if (wid >= 4) {
    unsigned pos = lane, tabA, tabB;
    if (tm && lane < SW) {
      if (lane & 1) { if (lane >= (lane < 8 ? 3u : 11u)) pos = lane - 3; }
      else if (lane + 3 < (lane < 8 ? 8u : (unsigned)SW)) pos = lane + 3;
    }
    { const unsigned u = pos; unsigned src, kt;
      if (u < 8) { src = rank; kt = u; tabA = kt * 128; }
      else { const unsigned j = u - 8; kt = (j * 37) >> 8; src = (rank + (j - kt * 7 + 1)) & 7u; tabA = src * BLK + kt * 65536u; }
      tabB = (src * 512u + kt * 64u) * 32768u; }
    if (wid < 6) {
      const unsigned h = wid - 4;
      unsigned callv;
      asm volatile("global_load_dword %0, %1, off sc1" : "=v"(callv) : "v"(myws + WS_CTR + b * 128) : "memory");
      const i32x4 srdL = make_srd(P.x + (size_t)rank * BLK, BLK);
      const unsigned arow = tm * 256 + h * 128 + (lane >> 3), ach = ((lane & 7) ^ (lane >> 3)) * 16;
      unsigned vL[4];
      #pragma unroll
      for (int i = 0; i < 4; ++i) vL[i] = (arow + 8 * i) * 1024 + ach - i * 1024;
      const unsigned vR = arow * 128 + ach, m0A = A_RING + h * 16384;
      auto issueL = [&](int t, unsigned slot) {
        const unsigned so = __builtin_amdgcn_readlane(tabA, t), m = m0A + slot * A_ST;
        dma4<1, 3>(m, vL[0], vL[1], vL[2], vL[3], srdL, so);
        #pragma unroll
        for (int j = 1; j < 4; ++j) dma4<1, 0>(m + j * 4096, vL[0], vL[1], vL[2], vL[3], srdL, so + j * 32768);
      };
      issueL(0, 0);
      issueL(1, 1);
      asm volatile("s_waitcnt vmcnt(16)" : "+v"(callv) :: "memory");
      BAR();
      issueL(2, 2);
      const unsigned call = rfl(callv) + 1;
      i32x4 srdR = make_srd(myws + WS_X + (call & 1) * XPAR, XPAR);
      const char* fl = myws + WS_FLAGS + (tm * 2 + h) * 256 + pos * 4;
      const char* tmo = myws + WS_TMO;
      #define SAMPLE() asm volatile("global_load_dword v200, %0, off sc0 sc1" :: "v"(fl) : "memory", "v200")
      unsigned long long ready = 0xffull;
      bool pend = true;
      unsigned slot = 0;
      for (int u = 0; u < 5; ++u) {
        asm volatile("s_waitcnt vmcnt(16)" ::: "memory"); BAR();
        if (u == 4) SAMPLE();
        issueL(u + 3, slot);
        slot = slot == 2 ? 0 : slot + 1;
      }
      for (int u = 5; u < 63; ++u) {
        const int t = u + 3;
        if (ready != ~0ull) {
          if (pend) {
            unsigned long long mk;
            asm volatile("s_waitcnt vmcnt(16)\n\ts_barrier\n\tv_cmp_ge_u32 %0, v200, %1" : "=s"(mk) : "v"(call * 8u) : "memory", "v200");
            ready |= mk;
          } else {
            asm volatile("s_waitcnt vmcnt(16)\n\ts_barrier" ::: "memory");
          }
          if (!((ready >> t) & 1)) {
            const unsigned long long t0 = __builtin_amdgcn_s_memrealtime();
            for (;;) {
              unsigned fv, tv;
              asm volatile("global_load_dword %0, %2, off sc0 sc1\n\tglobal_load_dword %1, %3, off sc1\n\ts_waitcnt vmcnt(0)"
                           : "=&v"(fv), "=&v"(tv) : "v"(fl), "v"(tmo) : "memory");
              ready |= __ballot(fv >= call * 8u);
              if ((ready >> t) & 1) break;
              if (rfl(tv) == call || __builtin_amdgcn_s_memrealtime() - t0 > DEADLINE_TICKS) {
                if (lane == 0) __hip_atomic_store((unsigned*)tmo, call, __ATOMIC_RELAXED, __HIP_MEMORY_SCOPE_AGENT);
                asm volatile("s_waitcnt vmcnt(0)" ::: "memory");
                ready = ~0ull; srdR[2] = 0;
                break;
              }
              __builtin_amdgcn_s_sleep(2);
            }
          }
          ready |= (2ull << t) - 1;
          pend = ((~ready >> t) & ((2ull << LOOK) - 1)) != 0;
          if (pend) SAMPLE();
        } else {
          asm volatile("s_waitcnt vmcnt(16)\n\ts_barrier" ::: "memory");
        }
        unsigned so = __builtin_amdgcn_readlane(tabA, t & 63);
        if (t >= 64) { srdR[2] = 0; so = 0; }
        const unsigned m = m0A + slot * A_ST;
        dma4<1, 3>(m, vR, vR, vR, vR, srdR, so);
        #pragma unroll
        for (int j = 1; j < 4; ++j) dma4<1, 0>(m + j * 4096, vR, vR, vR, vR, srdR, so + j * 4096);
        slot = slot == 2 ? 0 : slot + 1;
      }
      if (h == 0) STAMP(15, ((q0 - rt0) & 0xffff) | (((q1 - rt0) & 0xffff) << 16) | (((q2 - rt0) & 0xffff) << 32) | ((rt0 & 0xffff) << 48));
      STAMP(5 + 6 * h, spin); STAMP(6 + 6 * h, nspin); STAMP(7 + 6 * h, firstspin);
    } else if (wid == 6) {
      const unsigned tp0 = (unsigned)__builtin_amdgcn_s_memrealtime();
      i32x4 srdW = make_srd(P.w, (unsigned)KTOT * NTOT * 2u);
      const i32x4 srdWp = srdW, srdXL = make_srd(P.x + (size_t)rank * BLK, BLK);
      unsigned vB[4], dummy = 0;
      #pragma unroll
      for (int i = 0; i < 4; ++i) {
        const unsigned ch = (lane & 15) ^ (((lane >> 4) << 2) | (i >> 1));
        vB[i] = (4 * i + (lane >> 4)) * 32768 + colb + (ch >> 3) * 512 + (ch & 7) * 16 - i * 1024;
      }
      const unsigned vW = ((idx & 7) * 8 + (lane >> 3)) * 32768 + (tn >> 2) * 1024 + (lane & 7) * 128;
      const unsigned vXL = (tm * 256 + (idx >> 1) * 16 + (lane & 15)) * 1024 + (lane >> 4) * 128;
      auto issue = [&](int t, unsigned slot) {
        const unsigned so = __builtin_amdgcn_readlane(tabB, t & 63), m = B_RING + slot * B_ST;
        dma4<0, 3>(m, vB[0], vB[1], vB[2], vB[3], srdW, so);
        #pragma unroll
        for (int j = 1; j < 4; ++j) dma4<0, 0>(m + j * 4096, vB[0], vB[1], vB[2], vB[3], srdW, so + j * 16 * 32768);
      };
      auto pfw = [&](int t) {
        const unsigned sw = __builtin_amdgcn_readlane(tabB, t);
        asm volatile("s_nop 4\n\tbuffer_load_dword %0, %1, %2, %3 offen" : "+v"(dummy) : "v"(vW), "s"(srdWp), "s"(sw) : "memory");
      };
      issue(0, 0);
      issue(1, 1);
      asm volatile("s_waitcnt vmcnt(16)" ::: "memory");
      BAR();
      issue(2, 2); issue(3, 3);
      asm volatile("buffer_load_dword %0, %1, %2, 0 offen offset:512" : "+v"(dummy) : "v"(vXL), "s"(srdXL) : "memory");
      for (int t = 4; t < PFD; ++t) if (t >= SW) pfw(t);
      int npf = SW > PFD ? SW : PFD;
      unsigned tp = tp0 + PACE0 + PJX * xcd + PJW * ((idx * 13) & 31);
      for (int u = 0; u < 63; ++u) {
        asm volatile("s_waitcnt vmcnt(32)" ::: "memory");
        if (PACE && u <= PACEN) { while ((int)((unsigned)__builtin_amdgcn_s_memrealtime() - tp) < 0) __builtin_amdgcn_s_sleep(1); tp += PACE; }
        BAR();
        if (u >= 60) srdW[2] = 0;
        if (u + 4 < SW) __builtin_amdgcn_s_sleep(BSL);
        issue(u + 4, u & 3);
        if (u + PFD >= SW && npf < 64) {
          bool go = true;
          if (PLEN) go = ((unsigned)__builtin_amdgcn_s_memrealtime() % PPER) >= PLEN || npf <= u + 5;
          if (go && npf <= u + PFX) {
            pfw(npf++);
            if (u >= RM0 && npf <= u + PFX && npf < 64 && (u % RMP) == 0) pfw(npf++);
          }
        }
      }
      asm volatile("s_waitcnt vmcnt(0)" :: "v"(dummy) : "memory");
      STAMP(14, ((q0 - rt0) & 0xffff) | (((q1 - rt0) & 0xffff) << 16) | (((q2 - rt0) & 0xffff) << 32) | ((rt0 & 0xffff) << 48));
    } else {
      int nbar = 0;
      unsigned long long tb = __builtin_amdgcn_s_memrealtime() + (PT0 - PT1);
      auto bar = [&]() { BAR(); ++nbar; tb = __builtin_amdgcn_s_memrealtime(); };
      auto drain = [&]() {
        while ((__builtin_amdgcn_s_getreg(7 | (31 << 11)) & 0xC0000Fu) != 0 && nbar < 60) {
          if ((long long)(__builtin_amdgcn_s_memrealtime() - tb) > PT1) bar();
          else __builtin_amdgcn_s_sleep(1);
        }
        asm volatile("s_waitcnt vmcnt(0)" ::: "memory");
      };
      if (b < 224) {
        unsigned callv;
        u32x4 d[16];
        {
          const unsigned s = (b >> 5) + 1, row = (b & 31) * 16, dest = (rank - s) & 7u;
          const char* src = P.x + (size_t)dest * BLK + (size_t)(row + (lane >> 3)) * 1024 + (lane & 7) * 16;
          asm volatile("global_load_dword %0, %3, off sc1\n\tglobal_load_dwordx4 %1, %4, off\n\tglobal_load_dwordx4 %2, %5, off"
                       : "=&v"(callv), "=&v"(d[0]), "=&v"(d[1]) : "v"(myws + WS_CTR + b * 128), "v"(src), "v"(src + 8192) : "memory");
          char* const pws = P.ws[dest];
          drain();
          asm volatile("" : "+v"(callv), "+v"(d[0]), "+v"(d[1]));
          char* dst = pws + WS_X + ((rfl(callv) + 1) & 1) * XPAR + rank * BLK + row * 128 + lane * 16;
          asm volatile("global_store_dwordx4 %0, %1, off sc0 sc1\n\tglobal_store_dwordx4 %0, %2, off offset:1024 sc0 sc1\n\ts_nop 1" :: "v"(dst), "v"(d[0]), "v"(d[1]) : "memory");
          drain();
          if (lane == 0) __hip_atomic_fetch_add((unsigned*)(pws + WS_FLAGS + (row >> 7) * 256 + (8 + s - 1) * 4), 1u, __ATOMIC_RELAXED, __HIP_MEMORY_SCOPE_SYSTEM);
          STAMP(8, rp0); STAMP(9, RT()); STAMP(10, RT());
        }
        if (b < 196) {
          const unsigned call = rfl(callv) + 1;
          const unsigned kt = b / 28 + 1, r = b % 28, s = (r >> 2) + 1, qd = r & 3, dest = (rank - s) & 7u;
          const char* src = P.x + (size_t)dest * BLK + (size_t)(qd * 128 + (lane >> 3)) * 1024 + kt * 128 + (lane & 7) * 16;
          const int su = (int)(4 * kt + qd);
          while (nbar < su - 2) bar();
          #pragma unroll
          for (int i = 0; i < 16; i += 4)
            asm volatile("global_load_dwordx4 %0, %4, off\n\tglobal_load_dwordx4 %1, %5, off\n\tglobal_load_dwordx4 %2, %6, off\n\tglobal_load_dwordx4 %3, %7, off"
                         : "=&v"(d[i]), "=&v"(d[i + 1]), "=&v"(d[i + 2]), "=&v"(d[i + 3])
                         : "v"(src + i * 8192), "v"(src + (i + 1) * 8192), "v"(src + (i + 2) * 8192), "v"(src + (i + 3) * 8192) : "memory");
          char* const pws = P.ws[dest];
          char* dst = pws + WS_X + (call & 1) * XPAR + rank * BLK + kt * 65536 + qd * 16384 + lane * 16;
          drain();
          #pragma unroll
          for (int i = 0; i < 16; i += 4) asm volatile("" : "+v"(d[i]), "+v"(d[i + 1]), "+v"(d[i + 2]), "+v"(d[i + 3]));
          while (nbar < su) bar();
          #pragma unroll
          for (int i = 0; i < 16; ++i)
            asm volatile("global_store_dwordx4 %0, %1, off sc0 sc1\n\ts_nop 1" :: "v"(dst + i * 1024), "v"(d[i]) : "memory");
          tb -= PT1;
          drain();
          if (lane == 0)
            __hip_atomic_store((unsigned*)(pws + WS_FLAGS + qd * 256 + (8 + kt * 7 + s - 1) * 4), call * 8u, __ATOMIC_RELAXED, __HIP_MEMORY_SCOPE_SYSTEM);
        }
      }
    }
    return;
  }
  const unsigned wm = wid >> 1, wn = wid & 1, fr = lane & 15, fq = lane >> 4, q = (lane & 15) >> 2, p = lane & 3;
  const unsigned ra0 = A_RING + (wm * 128 + fr) * 128 + ((fq ^ (fr & 7)) * 16);
  const unsigned rb00 = B_RING + (8 * fq + q) * 256 + (((wn * 8 + p) ^ ((q << 2) | (fq & 1))) * 16) + (p & 1) * 8;
  const unsigned call = rfl(ld_agent(myws + WS_CTR + b * 128)) + 1;
  unsigned tmov = 0;
  int ulast = 61; asm volatile("" : "+s"(ulast));
  f32x4 acc[8][4];
  #pragma unroll
  for (int m = 0; m < 8; ++m)
    #pragma unroll
    for (int n = 0; n < 4; ++n) acc[m][n] = (f32x4){0.f, 0.f, 0.f, 0.f};
  bf16x8 af[8]; s16x4 b0l[4], b0h[4], b1l[4], b1h[4];
  #define MF(m, n_, L, H) MF2(m, (((m) & 1) ? 3 - (n_) : (n_)), L, H)
  #define MF2(m, n, L, H) do { acc[m][n] = __builtin_amdgcn_mfma_f32_16x16x32_bf16(__builtin_bit_cast(bf16x8, __builtin_shufflevector(L[n], H[n], 0, 1, 2, 3, 4, 5, 6, 7)), af[m], acc[m][n], 0, 0, 0); \
                              asm volatile("" : "+v"(acc[m][n])); SB(); } while (0)
  #define RA(dst, base, m) do { dst = rda((base) + (m) * 2048); SB(); } while (0)
  #define RL(L, n, ks) do { L[n] = trd(bb[n] + (ks) * 8192); SB(); } while (0)
  #define RH(H, n, ks) do { H[n] = trd(bb[n] + (ks) * 8192 + 1024); SB(); } while (0)
  #define PINV(x) do { asm volatile("" : "+v"(x)); SB(); } while (0)
  #define PINS(x) do { asm volatile("" : "+s"(x)); SB(); } while (0)
  unsigned bB[4], bb[4];
  #pragma unroll
  for (int n = 0; n < 4; ++n) { bB[n] = rb00 ^ ((n >> 1) * 64 + (n & 1) * 8); bb[n] = bB[n]; }
  unsigned a0 = ra0, a1 = ra0 ^ 64;
  BAR();
  #pragma unroll
  for (int n = 0; n < 4; ++n) { RL(b0l, n, 0); RH(b0h, n, 0); }
  #pragma unroll
  for (int m = 0; m < 7; ++m) RA(af[m], a0, m);
  unsigned sa = 0, sb = 0;
  for (int u = 0; u < 63; ++u) {
    MF(0, 0, b0l, b0h); RA(af[7], a0, 7); MF(0, 1, b0l, b0h); RL(b1l, 0, 1); MF(0, 2, b0l, b0h); RH(b1h, 0, 1); MF(0, 3, b0l, b0h);
    #pragma unroll
    for (int g = 1; g < 4; ++g) {
      MF(g, 0, b0l, b0h); RA(af[g - 1], a1, g - 1); MF(g, 1, b0l, b0h); RL(b1l, g, 1); MF(g, 2, b0l, b0h); RH(b1h, g, 1); MF(g, 3, b0l, b0h);
      if (g == 1) { sb = sb + B_ST; PINS(sb); }
      if (g == 2) { sb = sb & (4 * B_ST - 1); PINS(sb); }
    }
    #pragma unroll
    for (int g = 4; g < 8; ++g) {
      MF(g, 0, b0l, b0h); RA(af[g - 1], a1, g - 1); MF(g, 1, b0l, b0h); bb[g - 4] = bB[g - 4] + sb; PINV(bb[g - 4]); MF(g, 2, b0l, b0h);
      if (g == 4) { sa = sa == 2 * A_ST ? 0 : sa + A_ST; PINS(sa); }
      if (g == 5) { a0 = ra0 + sa; PINV(a0); }
      MF(g, 3, b0l, b0h);
    }
    MF(0, 0, b1l, b1h); RA(af[7], a1, 7); MF(0, 1, b1l, b1h); MF(0, 2, b1l, b1h); a1 = a0 ^ 64; PINV(a1); MF(0, 3, b1l, b1h);
    MF(1, 0, b1l, b1h); MF(1, 1, b1l, b1h);
    MF(1, 2, b1l, b1h); MF(1, 3, b1l, b1h);
    asm volatile("s_waitcnt lgkmcnt(0)\n\ts_barrier" ::: "memory"); SB();
    if (u == ulast) tmov = ld_agent(myws + WS_TMO);
    SB();
    MF(2, 0, b1l, b1h); RA(af[0], a0, 0); MF(2, 1, b1l, b1h); RL(b0l, 0, 0); MF(2, 2, b1l, b1h); RH(b0h, 0, 0); MF(2, 3, b1l, b1h); RA(af[1], a0, 1);
    #pragma unroll
    for (int g = 3; g < 8; ++g) {
      MF(g, 0, b1l, b1h); RA(af[g - 1], a0, g - 1); MF(g, 1, b1l, b1h); if (g < 6) RL(b0l, g - 2, 0); MF(g, 2, b1l, b1h); if (g < 6) RH(b0h, g - 2, 0); MF(g, 3, b1l, b1h);
    }
  }
  bf16x8 ag[8];
  #pragma unroll
  for (int n = 0; n < 4; ++n) { RL(b1l, n, 1); RH(b1h, n, 1); }
  RA(ag[0], a1, 0); RA(ag[1], a1, 1); RA(af[7], a0, 7);
  #pragma unroll
  for (int m = 2; m < 8; ++m) RA(ag[m], a1, m);
  const bool bad = rfl(tmov) == call;
  char* orow = P.out + (size_t)(tm * 256 + wm * 128 + (fr & 7)) * NTOT * 2 + colb + wn * 512 + (fr & 8) * 8 + fq * 16;
  const bool odd = fq & 1;
  auto epi = [&](const f32x4 (&ac)[4], int m) {
    u32x4 v[2], lo, hi;
    #pragma unroll
    for (int pr = 0; pr < 2; ++pr) {
      const f32x4 a = ac[2 * pr], c = ac[2 * pr + 1];
      const unsigned a0 = relu2(cvtpk(a[0], a[1])), a1 = relu2(cvtpk(a[2], a[3])), c0 = relu2(cvtpk(c[0], c[1])), c1 = relu2(cvtpk(c[2], c[3]));
      v[pr][0] = odd ? c0 : a0; v[pr][1] = odd ? c1 : a1; v[pr][2] = odd ? a0 : c0; v[pr][3] = odd ? a1 : c1;
    }
    #pragma unroll
    for (int e = 0; e < 4; ++e) {
      lo[e] = __builtin_amdgcn_update_dpp(v[0][e], v[1][e], 0x128, 0xf, 0xc, false);
      hi[e] = __builtin_amdgcn_update_dpp(v[1][e], v[0][e], 0x128, 0xf, 0x3, false);
    }
    char* pp = orow + (size_t)m * 16 * NTOT * 2;
    asm volatile("global_store_dwordx4 %0, %1, off sc1\n\ts_nop 1" :: "v"(pp), "v"(lo) : "memory");
    asm volatile("global_store_dwordx4 %0, %1, off sc1\n\ts_nop 1" :: "v"(pp + (size_t)8 * NTOT * 2), "v"(hi) : "memory");
  };
  if (bad) {
    const u32x4 qn = {0x7fc07fc0u, 0x7fc07fc0u, 0x7fc07fc0u, 0x7fc07fc0u};
    for (int i = 0; i < 16; ++i) *(u32x4*)(orow + (size_t)i * 8 * NTOT * 2) = qn;
  } else {
    SB();
    #pragma unroll
    for (int m = 0; m < 8; ++m) {
      #pragma unroll
      for (int n = 0; n < 4; ++n) {
        acc[m][n] = __builtin_amdgcn_mfma_f32_16x16x32_bf16(__builtin_bit_cast(bf16x8, __builtin_shufflevector(b0l[n], b0h[n], 0, 1, 2, 3, 4, 5, 6, 7)), af[m], acc[m][n], 0, 0, 0);
        asm volatile("" : "+v"(acc[m][n]));
      }
      #pragma unroll
      for (int n = 0; n < 4; ++n) {
        acc[m][n] = __builtin_amdgcn_mfma_f32_16x16x32_bf16(__builtin_bit_cast(bf16x8, __builtin_shufflevector(b1l[n], b1h[n], 0, 1, 2, 3, 4, 5, 6, 7)), ag[m], acc[m][n], 0, 0, 0);
        asm volatile("" : "+v"(acc[m][n]));
      }
      if (m > 0) {
        epi(acc[m - 1], m - 1);
        #pragma unroll
        for (int k = 0; k < 8; ++k) { __builtin_amdgcn_sched_group_barrier(0x8, 1, 0); __builtin_amdgcn_sched_group_barrier(0x2, 5, 0); }
      }
      SB();
    }
    epi(acc[7], 7);
  }
  if (tid == 0) __hip_atomic_store((unsigned*)(myws + WS_CTR + b * 128), call, __ATOMIC_RELAXED, __HIP_MEMORY_SCOPE_AGENT);
}

extern "C" void kernel_launch(int rank, int nranks, ncclComm_t comm, hipStream_t stream,
                              void* const* d_in, const int* in_sizes, int n_in,
                              void* d_out, int out_size, const int* in_dtypes, int out_dtype,
                              void* const* d_out_all, void* const* d_ws_all, size_t ws_bytes) {
  (void)comm; (void)in_sizes; (void)n_in; (void)out_size; (void)in_dtypes; (void)out_dtype; (void)d_out_all;
  if (nranks != NR || (unsigned)rank >= (unsigned)NR || ws_bytes < WS_NEED) { fprintf(stderr, "unexpected launch geometry\n"); return; }
  static bool init = false;
  if (!init) {
    hipError_t e = hipFuncSetAttribute((const void*)a2a_gemm_relu, hipFuncAttributeMaxDynamicSharedMemorySize, (int)LDS_BYTES);
    if (e != hipSuccess) { fprintf(stderr, "hipFuncSetAttribute: %s\n", hipGetErrorString(e)); return; }
    init = true;
  }
  Params p;
  p.x = (const char*)d_in[0]; p.w = (const char*)d_in[1]; p.out = (char*)d_out;
  for (int i = 0; i < NR; ++i) p.ws[i] = (char*)d_ws_all[i];
  p.rank = rank; p.pad = 0;
  hipLaunchKernelGGL(a2a_gemm_relu, dim3(256), dim3(512), LDS_BYTES, stream, p);
  hipError_t e = hipGetLastError();
  if (e != hipSuccess) fprintf(stderr, "launch: %s\n", hipGetErrorString(e));
}
```

```cpp
#include <hip/hip_runtime.h>
#include <rccl/rccl.h>
#include <cstdio>
#include <cstdint>

typedef __bf16 bf16x8 __attribute__((ext_vector_type(8)));
typedef __bf16 bf16x2 __attribute__((ext_vector_type(2)));
typedef short s16x4 __attribute__((ext_vector_type(4)));
typedef short s16x2 __attribute__((ext_vector_type(2)));
typedef float f32x4 __attribute__((ext_vector_type(4)));
typedef float f32x2 __attribute__((ext_vector_type(2)));
typedef unsigned u32x4 __attribute__((ext_vector_type(4)));
typedef int i32x4 __attribute__((ext_vector_type(4)));
#define LAS __attribute__((address_space(3)))
#define DI __device__ __forceinline__
#define SB() __builtin_amdgcn_sched_barrier(0)
#define BAR() asm volatile("s_barrier" ::: "memory")

#define PACE 86
#define PACEN 34
#define PACE0 330
#define BSL 9
#define PT0 120
#define PT1 30
#define LOOK 3
constexpr int NR = 8, KTOT = 4096, NTOT = 16384;
constexpr unsigned A_RING = 0, A_ST = 32768, B_RING = 3 * 32768, B_ST = 16384, LDS_BYTES = 163840;
constexpr unsigned WS_FLAGS = 0;
constexpr unsigned WS_TMO = 4096;
constexpr unsigned WS_CTR = 8192;
constexpr unsigned WS_X = 1u << 20;
constexpr unsigned BLK = 524288;
constexpr unsigned XPAR = 8 * BLK;
constexpr size_t WS_NEED = (size_t)WS_X + 2 * (size_t)XPAR;
constexpr unsigned long long DEADLINE_TICKS = 100000000ull;

#define STAMP(i, v) do {} while (0)
struct Params { const char* x; const char* w; char* out; char* ws[NR]; int rank; int pad; };

DI unsigned rfl(unsigned x) { return __builtin_amdgcn_readfirstlane(x); }
DI unsigned cvtpk(float lo, float hi) { f32x2 v = {lo, hi}; bf16x2 b = __builtin_convertvector(v, bf16x2); return __builtin_bit_cast(unsigned, b); }
DI unsigned relu2(unsigned x) { const s16x2 v = __builtin_bit_cast(s16x2, x), z = {0, 0}; return __builtin_bit_cast(unsigned, __builtin_elementwise_max(v, z)); }
DI i32x4 make_srd(const void* p, unsigned bytes) {
  unsigned long long a = (unsigned long long)p;
  i32x4 r; r[0] = rfl((unsigned)a); r[1] = rfl((unsigned)(a >> 32)) & 0xffff; r[2] = rfl(bytes); r[3] = 0x00020000; return r;
}
#define DMA1(v, off, pol) "buffer_load_dwordx4 " v ", %5, %6 offen offset:" off pol " lds\n\t"
#define DMA4_ASM(pol) asm volatile("s_mov_b32 m0, %0\n\ts_nop %7\n\t" DMA1("%1", "0", pol) DMA1("%2", "1024", pol) DMA1("%3", "2048", pol) DMA1("%4", "3072", pol) \
    :: "s"(m0v), "v"(v0), "v"(v1), "v"(v2), "v"(v3), "s"(srd), "s"(so), "i"(PAD) : "memory")
template <int SC, int PAD> DI void dma4(unsigned m0v, unsigned v0, unsigned v1, unsigned v2, unsigned v3, i32x4 srd, unsigned so) {
  if constexpr (SC) DMA4_ASM(" sc0 sc1"); else DMA4_ASM("");
}
DI s16x4 trd(unsigned a) { return __builtin_amdgcn_ds_read_tr16_b64_v4i16((LAS s16x4*)(size_t)a); }
DI bf16x8 rda(unsigned a) { return *(const LAS bf16x8*)(size_t)a; }
DI unsigned ld_agent(const char* p) { return __hip_atomic_load((const unsigned*)p, __ATOMIC_RELAXED, __HIP_MEMORY_SCOPE_AGENT); }

__global__ __launch_bounds__(512, 2) void a2a_gemm_relu(const Params P) {
  const int tid = threadIdx.x, lane = tid & 63;
  const unsigned wid = rfl(tid >> 6);
  const unsigned b = blockIdx.x, xcd = b & 7, idx = b >> 3, tn = xcd * 16 + (idx >> 1), tm = idx & 1;
  const unsigned rank = (unsigned)P.rank & 7u;
  char* const myws = P.ws[rank];
  const unsigned colb = (tn >> 2) * 1024 + (tn & 3) * 128;

  if (wid >= 4) {
    unsigned pos = lane, tabA, tabB;
    if (tm) {
      if (lane & 1) { if (lane >= (lane < 8 ? 3 : 11)) pos = lane - 3; }
      else if (lane + 3 < (lane < 8 ? 8 : 64)) pos = lane + 3;
    }
    { const unsigned u = pos; unsigned src, kt;
      if (u < 8) { src = rank; kt = u; tabA = kt * 128; }
      else { const unsigned j = u - 8; kt = (j * 37) >> 8; src = (rank + (j - kt * 7 + 1)) & 7u; tabA = src * BLK + kt * 65536u; }
      tabB = (src * 512u + kt * 64u) * 32768u; }
    if (wid < 6) {
      const unsigned h = wid - 4;
      unsigned callv;
      asm volatile("global_load_dword %0, %1, off sc1" : "=v"(callv) : "v"(myws + WS_CTR + b * 128) : "memory");
      const i32x4 srdL = make_srd(P.x + (size_t)rank * BLK, BLK);
      const unsigned arow = tm * 256 + h * 128 + (lane >> 3), ach = ((lane & 7) ^ (lane >> 3)) * 16;
      unsigned vL[4];
      #pragma unroll
      for (int i = 0; i < 4; ++i) vL[i] = (arow + 8 * i) * 1024 + ach - i * 1024;
      const unsigned vR = arow * 128 + ach, m0A = A_RING + h * 16384;
      auto issueL = [&](int t, unsigned slot) {
        const unsigned so = __builtin_amdgcn_readlane(tabA, t), m = m0A + slot * A_ST;
        dma4<1, 3>(m, vL[0], vL[1], vL[2], vL[3], srdL, so);
        #pragma unroll
        for (int j = 1; j < 4; ++j) dma4<1, 0>(m + j * 4096, vL[0], vL[1], vL[2], vL[3], srdL, so + j * 32768);
      };
      issueL(0, 0);
      issueL(1, 1);
      asm volatile("s_waitcnt vmcnt(16)" : "+v"(callv) :: "memory");
      BAR();
      issueL(2, 2);
      const unsigned call = rfl(callv) + 1;
      i32x4 srdR = make_srd(myws + WS_X + (call & 1) * XPAR, XPAR);
      const char* fl = myws + WS_FLAGS + (tm * 2 + h) * 256 + pos * 4;
      const char* tmo = myws + WS_TMO;
      #define SAMPLE() asm volatile("global_load_dword v200, %0, off sc0 sc1" :: "v"(fl) : "memory", "v200")
      unsigned long long ready = 0xffull;
      bool pend = true;
      unsigned slot = 0;
      for (int u = 0; u < 5; ++u) {
        asm volatile("s_waitcnt vmcnt(16)" ::: "memory"); BAR();
        if (u == 4) SAMPLE();
        issueL(u + 3, slot);
        slot = slot == 2 ? 0 : slot + 1;
      }
      for (int u = 5; u < 63; ++u) {
        const int t = u + 3;
        if (ready != ~0ull) {
          if (pend) {
            unsigned long long mk;
            asm volatile("s_waitcnt vmcnt(16)\n\ts_barrier\n\tv_cmp_ge_u32 %0, v200, %1" : "=s"(mk) : "v"(call * 8u) : "memory", "v200");
            ready |= mk;
          } else {
            asm volatile("s_waitcnt vmcnt(16)\n\ts_barrier" ::: "memory");
          }
          if (!((ready >> t) & 1)) {
            const unsigned long long t0 = __builtin_amdgcn_s_memrealtime();
            for (;;) {
              unsigned fv, tv;
              asm volatile("global_load_dword %0, %2, off sc0 sc1\n\tglobal_load_dword %1, %3, off sc1\n\ts_waitcnt vmcnt(0)"
                           : "=&v"(fv), "=&v"(tv) : "v"(fl), "v"(tmo) : "memory");
              ready |= __ballot(fv >= call * 8u);
              if ((ready >> t) & 1) break;
              if (rfl(tv) == call || __builtin_amdgcn_s_memrealtime() - t0 > DEADLINE_TICKS) {
                if (lane == 0) __hip_atomic_store((unsigned*)tmo, call, __ATOMIC_RELAXED, __HIP_MEMORY_SCOPE_AGENT);
                asm volatile("s_waitcnt vmcnt(0)" ::: "memory");
                ready = ~0ull; srdR[2] = 0;
                break;
              }
              __builtin_amdgcn_s_sleep(2);
            }
          }
          ready |= (2ull << t) - 1;
          pend = ((~ready >> t) & ((2ull << LOOK) - 1)) != 0;
          if (pend) SAMPLE();
        } else {
          asm volatile("s_waitcnt vmcnt(16)\n\ts_barrier" ::: "memory");
        }
        unsigned so = __builtin_amdgcn_readlane(tabA, t & 63);
        if (t >= 64) { srdR[2] = 0; so = 0; }
        const unsigned m = m0A + slot * A_ST;
        dma4<1, 3>(m, vR, vR, vR, vR, srdR, so);
        #pragma unroll
        for (int j = 1; j < 4; ++j) dma4<1, 0>(m + j * 4096, vR, vR, vR, vR, srdR, so + j * 4096);
        slot = slot == 2 ? 0 : slot + 1;
      }
      if (h == 0) STAMP(15, ((q0 - rt0) & 0xffff) | (((q1 - rt0) & 0xffff) << 16) | (((q2 - rt0) & 0xffff) << 32) | ((rt0 & 0xffff) << 48));
      STAMP(5 + 6 * h, spin); STAMP(6 + 6 * h, nspin); STAMP(7 + 6 * h, firstspin);
    } else if (wid == 6) {
      const unsigned tp0 = (unsigned)__builtin_amdgcn_s_memrealtime();
      i32x4 srdW = make_srd(P.w, (unsigned)KTOT * NTOT * 2u);
      const i32x4 srdXL = make_srd(P.x + (size_t)rank * BLK, BLK);
      unsigned vB[4], dummy = 0;
      #pragma unroll
      for (int i = 0; i < 4; ++i) {
        const unsigned ch = (lane & 15) ^ (((lane >> 4) << 2) | (i >> 1));
        vB[i] = (4 * i + (lane >> 4)) * 32768 + colb + (ch >> 3) * 512 + (ch & 7) * 16 - i * 1024;
      }
      const unsigned vXL = (tm * 256 + (idx >> 1) * 16 + (lane & 15)) * 1024 + (lane >> 4) * 128;
      auto issue = [&](int t, unsigned slot) {
        const unsigned so = __builtin_amdgcn_readlane(tabB, t & 63), m = B_RING + slot * B_ST;
        dma4<0, 3>(m, vB[0], vB[1], vB[2], vB[3], srdW, so);
        #pragma unroll
        for (int j = 1; j < 4; ++j) dma4<0, 0>(m + j * 4096, vB[0], vB[1], vB[2], vB[3], srdW, so + j * 16 * 32768);
      };
      issue(0, 0);
      issue(1, 1);
      asm volatile("s_waitcnt vmcnt(16)" ::: "memory");
      BAR();
      issue(2, 2); issue(3, 3);
      asm volatile("buffer_load_dword %0, %1, %2, 0 offen offset:512" : "+v"(dummy) : "v"(vXL), "s"(srdXL) : "memory");
      unsigned tp = tp0 + PACE0;
      for (int u = 0; u < 63; ++u) {
        asm volatile("s_waitcnt vmcnt(32)" ::: "memory");
        if (u <= PACEN) { while ((int)((unsigned)__builtin_amdgcn_s_memrealtime() - tp) < 0) __builtin_amdgcn_s_sleep(1); tp += PACE; }
        BAR();
        if (u >= 60) srdW[2] = 0;
        __builtin_amdgcn_s_sleep(BSL);
        issue(u + 4, u & 3);
      }
      asm volatile("s_waitcnt vmcnt(0)" :: "v"(dummy) : "memory");
      STAMP(14, ((q0 - rt0) & 0xffff) | (((q1 - rt0) & 0xffff) << 16) | (((q2 - rt0) & 0xffff) << 32) | ((rt0 & 0xffff) << 48));
    } else {
      int nbar = 0;
      unsigned long long tb = __builtin_amdgcn_s_memrealtime() + (PT0 - PT1);
      auto bar = [&]() { BAR(); ++nbar; tb = __builtin_amdgcn_s_memrealtime(); };
      auto drain = [&]() {
        while ((__builtin_amdgcn_s_getreg(7 | (31 << 11)) & 0xC0000Fu) != 0 && nbar < 60) {
          if ((long long)(__builtin_amdgcn_s_memrealtime() - tb) > PT1) bar();
          else __builtin_amdgcn_s_sleep(1);
        }
        asm volatile("s_waitcnt vmcnt(0)" ::: "memory");
      };
      if (b < 224) {
        unsigned callv;
        u32x4 d[16];
        {
          const unsigned s = (b >> 5) + 1, row = (b & 31) * 16, dest = (rank - s) & 7u;
          const char* src = P.x + (size_t)dest * BLK + (size_t)(row + (lane >> 3)) * 1024 + (lane & 7) * 16;
          asm volatile("global_load_dword %0, %3, off sc1\n\tglobal_load_dwordx4 %1, %4, off\n\tglobal_load_dwordx4 %2, %5, off"
                       : "=&v"(callv), "=&v"(d[0]), "=&v"(d[1]) : "v"(myws + WS_CTR + b * 128), "v"(src), "v"(src + 8192) : "memory");
          char* const pws = P.ws[dest];
          drain();
          asm volatile("" : "+v"(callv), "+v"(d[0]), "+v"(d[1]));
          char* dst = pws + WS_X + ((rfl(callv) + 1) & 1) * XPAR + rank * BLK + row * 128 + lane * 16;
          asm volatile("global_store_dwordx4 %0, %1, off sc0 sc1\n\tglobal_store_dwordx4 %0, %2, off offset:1024 sc0 sc1\n\ts_nop 1" :: "v"(dst), "v"(d[0]), "v"(d[1]) : "memory");
          drain();
          if (lane == 0) __hip_atomic_fetch_add((unsigned*)(pws + WS_FLAGS + (row >> 7) * 256 + (8 + s - 1) * 4), 1u, __ATOMIC_RELAXED, __HIP_MEMORY_SCOPE_SYSTEM);
          STAMP(8, rp0); STAMP(9, RT()); STAMP(10, RT());
        }
        if (b < 196) {
          const unsigned call = rfl(callv) + 1;
          const unsigned kt = b / 28 + 1, r = b % 28, s = (r >> 2) + 1, qd = r & 3, dest = (rank - s) & 7u;
          const char* src = P.x + (size_t)dest * BLK + (size_t)(qd * 128 + (lane >> 3)) * 1024 + kt * 128 + (lane & 7) * 16;
          const int su = (int)(4 * kt + qd);
          while (nbar < su - 2) bar();
          #pragma unroll
          for (int i = 0; i < 16; i += 4)
            asm volatile("global_load_dwordx4 %0, %4, off\n\tglobal_load_dwordx4 %1, %5, off\n\tglobal_load_dwordx4 %2, %6, off\n\tglobal_load_dwordx4 %3, %7, off"
                         : "=&v"(d[i]), "=&v"(d[i + 1]), "=&v"(d[i + 2]), "=&v"(d[i + 3])
                         : "v"(src + i * 8192), "v"(src + (i + 1) * 8192), "v"(src + (i + 2) * 8192), "v"(src + (i + 3) * 8192) : "memory");
          char* const pws = P.ws[dest];
          char* dst = pws + WS_X + (call & 1) * XPAR + rank * BLK + kt * 65536 + qd * 16384 + lane * 16;
          drain();
          #pragma unroll
          for (int i = 0; i < 16; i += 4) asm volatile("" : "+v"(d[i]), "+v"(d[i + 1]), "+v"(d[i + 2]), "+v"(d[i + 3]));
          while (nbar < su) bar();
          #pragma unroll
          for (int i = 0; i < 16; ++i)
            asm volatile("global_store_dwordx4 %0, %1, off sc0 sc1\n\ts_nop 1" :: "v"(dst + i * 1024), "v"(d[i]) : "memory");
          tb -= PT1;
          drain();
          if (lane == 0)
            __hip_atomic_store((unsigned*)(pws + WS_FLAGS + qd * 256 + (8 + kt * 7 + s - 1) * 4), call * 8u, __ATOMIC_RELAXED, __HIP_MEMORY_SCOPE_SYSTEM);
        }
      }
    }
    return;
  }
  const unsigned wm = wid >> 1, wn = wid & 1, fr = lane & 15, fq = lane >> 4, q = (lane & 15) >> 2, p = lane & 3;
  const unsigned ra0 = A_RING + (wm * 128 + fr) * 128 + ((fq ^ (fr & 7)) * 16);
  const unsigned rb00 = B_RING + (8 * fq + q) * 256 + (((wn * 8 + p) ^ ((q << 2) | (fq & 1))) * 16) + (p & 1) * 8;
  const unsigned call = rfl(ld_agent(myws + WS_CTR + b * 128)) + 1;
  unsigned tmov = 0;
  int ulast = 61; asm volatile("" : "+s"(ulast));
  f32x4 acc[8][4];
  #pragma unroll
  for (int m = 0; m < 8; ++m)
    #pragma unroll
    for (int n = 0; n < 4; ++n) acc[m][n] = (f32x4){0.f, 0.f, 0.f, 0.f};
  bf16x8 af[8]; s16x4 b0l[4], b0h[4], b1l[4], b1h[4];
  #define MF(m, n_, L, H) MF2(m, (((m) & 1) ? 3 - (n_) : (n_)), L, H)
  #define MF2(m, n, L, H) do { acc[m][n] = __builtin_amdgcn_mfma_f32_16x16x32_bf16(__builtin_bit_cast(bf16x8, __builtin_shufflevector(L[n], H[n], 0, 1, 2, 3, 4, 5, 6, 7)), af[m], acc[m][n], 0, 0, 0); \
                              asm volatile("" : "+v"(acc[m][n])); SB(); } while (0)
  #define RA(dst, base, m) do { dst = rda((base) + (m) * 2048); SB(); } while (0)
  #define RL(L, n, ks) do { L[n] = trd(bb[n] + (ks) * 8192); SB(); } while (0)
  #define RH(H, n, ks) do { H[n] = trd(bb[n] + (ks) * 8192 + 1024); SB(); } while (0)
  #define PINV(x) do { asm volatile("" : "+v"(x)); SB(); } while (0)
  #define PINS(x) do { asm volatile("" : "+s"(x)); SB(); } while (0)
  unsigned bB[4], bb[4];
  #pragma unroll
  for (int n = 0; n < 4; ++n) { bB[n] = rb00 ^ ((n >> 1) * 64 + (n & 1) * 8); bb[n] = bB[n]; }
  unsigned a0 = ra0, a1 = ra0 ^ 64;
  BAR();
  #pragma unroll
  for (int n = 0; n < 4; ++n) { RL(b0l, n, 0); RH(b0h, n, 0); }
  #pragma unroll
  for (int m = 0; m < 7; ++m) RA(af[m], a0, m);
  unsigned sa = 0, sb = 0;
  for (int u = 0; u < 63; ++u) {
    MF(0, 0, b0l, b0h); RA(af[7], a0, 7); MF(0, 1, b0l, b0h); RL(b1l, 0, 1); MF(0, 2, b0l, b0h); RH(b1h, 0, 1); MF(0, 3, b0l, b0h);
    #pragma unroll
    for (int g = 1; g < 4; ++g) {
      MF(g, 0, b0l, b0h); RA(af[g - 1], a1, g - 1); MF(g, 1, b0l, b0h); RL(b1l, g, 1); MF(g, 2, b0l, b0h); RH(b1h, g, 1); MF(g, 3, b0l, b0h);
      if (g == 1) { sb = sb + B_ST; PINS(sb); }
      if (g == 2) { sb = sb & (4 * B_ST - 1); PINS(sb); }
    }
    #pragma unroll
    for (int g = 4; g < 8; ++g) {
      MF(g, 0, b0l, b0h); RA(af[g - 1], a1, g - 1); MF(g, 1, b0l, b0h); bb[g - 4] = bB[g - 4] + sb; PINV(bb[g - 4]); MF(g, 2, b0l, b0h);
      if (g == 4) { sa = sa == 2 * A_ST ? 0 : sa + A_ST; PINS(sa); }
      if (g == 5) { a0 = ra0 + sa; PINV(a0); }
      MF(g, 3, b0l, b0h);
    }
    MF(0, 0, b1l, b1h); RA(af[7], a1, 7); MF(0, 1, b1l, b1h); MF(0, 2, b1l, b1h); a1 = a0 ^ 64; PINV(a1); MF(0, 3, b1l, b1h);
    MF(1, 0, b1l, b1h); MF(1, 1, b1l, b1h);
    MF(1, 2, b1l, b1h); MF(1, 3, b1l, b1h);
    asm volatile("s_waitcnt lgkmcnt(0)\n\ts_barrier" ::: "memory"); SB();
    if (u == ulast) tmov = ld_agent(myws + WS_TMO);
    SB();
    MF(2, 0, b1l, b1h); RA(af[0], a0, 0); MF(2, 1, b1l, b1h); RL(b0l, 0, 0); MF(2, 2, b1l, b1h); RH(b0h, 0, 0); MF(2, 3, b1l, b1h); RA(af[1], a0, 1);
    #pragma unroll
    for (int g = 3; g < 8; ++g) {
      MF(g, 0, b1l, b1h); RA(af[g - 1], a0, g - 1); MF(g, 1, b1l, b1h); if (g < 6) RL(b0l, g - 2, 0); MF(g, 2, b1l, b1h); if (g < 6) RH(b0h, g - 2, 0); MF(g, 3, b1l, b1h);
    }
  }
  bf16x8 ag[8];
  #pragma unroll
  for (int n = 0; n < 4; ++n) { RL(b1l, n, 1); RH(b1h, n, 1); }
  RA(ag[0], a1, 0); RA(ag[1], a1, 1); RA(af[7], a0, 7);
  #pragma unroll
  for (int m = 2; m < 8; ++m) RA(ag[m], a1, m);
  const bool bad = rfl(tmov) == call;
  char* orow = P.out + (size_t)(tm * 256 + wm * 128 + (fr & 7)) * NTOT * 2 + colb + wn * 512 + (fr & 8) * 8 + fq * 16;
  const bool odd = fq & 1;
  auto epi = [&](const f32x4 (&ac)[4], int m) {
    u32x4 v[2], lo, hi;
    #pragma unroll
    for (int pr = 0; pr < 2; ++pr) {
      const f32x4 a = ac[2 * pr], c = ac[2 * pr + 1];
      const unsigned a0 = relu2(cvtpk(a[0], a[1])), a1 = relu2(cvtpk(a[2], a[3])), c0 = relu2(cvtpk(c[0], c[1])), c1 = relu2(cvtpk(c[2], c[3]));
      v[pr][0] = odd ? c0 : a0; v[pr][1] = odd ? c1 : a1; v[pr][2] = odd ? a0 : c0; v[pr][3] = odd ? a1 : c1;
    }
    #pragma unroll
    for (int e = 0; e < 4; ++e) {
      lo[e] = __builtin_amdgcn_update_dpp(v[0][e], v[1][e], 0x128, 0xf, 0xc, false);
      hi[e] = __builtin_amdgcn_update_dpp(v[1][e], v[0][e], 0x128, 0xf, 0x3, false);
    }
    char* pp = orow + (size_t)m * 16 * NTOT * 2;
    asm volatile("global_store_dwordx4 %0, %1, off sc1\n\ts_nop 1" :: "v"(pp), "v"(lo) : "memory");
    asm volatile("global_store_dwordx4 %0, %1, off sc1\n\ts_nop 1" :: "v"(pp + (size_t)8 * NTOT * 2), "v"(hi) : "memory");
  };
  if (bad) {
    const u32x4 qn = {0x7fc07fc0u, 0x7fc07fc0u, 0x7fc07fc0u, 0x7fc07fc0u};
    for (int i = 0; i < 16; ++i) *(u32x4*)(orow + (size_t)i * 8 * NTOT * 2) = qn;
  } else {
    SB();
    #pragma unroll
    for (int m = 0; m < 8; ++m) {
      #pragma unroll
      for (int n = 0; n < 4; ++n) {
        acc[m][n] = __builtin_amdgcn_mfma_f32_16x16x32_bf16(__builtin_bit_cast(bf16x8, __builtin_shufflevector(b0l[n], b0h[n], 0, 1, 2, 3, 4, 5, 6, 7)), af[m], acc[m][n], 0, 0, 0);
        asm volatile("" : "+v"(acc[m][n]));
      }
      #pragma unroll
      for (int n = 0; n < 4; ++n) {
        acc[m][n] = __builtin_amdgcn_mfma_f32_16x16x32_bf16(__builtin_bit_cast(bf16x8, __builtin_shufflevector(b1l[n], b1h[n], 0, 1, 2, 3, 4, 5, 6, 7)), ag[m], acc[m][n], 0, 0, 0);
        asm volatile("" : "+v"(acc[m][n]));
      }
      if (m > 0) {
        epi(acc[m - 1], m - 1);
        #pragma unroll
        for (int k = 0; k < 8; ++k) { __builtin_amdgcn_sched_group_barrier(0x8, 1, 0); __builtin_amdgcn_sched_group_barrier(0x2, 5, 0); }
      }
      SB();
    }
    epi(acc[7], 7);
  }
  if (tid == 0) __hip_atomic_store((unsigned*)(myws + WS_CTR + b * 128), call, __ATOMIC_RELAXED, __HIP_MEMORY_SCOPE_AGENT);
}

extern "C" void kernel_launch(int rank, int nranks, ncclComm_t comm, hipStream_t stream,
                              void* const* d_in, const int* in_sizes, int n_in,
                              void* d_out, int out_size, const int* in_dtypes, int out_dtype,
                              void* const* d_out_all, void* const* d_ws_all, size_t ws_bytes) {
  (void)comm; (void)in_sizes; (void)n_in; (void)out_size; (void)in_dtypes; (void)out_dtype; (void)d_out_all;
  if (nranks != NR || (unsigned)rank >= (unsigned)NR || ws_bytes < WS_NEED) { fprintf(stderr, "unexpected launch geometry\n"); return; }
  static bool init = false;
  if (!init) {
    hipError_t e = hipFuncSetAttribute((const void*)a2a_gemm_relu, hipFuncAttributeMaxDynamicSharedMemorySize, (int)LDS_BYTES);
    if (e != hipSuccess) { fprintf(stderr, "hipFuncSetAttribute: %s\n", hipGetErrorString(e)); return; }
    init = true;
  }
  Params p;
  p.x = (const char*)d_in[0]; p.w = (const char*)d_in[1]; p.out = (char*)d_out;
  for (int i = 0; i < NR; ++i) p.ws[i] = (char*)d_ws_all[i];
  p.rank = rank; p.pad = 0;
  hipLaunchKernelGGL(a2a_gemm_relu, dim3(256), dim3(512), LDS_BYTES, stream, p);
  hipError_t e = hipGetLastError();
  if (e != hipSuccess) fprintf(stderr, "launch: %s\n", hipGetErrorString(e));
}
```

```cpp
#include <hip/hip_runtime.h>
#include <rccl/rccl.h>
#include <cstdio>
#include <cstdint>

typedef __bf16 bf16x8 __attribute__((ext_vector_type(8)));
typedef __bf16 bf16x2 __attribute__((ext_vector_type(2)));
typedef short s16x4 __attribute__((ext_vector_type(4)));
typedef short s16x2 __attribute__((ext_vector_type(2)));
typedef float f32x4 __attribute__((ext_vector_type(4)));
typedef float f32x2 __attribute__((ext_vector_type(2)));
typedef unsigned u32x4 __attribute__((ext_vector_type(4)));
typedef int i32x4 __attribute__((ext_vector_type(4)));
#define LAS __attribute__((address_space(3)))
#define DI __device__ __forceinline__
#define SB() __builtin_amdgcn_sched_barrier(0)
#define BAR() asm volatile("s_barrier" ::: "memory")

#define PACE 86
#define PACEN 34
#define PACE0 200
#define BLA 3
#define BSL 9
#define PT0 120
#define PT1 30
#define LOOK 3
constexpr int NR = 8, KTOT = 4096, NTOT = 16384;
constexpr unsigned A_RING = 0, A_ST = 32768, B_RING = 3 * 32768, B_ST = 16384, LDS_BYTES = 163840;
constexpr unsigned WS_FLAGS = 0;
constexpr unsigned WS_TMO = 4096;
constexpr unsigned WS_CTR = 8192;
constexpr unsigned WS_X = 1u << 20;
constexpr unsigned BLK = 524288;
constexpr unsigned XPAR = 8 * BLK;
constexpr size_t WS_NEED = (size_t)WS_X + 2 * (size_t)XPAR;
constexpr unsigned long long DEADLINE_TICKS = 100000000ull;

#define STAMP(i, v) do {} while (0)
struct Params { const char* x; const char* w; char* out; char* ws[NR]; int rank; int pad; };

DI unsigned rfl(unsigned x) { return __builtin_amdgcn_readfirstlane(x); }
DI unsigned cvtpk(float lo, float hi) { f32x2 v = {lo, hi}; bf16x2 b = __builtin_convertvector(v, bf16x2); return __builtin_bit_cast(unsigned, b); }
DI unsigned relu2(unsigned x) { const s16x2 v = __builtin_bit_cast(s16x2, x), z = {0, 0}; return __builtin_bit_cast(unsigned, __builtin_elementwise_max(v, z)); }
DI i32x4 make_srd(const void* p, unsigned bytes) {
  unsigned long long a = (unsigned long long)p;
  i32x4 r; r[0] = rfl((unsigned)a); r[1] = rfl((unsigned)(a >> 32)) & 0xffff; r[2] = rfl(bytes); r[3] = 0x00020000; return r;
}
#define DMA1(v, off, pol) "buffer_load_dwordx4 " v ", %5, %6 offen offset:" off pol " lds\n\t"
#define DMA4_ASM(pol) asm volatile("s_mov_b32 m0, %0\n\ts_nop %7\n\t" DMA1("%1", "0", pol) DMA1("%2", "1024", pol) DMA1("%3", "2048", pol) DMA1("%4", "3072", pol) \
    :: "s"(m0v), "v"(v0), "v"(v1), "v"(v2), "v"(v3), "s"(srd), "s"(so), "i"(PAD) : "memory")
template <int SC, int PAD> DI void dma4(unsigned m0v, unsigned v0, unsigned v1, unsigned v2, unsigned v3, i32x4 srd, unsigned so) {
  if constexpr (SC) DMA4_ASM(" sc0 sc1"); else DMA4_ASM("");
}
DI s16x4 trd(unsigned a) { return __builtin_amdgcn_ds_read_tr16_b64_v4i16((LAS s16x4*)(size_t)a); }
DI bf16x8 rda(unsigned a) { return *(const LAS bf16x8*)(size_t)a; }
DI unsigned ld_agent(const char* p) { return __hip_atomic_load((const unsigned*)p, __ATOMIC_RELAXED, __HIP_MEMORY_SCOPE_AGENT); }

__global__ __launch_bounds__(512, 2) void a2a_gemm_relu(const Params P) {
  const int tid = threadIdx.x, lane = tid & 63;
  const unsigned wid = rfl(tid >> 6);
  const unsigned b = blockIdx.x, xcd = b & 7, idx = b >> 3, tn = xcd * 16 + (idx >> 1), tm = idx & 1;
  const unsigned rank = (unsigned)P.rank & 7u;
  char* const myws = P.ws[rank];
  const unsigned colb = (tn >> 2) * 1024 + (tn & 3) * 128;

  if (wid >= 4) {
    unsigned pos = lane, tabA, tabB;
    if (tm) {
      if (lane & 1) { if (lane >= (lane < 8 ? 3 : 11)) pos = lane - 3; }
      else if (lane + 3 < (lane < 8 ? 8 : 64)) pos = lane + 3;
    }
    { const unsigned u = pos; unsigned src, kt;
      if (u < 8) { src = rank; kt = u; tabA = kt * 128; }
      else { const unsigned j = u - 8; kt = (j * 37) >> 8; src = (rank + (j - kt * 7 + 1)) & 7u; tabA = src * BLK + kt * 65536u; }
      tabB = (src * 512u + kt * 64u) * 32768u; }
    if (wid < 6) {
      const unsigned h = wid - 4;
      unsigned callv;
      asm volatile("global_load_dword %0, %1, off sc1" : "=v"(callv) : "v"(myws + WS_CTR + b * 128) : "memory");
      const i32x4 srdL = make_srd(P.x + (size_t)rank * BLK, BLK);
      const unsigned arow = tm * 256 + h * 128 + (lane >> 3), ach = ((lane & 7) ^ (lane >> 3)) * 16;
      unsigned vL[4];
      #pragma unroll
      for (int i = 0; i < 4; ++i) vL[i] = (arow + 8 * i) * 1024 + ach - i * 1024;
      const unsigned vR = arow * 128 + ach, m0A = A_RING + h * 16384;
      auto issueL = [&](int t, unsigned slot) {
        const unsigned so = __builtin_amdgcn_readlane(tabA, t), m = m0A + slot * A_ST;
        dma4<1, 3>(m, vL[0], vL[1], vL[2], vL[3], srdL, so);
        #pragma unroll
        for (int j = 1; j < 4; ++j) dma4<1, 0>(m + j * 4096, vL[0], vL[1], vL[2], vL[3], srdL, so + j * 32768);
      };
      issueL(0, 0);
      issueL(1, 1);
      asm volatile("s_waitcnt vmcnt(16)" : "+v"(callv) :: "memory");
      BAR();
      issueL(2, 2);
      const unsigned call = rfl(callv) + 1;
      i32x4 srdR = make_srd(myws + WS_X + (call & 1) * XPAR, XPAR);
      const char* fl = myws + WS_FLAGS + (tm * 2 + h) * 256 + pos * 4;
      const char* tmo = myws + WS_TMO;
      #define SAMPLE() asm volatile("global_load_dword v200, %0, off sc0 sc1" :: "v"(fl) : "memory", "v200")
      unsigned long long ready = 0xffull;
      bool pend = true;
      unsigned slot = 0;
      for (int u = 0; u < 5; ++u) {
        asm volatile("s_waitcnt vmcnt(16)" ::: "memory"); BAR();
        if (u == 4) SAMPLE();
        issueL(u + 3, slot);
        slot = slot == 2 ? 0 : slot + 1;
      }
      for (int u = 5; u < 63; ++u) {
        const int t = u + 3;
        if (ready != ~0ull) {
          if (pend) {
            unsigned long long mk;
            asm volatile("s_waitcnt vmcnt(16)\n\ts_barrier\n\tv_cmp_ge_u32 %0, v200, %1" : "=s"(mk) : "v"(call * 8u) : "memory", "v200");
            ready |= mk;
          } else {
            asm volatile("s_waitcnt vmcnt(16)\n\ts_barrier" ::: "memory");
          }
          if (!((ready >> t) & 1)) {
            const unsigned long long t0 = __builtin_amdgcn_s_memrealtime();
            for (;;) {
              unsigned fv, tv;
              asm volatile("global_load_dword %0, %2, off sc0 sc1\n\tglobal_load_dword %1, %3, off sc1\n\ts_waitcnt vmcnt(0)"
                           : "=&v"(fv), "=&v"(tv) : "v"(fl), "v"(tmo) : "memory");
              ready |= __ballot(fv >= call * 8u);
              if ((ready >> t) & 1) break;
              if (rfl(tv) == call || __builtin_amdgcn_s_memrealtime() - t0 > DEADLINE_TICKS) {
                if (lane == 0) __hip_atomic_store((unsigned*)tmo, call, __ATOMIC_RELAXED, __HIP_MEMORY_SCOPE_AGENT);
                asm volatile("s_waitcnt vmcnt(0)" ::: "memory");
                ready = ~0ull; srdR[2] = 0;
                break;
              }
              __builtin_amdgcn_s_sleep(2);
            }
          }
          ready |= (2ull << t) - 1;
          pend = ((~ready >> t) & ((2ull << LOOK) - 1)) != 0;
          if (pend) SAMPLE();
        } else {
          asm volatile("s_waitcnt vmcnt(16)\n\ts_barrier" ::: "memory");
        }
        unsigned so = __builtin_amdgcn_readlane(tabA, t & 63);
        if (t >= 64) { srdR[2] = 0; so = 0; }
        const unsigned m = m0A + slot * A_ST;
        dma4<1, 3>(m, vR, vR, vR, vR, srdR, so);
        #pragma unroll
        for (int j = 1; j < 4; ++j) dma4<1, 0>(m + j * 4096, vR, vR, vR, vR, srdR, so + j * 4096);
        slot = slot == 2 ? 0 : slot + 1;
      }
      if (h == 0) STAMP(15, ((q0 - rt0) & 0xffff) | (((q1 - rt0) & 0xffff) << 16) | (((q2 - rt0) & 0xffff) << 32) | ((rt0 & 0xffff) << 48));
      STAMP(5 + 6 * h, spin); STAMP(6 + 6 * h, nspin); STAMP(7 + 6 * h, firstspin);
    } else if (wid == 6) {
      const unsigned tp0 = (unsigned)__builtin_amdgcn_s_memrealtime();
      i32x4 srdW = make_srd(P.w, (unsigned)KTOT * NTOT * 2u);
      const i32x4 srdXL = make_srd(P.x + (size_t)rank * BLK, BLK);
      unsigned vB[4], dummy = 0;
      #pragma unroll
      for (int i = 0; i < 4; ++i) {
        const unsigned ch = (lane & 15) ^ (((lane >> 4) << 2) | (i >> 1));
        vB[i] = (4 * i + (lane >> 4)) * 32768 + colb + (ch >> 3) * 512 + (ch & 7) * 16 - i * 1024;
      }
      const unsigned vXL = (tm * 256 + (idx >> 1) * 16 + (lane & 15)) * 1024 + (lane >> 4) * 128;
      auto issue = [&](int t, unsigned slot) {
        const unsigned so = __builtin_amdgcn_readlane(tabB, t & 63), m = B_RING + slot * B_ST;
        dma4<0, 3>(m, vB[0], vB[1], vB[2], vB[3], srdW, so);
        #pragma unroll
        for (int j = 1; j < 4; ++j) dma4<0, 0>(m + j * 4096, vB[0], vB[1], vB[2], vB[3], srdW, so + j * 16 * 32768);
      };
      issue(0, 0);
      issue(1, 1);
      asm volatile("s_waitcnt vmcnt(16)" ::: "memory");
      BAR();
      issue(2, 2); if (BLA == 4) issue(3, 3);
      asm volatile("buffer_load_dword %0, %1, %2, 0 offen offset:512" : "+v"(dummy) : "v"(vXL), "s"(srdXL) : "memory");
      unsigned tp = tp0 + PACE0;
      for (int u = 0; u < 63; ++u) {
        if (BLA == 4) asm volatile("s_waitcnt vmcnt(32)" ::: "memory"); else asm volatile("s_waitcnt vmcnt(16)" ::: "memory");
        if (u <= PACEN) { while ((int)((unsigned)__builtin_amdgcn_s_memrealtime() - tp) < 0) __builtin_amdgcn_s_sleep(1); tp += PACE; }
        BAR();
        if (u + BLA >= 64) srdW[2] = 0;
        __builtin_amdgcn_s_sleep(BSL);
        issue(u + BLA, (u + BLA) & 3);
      }
      asm volatile("s_waitcnt vmcnt(0)" :: "v"(dummy) : "memory");
      STAMP(14, ((q0 - rt0) & 0xffff) | (((q1 - rt0) & 0xffff) << 16) | (((q2 - rt0) & 0xffff) << 32) | ((rt0 & 0xffff) << 48));
    } else {
      int nbar = 0;
      unsigned long long tb = __builtin_amdgcn_s_memrealtime() + (PT0 - PT1);
      auto bar = [&]() { BAR(); ++nbar; tb = __builtin_amdgcn_s_memrealtime(); };
      auto drain = [&]() {
        while ((__builtin_amdgcn_s_getreg(7 | (31 << 11)) & 0xC0000Fu) != 0 && nbar < 60) {
          if ((long long)(__builtin_amdgcn_s_memrealtime() - tb) > PT1) bar();
          else __builtin_amdgcn_s_sleep(1);
        }
        asm volatile("s_waitcnt vmcnt(0)" ::: "memory");
      };
      if (b < 224) {
        unsigned callv;
        u32x4 d[16];
        {
          const unsigned s = (b >> 5) + 1, row = (b & 31) * 16, dest = (rank - s) & 7u;
          const char* src = P.x + (size_t)dest * BLK + (size_t)(row + (lane >> 3)) * 1024 + (lane & 7) * 16;
          asm volatile("global_load_dword %0, %3, off sc1\n\tglobal_load_dwordx4 %1, %4, off\n\tglobal_load_dwordx4 %2, %5, off"
                       : "=&v"(callv), "=&v"(d[0]), "=&v"(d[1]) : "v"(myws + WS_CTR + b * 128), "v"(src), "v"(src + 8192) : "memory");
          char* const pws = P.ws[dest];
          drain();
          asm volatile("" : "+v"(callv), "+v"(d[0]), "+v"(d[1]));
          char* dst = pws + WS_X + ((rfl(callv) + 1) & 1) * XPAR + rank * BLK + row * 128 + lane * 16;
          asm volatile("global_store_dwordx4 %0, %1, off sc0 sc1\n\tglobal_store_dwordx4 %0, %2, off offset:1024 sc0 sc1\n\ts_nop 1" :: "v"(dst), "v"(d[0]), "v"(d[1]) : "memory");
          drain();
          if (lane == 0) __hip_atomic_fetch_add((unsigned*)(pws + WS_FLAGS + (row >> 7) * 256 + (8 + s - 1) * 4), 1u, __ATOMIC_RELAXED, __HIP_MEMORY_SCOPE_SYSTEM);
          STAMP(8, rp0); STAMP(9, RT()); STAMP(10, RT());
        }
        if (b < 196) {
          const unsigned call = rfl(callv) + 1;
          const unsigned kt = b / 28 + 1, r = b % 28, s = (r >> 2) + 1, qd = r & 3, dest = (rank - s) & 7u;
          const char* src = P.x + (size_t)dest * BLK + (size_t)(qd * 128 + (lane >> 3)) * 1024 + kt * 128 + (lane & 7) * 16;
          const int su = (int)(4 * kt + qd);
          while (nbar < su - 2) bar();
          #pragma unroll
          for (int i = 0; i < 16; i += 4)
            asm volatile("global_load_dwordx4 %0, %4, off\n\tglobal_load_dwordx4 %1, %5, off\n\tglobal_load_dwordx4 %2, %6, off\n\tglobal_load_dwordx4 %3, %7, off"
                         : "=&v"(d[i]), "=&v"(d[i + 1]), "=&v"(d[i + 2]), "=&v"(d[i + 3])
                         : "v"(src + i * 8192), "v"(src + (i + 1) * 8192), "v"(src + (i + 2) * 8192), "v"(src + (i + 3) * 8192) : "memory");
          char* const pws = P.ws[dest];
          char* dst = pws + WS_X + (call & 1) * XPAR + rank * BLK + kt * 65536 + qd * 16384 + lane * 16;
          drain();
          #pragma unroll
          for (int i = 0; i < 16; i += 4) asm volatile("" : "+v"(d[i]), "+v"(d[i + 1]), "+v"(d[i + 2]), "+v"(d[i + 3]));
          while (nbar < su) bar();
          #pragma unroll
          for (int i = 0; i < 16; ++i)
            asm volatile("global_store_dwordx4 %0, %1, off sc0 sc1\n\ts_nop 1" :: "v"(dst + i * 1024), "v"(d[i]) : "memory");
          tb -= PT1;
          drain();
          if (lane == 0)
            __hip_atomic_store((unsigned*)(pws + WS_FLAGS + qd * 256 + (8 + kt * 7 + s - 1) * 4), call * 8u, __ATOMIC_RELAXED, __HIP_MEMORY_SCOPE_SYSTEM);
        }
      }
    }
    return;
  }
  const unsigned wm = wid >> 1, wn = wid & 1, fr = lane & 15, fq = lane >> 4, q = (lane & 15) >> 2, p = lane & 3;
  const unsigned ra0 = A_RING + (wm * 128 + fr) * 128 + ((fq ^ (fr & 7)) * 16);
  const unsigned rb00 = B_RING + (8 * fq + q) * 256 + (((wn * 8 + p) ^ ((q << 2) | (fq & 1))) * 16) + (p & 1) * 8;
  const unsigned call = rfl(ld_agent(myws + WS_CTR + b * 128)) + 1;
  unsigned tmov = 0;
  int ulast = 61; asm volatile("" : "+s"(ulast));
  f32x4 acc[8][4];
  #pragma unroll
  for (int m = 0; m < 8; ++m)
    #pragma unroll
    for (int n = 0; n < 4; ++n) acc[m][n] = (f32x4){0.f, 0.f, 0.f, 0.f};
  bf16x8 af[8]; s16x4 b0l[4], b0h[4], b1l[4], b1h[4];
  #define MF(m, n_, L, H) MF2(m, (((m) & 1) ? 3 - (n_) : (n_)), L, H)
  #define MF2(m, n, L, H) do { acc[m][n] = __builtin_amdgcn_mfma_f32_16x16x32_bf16(__builtin_bit_cast(bf16x8, __builtin_shufflevector(L[n], H[n], 0, 1, 2, 3, 4, 5, 6, 7)), af[m], acc[m][n], 0, 0, 0); \
                              asm volatile("" : "+v"(acc[m][n])); SB(); } while (0)
  #define RA(dst, base, m) do { dst = rda((base) + (m) * 2048); SB(); } while (0)
  #define RL(L, n, ks) do { L[n] = trd(bb[n] + (ks) * 8192); SB(); } while (0)
  #define RH(H, n, ks) do { H[n] = trd(bb[n] + (ks) * 8192 + 1024); SB(); } while (0)
  #define PINV(x) do { asm volatile("" : "+v"(x)); SB(); } while (0)
  #define PINS(x) do { asm volatile("" : "+s"(x)); SB(); } while (0)
  unsigned bB[4], bb[4];
  #pragma unroll
  for (int n = 0; n < 4; ++n) { bB[n] = rb00 ^ ((n >> 1) * 64 + (n & 1) * 8); bb[n] = bB[n]; }
  unsigned a0 = ra0, a1 = ra0 ^ 64;
  BAR();
  #pragma unroll
  for (int n = 0; n < 4; ++n) { RL(b0l, n, 0); RH(b0h, n, 0); }
  #pragma unroll
  for (int m = 0; m < 7; ++m) RA(af[m], a0, m);
  unsigned sa = 0, sb = 0;
  for (int u = 0; u < 63; ++u) {
    MF(0, 0, b0l, b0h); RA(af[7], a0, 7); MF(0, 1, b0l, b0h); RL(b1l, 0, 1); MF(0, 2, b0l, b0h); RH(b1h, 0, 1); MF(0, 3, b0l, b0h);
    #pragma unroll
    for (int g = 1; g < 4; ++g) {
      MF(g, 0, b0l, b0h); RA(af[g - 1], a1, g - 1); MF(g, 1, b0l, b0h); RL(b1l, g, 1); MF(g, 2, b0l, b0h); RH(b1h, g, 1); MF(g, 3, b0l, b0h);
      if (g == 1) { sb = sb + B_ST; PINS(sb); }
      if (g == 2) { sb = sb & (4 * B_ST - 1); PINS(sb); }
    }
    #pragma unroll
    for (int g = 4; g < 8; ++g) {
      MF(g, 0, b0l, b0h); RA(af[g - 1], a1, g - 1); MF(g, 1, b0l, b0h); bb[g - 4] = bB[g - 4] + sb; PINV(bb[g - 4]); MF(g, 2, b0l, b0h);
      if (g == 4) { sa = sa == 2 * A_ST ? 0 : sa + A_ST; PINS(sa); }
      if (g == 5) { a0 = ra0 + sa; PINV(a0); }
      MF(g, 3, b0l, b0h);
    }
    MF(0, 0, b1l, b1h); RA(af[7], a1, 7); MF(0, 1, b1l, b1h); MF(0, 2, b1l, b1h); a1 = a0 ^ 64; PINV(a1); MF(0, 3, b1l, b1h);
    MF(1, 0, b1l, b1h); MF(1, 1, b1l, b1h);
    MF(1, 2, b1l, b1h); MF(1, 3, b1l, b1h);
    asm volatile("s_waitcnt lgkmcnt(0)\n\ts_barrier" ::: "memory"); SB();
    if (u == ulast) tmov = ld_agent(myws + WS_TMO);
    SB();
    MF(2, 0, b1l, b1h); RA(af[0], a0, 0); MF(2, 1, b1l, b1h); RL(b0l, 0, 0); MF(2, 2, b1l, b1h); RH(b0h, 0, 0); MF(2, 3, b1l, b1h); RA(af[1], a0, 1);
    #pragma unroll
    for (int g = 3; g < 8; ++g) {
      MF(g, 0, b1l, b1h); RA(af[g - 1], a0, g - 1); MF(g, 1, b1l, b1h); if (g < 6) RL(b0l, g - 2, 0); MF(g, 2, b1l, b1h); if (g < 6) RH(b0h, g - 2, 0); MF(g, 3, b1l, b1h);
    }
  }
  bf16x8 ag[8];
  #pragma unroll
  for (int n = 0; n < 4; ++n) { RL(b1l, n, 1); RH(b1h, n, 1); }
  RA(ag[0], a1, 0); RA(ag[1], a1, 1); RA(af[7], a0, 7);
  #pragma unroll
  for (int m = 2; m < 8; ++m) RA(ag[m], a1, m);
  const bool bad = rfl(tmov) == call;
  char* orow = P.out + (size_t)(tm * 256 + wm * 128 + (fr & 7)) * NTOT * 2 + colb + wn * 512 + (fr & 8) * 8 + fq * 16;
  const bool odd = fq & 1;
  auto epi = [&](const f32x4 (&ac)[4], int m) {
    u32x4 v[2], lo, hi;
    #pragma unroll
    for (int pr = 0; pr < 2; ++pr) {
      const f32x4 a = ac[2 * pr], c = ac[2 * pr + 1];
      const unsigned a0 = relu2(cvtpk(a[0], a[1])), a1 = relu2(cvtpk(a[2], a[3])), c0 = relu2(cvtpk(c[0], c[1])), c1 = relu2(cvtpk(c[2], c[3]));
      v[pr][0] = odd ? c0 : a0; v[pr][1] = odd ? c1 : a1; v[pr][2] = odd ? a0 : c0; v[pr][3] = odd ? a1 : c1;
    }
    #pragma unroll
    for (int e = 0; e < 4; ++e) {
      lo[e] = __builtin_amdgcn_update_dpp(v[0][e], v[1][e], 0x128, 0xf, 0xc, false);
      hi[e] = __builtin_amdgcn_update_dpp(v[1][e], v[0][e], 0x128, 0xf, 0x3, false);
    }
    char* pp = orow + (size_t)m * 16 * NTOT * 2;
    asm volatile("global_store_dwordx4 %0, %1, off sc1\n\ts_nop 1" :: "v"(pp), "v"(lo) : "memory");
    asm volatile("global_store_dwordx4 %0, %1, off sc1\n\ts_nop 1" :: "v"(pp + (size_t)8 * NTOT * 2), "v"(hi) : "memory");
  };
  if (bad) {
    const u32x4 qn = {0x7fc07fc0u, 0x7fc07fc0u, 0x7fc07fc0u, 0x7fc07fc0u};
    for (int i = 0; i < 16; ++i) *(u32x4*)(orow + (size_t)i * 8 * NTOT * 2) = qn;
  } else {
    SB();
    #pragma unroll
    for (int m = 0; m < 8; ++m) {
      #pragma unroll
      for (int n = 0; n < 4; ++n) {
        acc[m][n] = __builtin_amdgcn_mfma_f32_16x16x32_bf16(__builtin_bit_cast(bf16x8, __builtin_shufflevector(b0l[n], b0h[n], 0, 1, 2, 3, 4, 5, 6, 7)), af[m], acc[m][n], 0, 0, 0);
        asm volatile("" : "+v"(acc[m][n]));
      }
      #pragma unroll
      for (int n = 0; n < 4; ++n) {
        acc[m][n] = __builtin_amdgcn_mfma_f32_16x16x32_bf16(__builtin_bit_cast(bf16x8, __builtin_shufflevector(b1l[n], b1h[n], 0, 1, 2, 3, 4, 5, 6, 7)), ag[m], acc[m][n], 0, 0, 0);
        asm volatile("" : "+v"(acc[m][n]));
      }
      if (m > 0) {
        epi(acc[m - 1], m - 1);
        #pragma unroll
        for (int k = 0; k < 8; ++k) { __builtin_amdgcn_sched_group_barrier(0x8, 1, 0); __builtin_amdgcn_sched_group_barrier(0x2, 5, 0); }
      }
      SB();
    }
    epi(acc[7], 7);
  }
  if (tid == 0) __hip_atomic_store((unsigned*)(myws + WS_CTR + b * 128), call, __ATOMIC_RELAXED, __HIP_MEMORY_SCOPE_AGENT);
}

extern "C" void kernel_launch(int rank, int nranks, ncclComm_t comm, hipStream_t stream,
                              void* const* d_in, const int* in_sizes, int n_in,
                              void* d_out, int out_size, const int* in_dtypes, int out_dtype,
                              void* const* d_out_all, void* const* d_ws_all, size_t ws_bytes) {
  (void)comm; (void)in_sizes; (void)n_in; (void)out_size; (void)in_dtypes; (void)out_dtype; (void)d_out_all;
  if (nranks != NR || (unsigned)rank >= (unsigned)NR || ws_bytes < WS_NEED) { fprintf(stderr, "unexpected launch geometry\n"); return; }
  static bool init = false;
  if (!init) {
    hipError_t e = hipFuncSetAttribute((const void*)a2a_gemm_relu, hipFuncAttributeMaxDynamicSharedMemorySize, (int)LDS_BYTES);
    if (e != hipSuccess) { fprintf(stderr, "hipFuncSetAttribute: %s\n", hipGetErrorString(e)); return; }
    init = true;
  }
  Params p;
  p.x = (const char*)d_in[0]; p.w = (const char*)d_in[1]; p.out = (char*)d_out;
  for (int i = 0; i < NR; ++i) p.ws[i] = (char*)d_ws_all[i];
  p.rank = rank; p.pad = 0;
  hipLaunchKernelGGL(a2a_gemm_relu, dim3(256), dim3(512), LDS_BYTES, stream, p);
  hipError_t e = hipGetLastError();
  if (e != hipSuccess) fprintf(stderr, "launch: %s\n", hipGetErrorString(e));
}
```

```cpp
#include <hip/hip_runtime.h>
#include <rccl/rccl.h>
#include <cstdio>
#include <cstdint>

typedef __bf16 bf16x8 __attribute__((ext_vector_type(8)));
typedef __bf16 bf16x2 __attribute__((ext_vector_type(2)));
typedef short s16x4 __attribute__((ext_vector_type(4)));
typedef short s16x2 __attribute__((ext_vector_type(2)));
typedef float f32x4 __attribute__((ext_vector_type(4)));
typedef float f32x2 __attribute__((ext_vector_type(2)));
typedef unsigned u32x4 __attribute__((ext_vector_type(4)));
typedef int i32x4 __attribute__((ext_vector_type(4)));
#define LAS __attribute__((address_space(3)))
#define DI __device__ __forceinline__
#define SB() __builtin_amdgcn_sched_barrier(0)
#define BAR() asm volatile("s_barrier" ::: "memory")

#define PACE 86
#define PACEN 34
#define PACE0 200
#define BSL 9
#define PT0 120
#define PT1 30
#define LOOK 3
constexpr int NR = 8, KTOT = 4096, NTOT = 16384;
constexpr unsigned A_RING = 0, A_ST = 32768, B_RING = 3 * 32768, B_ST = 16384, LDS_BYTES = 3 * (A_ST + B_ST);
constexpr unsigned WS_FLAGS = 0;
constexpr unsigned WS_TMO = 4096;
constexpr unsigned WS_CTR = 8192;
constexpr unsigned WS_X = 1u << 20;
constexpr unsigned BLK = 524288;
constexpr unsigned XPAR = 8 * BLK;
constexpr size_t WS_NEED = (size_t)WS_X + 2 * (size_t)XPAR;
constexpr unsigned long long DEADLINE_TICKS = 100000000ull;

#define STAMP(i, v) do {} while (0)
struct Params { const char* x; const char* w; char* out; char* ws[NR]; int rank; int pad; };

DI unsigned rfl(unsigned x) { return __builtin_amdgcn_readfirstlane(x); }
DI unsigned cvtpk(float lo, float hi) { f32x2 v = {lo, hi}; bf16x2 b = __builtin_convertvector(v, bf16x2); return __builtin_bit_cast(unsigned, b); }
DI unsigned relu2(unsigned x) { const s16x2 v = __builtin_bit_cast(s16x2, x), z = {0, 0}; return __builtin_bit_cast(unsigned, __builtin_elementwise_max(v, z)); }
DI i32x4 make_srd(const void* p, unsigned bytes) {
  unsigned long long a = (unsigned long long)p;
  i32x4 r; r[0] = rfl((unsigned)a); r[1] = rfl((unsigned)(a >> 32)) & 0xffff; r[2] = rfl(bytes); r[3] = 0x00020000; return r;
}
#define DMA1(v, off, pol) "buffer_load_dwordx4 " v ", %5, %6 offen offset:" off pol " lds\n\t"
#define DMA4_ASM(pol) asm volatile("s_mov_b32 m0, %0\n\ts_nop %7\n\t" DMA1("%1", "0", pol) DMA1("%2", "1024", pol) DMA1("%3", "2048", pol) DMA1("%4", "3072", pol) \
    :: "s"(m0v), "v"(v0), "v"(v1), "v"(v2), "v"(v3), "s"(srd), "s"(so), "i"(PAD) : "memory")
template <int SC, int PAD> DI void dma4(unsigned m0v, unsigned v0, unsigned v1, unsigned v2, unsigned v3, i32x4 srd, unsigned so) {
  if constexpr (SC) DMA4_ASM(" sc0 sc1"); else DMA4_ASM("");
}
DI s16x4 trd(unsigned a) { return __builtin_amdgcn_ds_read_tr16_b64_v4i16((LAS s16x4*)(size_t)a); }
DI bf16x8 rda(unsigned a) { return *(const LAS bf16x8*)(size_t)a; }
DI unsigned ld_agent(const char* p) { return __hip_atomic_load((const unsigned*)p, __ATOMIC_RELAXED, __HIP_MEMORY_SCOPE_AGENT); }

__global__ __launch_bounds__(512, 2) void a2a_gemm_relu(const Params P) {
  const int tid = threadIdx.x, lane = tid & 63;
  const unsigned wid = rfl(tid >> 6);
  const unsigned b = blockIdx.x, xcd = b & 7, idx = b >> 3, tn = xcd * 16 + (idx >> 1), tm = idx & 1;
  const unsigned rank = (unsigned)P.rank & 7u;
  char* const myws = P.ws[rank];
  const unsigned colb = (tn >> 2) * 1024 + (tn & 3) * 128;

  if (wid >= 4) {
    unsigned pos = lane, tabA, tabB;
    if (tm) {
      if (lane & 1) { if (lane >= (lane < 8 ? 3 : 11)) pos = lane - 3; }
      else if (lane + 3 < (lane < 8 ? 8 : 64)) pos = lane + 3;
    }
    { const unsigned u = pos; unsigned src, kt;
      if (u < 8) { src = rank; kt = u; tabA = kt * 128; }
      else { const unsigned j = u - 8; kt = (j * 37) >> 8; src = (rank + (j - kt * 7 + 1)) & 7u; tabA = src * BLK + kt * 65536u; }
      tabB = (src * 512u + kt * 64u) * 32768u; }
    if (wid < 6) {
      const unsigned h = wid - 4;
      unsigned callv;
      asm volatile("global_load_dword %0, %1, off sc1" : "=v"(callv) : "v"(myws + WS_CTR + b * 128) : "memory");
      const i32x4 srdL = make_srd(P.x + (size_t)rank * BLK, BLK);
      const unsigned arow = tm * 256 + h * 128 + (lane >> 3), ach = ((lane & 7) ^ (lane >> 3)) * 16;
      unsigned vL[4];
      #pragma unroll
      for (int i = 0; i < 4; ++i) vL[i] = (arow + 8 * i) * 1024 + ach - i * 1024;
      const unsigned vR = arow * 128 + ach, m0A = A_RING + h * 16384;
      auto issueL = [&](int t, unsigned slot) {
        const unsigned so = __builtin_amdgcn_readlane(tabA, t), m = m0A + slot * A_ST;
        dma4<1, 3>(m, vL[0], vL[1], vL[2], vL[3], srdL, so);
        #pragma unroll
        for (int j = 1; j < 4; ++j) dma4<1, 0>(m + j * 4096, vL[0], vL[1], vL[2], vL[3], srdL, so + j * 32768);
      };
      issueL(0, 0);
      issueL(1, 1);
      asm volatile("s_waitcnt vmcnt(16)" : "+v"(callv) :: "memory");
      BAR();
      issueL(2, 2);
      const unsigned call = rfl(callv) + 1;
      i32x4 srdR = make_srd(myws + WS_X + (call & 1) * XPAR, XPAR);
      const char* fl = myws + WS_FLAGS + (tm * 2 + h) * 256 + pos * 4;
      const char* tmo = myws + WS_TMO;
      #define SAMPLE() asm volatile("global_load_dword v200, %0, off sc0 sc1" :: "v"(fl) : "memory", "v200")
      unsigned long long ready = 0xffull;
      bool pend = true;
      unsigned slot = 0;
      for (int u = 0; u < 5; ++u) {
        asm volatile("s_waitcnt vmcnt(16)" ::: "memory"); BAR();
        if (u == 4) SAMPLE();
        issueL(u + 3, slot);
        slot = slot == 2 ? 0 : slot + 1;
      }
      for (int u = 5; u < 63; ++u) {
        const int t = u + 3;
        if (ready != ~0ull) {
          if (pend) {
            unsigned long long mk;
            asm volatile("s_waitcnt vmcnt(16)\n\ts_barrier\n\tv_cmp_ge_u32 %0, v200, %1" : "=s"(mk) : "v"(call * 8u) : "memory", "v200");
            ready |= mk;
          } else {
            asm volatile("s_waitcnt vmcnt(16)\n\ts_barrier" ::: "memory");
          }
          if (!((ready >> t) & 1)) {
            const unsigned long long t0 = __builtin_amdgcn_s_memrealtime();
            for (;;) {
              unsigned fv, tv;
              asm volatile("global_load_dword %0, %2, off sc0 sc1\n\tglobal_load_dword %1, %3, off sc1\n\ts_waitcnt vmcnt(0)"
                           : "=&v"(fv), "=&v"(tv) : "v"(fl), "v"(tmo) : "memory");
              ready |= __ballot(fv >= call * 8u);
              if ((ready >> t) & 1) break;
              if (rfl(tv) == call || __builtin_amdgcn_s_memrealtime() - t0 > DEADLINE_TICKS) {
                if (lane == 0) __hip_atomic_store((unsigned*)tmo, call, __ATOMIC_RELAXED, __HIP_MEMORY_SCOPE_AGENT);
                asm volatile("s_waitcnt vmcnt(0)" ::: "memory");
                ready = ~0ull; srdR[2] = 0;
                break;
              }
              __builtin_amdgcn_s_sleep(2);
            }
          }
          ready |= (2ull << t) - 1;
          pend = ((~ready >> t) & ((2ull << LOOK) - 1)) != 0;
          if (pend) SAMPLE();
        } else {
          asm volatile("s_waitcnt vmcnt(16)\n\ts_barrier" ::: "memory");
        }
        unsigned so = __builtin_amdgcn_readlane(tabA, t & 63);
        if (t >= 64) { srdR[2] = 0; so = 0; }
        const unsigned m = m0A + slot * A_ST;
        dma4<1, 3>(m, vR, vR, vR, vR, srdR, so);
        #pragma unroll
        for (int j = 1; j < 4; ++j) dma4<1, 0>(m + j * 4096, vR, vR, vR, vR, srdR, so + j * 4096);
        slot = slot == 2 ? 0 : slot + 1;
      }
      if (h == 0) STAMP(15, ((q0 - rt0) & 0xffff) | (((q1 - rt0) & 0xffff) << 16) | (((q2 - rt0) & 0xffff) << 32) | ((rt0 & 0xffff) << 48));
      STAMP(5 + 6 * h, spin); STAMP(6 + 6 * h, nspin); STAMP(7 + 6 * h, firstspin);
    } else if (wid == 6) {
      const unsigned tp0 = (unsigned)__builtin_amdgcn_s_memrealtime();
      i32x4 srdW = make_srd(P.w, (unsigned)KTOT * NTOT * 2u);
      const i32x4 srdXL = make_srd(P.x + (size_t)rank * BLK, BLK);
      unsigned vB[4], dummy = 0;
      #pragma unroll
      for (int i = 0; i < 4; ++i) {
        const unsigned ch = (lane & 15) ^ (((lane >> 4) << 2) | (i >> 1));
        vB[i] = (4 * i + (lane >> 4)) * 32768 + colb + (ch >> 3) * 512 + (ch & 7) * 16 - i * 1024;
      }
      const unsigned vXL = (tm * 256 + (idx >> 1) * 16 + (lane & 15)) * 1024 + (lane >> 4) * 128;
      auto issue = [&](int t, unsigned slot) {
        const unsigned so = __builtin_amdgcn_readlane(tabB, t & 63), m = B_RING + slot * B_ST;
        dma4<0, 3>(m, vB[0], vB[1], vB[2], vB[3], srdW, so);
        #pragma unroll
        for (int j = 1; j < 4; ++j) dma4<0, 0>(m + j * 4096, vB[0], vB[1], vB[2], vB[3], srdW, so + j * 16 * 32768);
      };
      issue(0, 0);
      issue(1, 1);
      asm volatile("s_waitcnt vmcnt(16)" ::: "memory");
      BAR();
      issue(2, 2);
      asm volatile("buffer_load_dword %0, %1, %2, 0 offen offset:512" : "+v"(dummy) : "v"(vXL), "s"(srdXL) : "memory");
      unsigned tp = tp0 + PACE0, slot = 0;
      for (int u = 0; u < 63; ++u) {
        asm volatile("s_waitcnt vmcnt(16)" ::: "memory");
        if (u <= PACEN) { while ((int)((unsigned)__builtin_amdgcn_s_memrealtime() - tp) < 0) __builtin_amdgcn_s_sleep(1); tp += PACE; }
        BAR();
        if (u >= 61) srdW[2] = 0;
        __builtin_amdgcn_s_sleep(BSL);
        issue(u + 3, slot);
        slot = slot == 2 ? 0 : slot + 1;
      }
      asm volatile("s_waitcnt vmcnt(0)" :: "v"(dummy) : "memory");
      STAMP(14, ((q0 - rt0) & 0xffff) | (((q1 - rt0) & 0xffff) << 16) | (((q2 - rt0) & 0xffff) << 32) | ((rt0 & 0xffff) << 48));
    } else {
      int nbar = 0;
      unsigned long long tb = __builtin_amdgcn_s_memrealtime() + (PT0 - PT1);
      auto bar = [&]() { BAR(); ++nbar; tb = __builtin_amdgcn_s_memrealtime(); };
      auto drain = [&]() {
        while ((__builtin_amdgcn_s_getreg(7 | (31 << 11)) & 0xC0000Fu) != 0 && nbar < 60) {
          if ((long long)(__builtin_amdgcn_s_memrealtime() - tb) > PT1) bar();
          else __builtin_amdgcn_s_sleep(1);
        }
        asm volatile("s_waitcnt vmcnt(0)" ::: "memory");
      };
      if (b < 224) {
        unsigned callv;
        u32x4 d[16];
        {
          const unsigned s = (b >> 5) + 1, row = (b & 31) * 16, dest = (rank - s) & 7u;
          const char* src = P.x + (size_t)dest * BLK + (size_t)(row + (lane >> 3)) * 1024 + (lane & 7) * 16;
          asm volatile("global_load_dword %0, %3, off sc1\n\tglobal_load_dwordx4 %1, %4, off\n\tglobal_load_dwordx4 %2, %5, off"
                       : "=&v"(callv), "=&v"(d[0]), "=&v"(d[1]) : "v"(myws + WS_CTR + b * 128), "v"(src), "v"(src + 8192) : "memory");
          char* const pws = P.ws[dest];
          drain();
          asm volatile("" : "+v"(callv), "+v"(d[0]), "+v"(d[1]));
          char* dst = pws + WS_X + ((rfl(callv) + 1) & 1) * XPAR + rank * BLK + row * 128 + lane * 16;
          asm volatile("global_store_dwordx4 %0, %1, off sc0 sc1\n\tglobal_store_dwordx4 %0, %2, off offset:1024 sc0 sc1\n\ts_nop 1" :: "v"(dst), "v"(d[0]), "v"(d[1]) : "memory");
          drain();
          if (lane == 0) __hip_atomic_fetch_add((unsigned*)(pws + WS_FLAGS + (row >> 7) * 256 + (8 + s - 1) * 4), 1u, __ATOMIC_RELAXED, __HIP_MEMORY_SCOPE_SYSTEM);
          STAMP(8, rp0); STAMP(9, RT()); STAMP(10, RT());
        }
        if (b < 196) {
          const unsigned call = rfl(callv) + 1;
          const unsigned kt = b / 28 + 1, r = b % 28, s = (r >> 2) + 1, qd = r & 3, dest = (rank - s) & 7u;
          const char* src = P.x + (size_t)dest * BLK + (size_t)(qd * 128 + (lane >> 3)) * 1024 + kt * 128 + (lane & 7) * 16;
          const int su = (int)(4 * kt + qd);
          while (nbar < su - 2) bar();
          #pragma unroll
          for (int i = 0; i < 16; i += 4)
            asm volatile("global_load_dwordx4 %0, %4, off\n\tglobal_load_dwordx4 %1, %5, off\n\tglobal_load_dwordx4 %2, %6, off\n\tglobal_load_dwordx4 %3, %7, off"
                         : "=&v"(d[i]), "=&v"(d[i + 1]), "=&v"(d[i + 2]), "=&v"(d[i + 3])
                         : "v"(src + i * 8192), "v"(src + (i + 1) * 8192), "v"(src + (i + 2) * 8192), "v"(src + (i + 3) * 8192) : "memory");
          char* const pws = P.ws[dest];
          char* dst = pws + WS_X + (call & 1) * XPAR + rank * BLK + kt * 65536 + qd * 16384 + lane * 16;
          drain();
          #pragma unroll
          for (int i = 0; i < 16; i += 4) asm volatile("" : "+v"(d[i]), "+v"(d[i + 1]), "+v"(d[i + 2]), "+v"(d[i + 3]));
          while (nbar < su) bar();
          #pragma unroll
          for (int i = 0; i < 16; ++i)
            asm volatile("global_store_dwordx4 %0, %1, off sc0 sc1\n\ts_nop 1" :: "v"(dst + i * 1024), "v"(d[i]) : "memory");
          tb -= PT1;
          drain();
          if (lane == 0)
            __hip_atomic_store((unsigned*)(pws + WS_FLAGS + qd * 256 + (8 + kt * 7 + s - 1) * 4), call * 8u, __ATOMIC_RELAXED, __HIP_MEMORY_SCOPE_SYSTEM);
        }
      }
    }
    return;
  }
  const unsigned wm = wid >> 1, wn = wid & 1, fr = lane & 15, fq = lane >> 4, q = (lane & 15) >> 2, p = lane & 3;
  const unsigned ra0 = A_RING + (wm * 128 + fr) * 128 + ((fq ^ (fr & 7)) * 16);
  const unsigned rb00 = B_RING + (8 * fq + q) * 256 + (((wn * 8 + p) ^ ((q << 2) | (fq & 1))) * 16) + (p & 1) * 8;
  const unsigned call = rfl(ld_agent(myws + WS_CTR + b * 128)) + 1;
  unsigned tmov = 0;
  int ulast = 20; asm volatile("" : "+s"(ulast));
  f32x4 acc[8][4];
  #pragma unroll
  for (int m = 0; m < 8; ++m)
    #pragma unroll
    for (int n = 0; n < 4; ++n) acc[m][n] = (f32x4){0.f, 0.f, 0.f, 0.f};
  bf16x8 af[8]; s16x4 b0l[4], b0h[4], b1l[4], b1h[4];
  #define MF(m, n_, L, H) MF2(m, (((m) & 1) ? 3 - (n_) : (n_)), L, H)
  #define MF2(m, n, L, H) do { asm volatile("v_mfma_f32_16x16x32_bf16 %0, %1, %2, %0" : "+v"(acc[m][n]) \
                                : "v"(__builtin_shufflevector(L[n], H[n], 0, 1, 2, 3, 4, 5, 6, 7)), "v"(af[m])); SB(); } while (0)
  #define RA(dst, S, KS, m) do { dst = rda(ab[(S) >> 1][KS] + ((S) & 1) * A_ST + (m) * 2048); SB(); } while (0)
  #define RL(L, S, KS, n) do { L[n] = trd(bb[n] + (S) * B_ST + (KS) * 8192); SB(); } while (0)
  #define RH(H, S, KS, n) do { H[n] = trd(bb[n] + (S) * B_ST + (KS) * 8192 + 1024); SB(); } while (0)
  unsigned ab[2][2], bb[4];
  #pragma unroll
  for (int i = 0; i < 4; ++i) {
    ab[i >> 1][i & 1] = (ra0 ^ ((i & 1) * 64)) + (i >> 1) * 2 * A_ST; asm volatile("" : "+v"(ab[i >> 1][i & 1]));
    bb[i] = rb00 ^ ((i >> 1) * 64 + (i & 1) * 8); asm volatile("" : "+v"(bb[i]));
  }
  BAR();
  #pragma unroll
  for (int n = 0; n < 4; ++n) { RL(b0l, 0, 0, n); RH(b0h, 0, 0, n); }
  #pragma unroll
  for (int m = 0; m < 7; ++m) RA(af[m], 0, 0, m);
  #define TILE(S, T, HOOK) do { \
    MF(0, 0, b0l, b0h); RA(af[7], S, 0, 7); MF(0, 1, b0l, b0h); RL(b1l, S, 1, 0); MF(0, 2, b0l, b0h); RH(b1h, S, 1, 0); MF(0, 3, b0l, b0h); \
    _Pragma("unroll") for (int g = 1; g < 8; ++g) { \
      MF(g, 0, b0l, b0h); RA(af[g - 1], S, 1, g - 1); MF(g, 1, b0l, b0h); if (g < 4) RL(b1l, S, 1, g); MF(g, 2, b0l, b0h); if (g < 4) RH(b1h, S, 1, g); MF(g, 3, b0l, b0h); } \
    MF(0, 0, b1l, b1h); RA(af[7], S, 1, 7); MF(0, 1, b1l, b1h); MF(0, 2, b1l, b1h); MF(0, 3, b1l, b1h); \
    MF(1, 0, b1l, b1h); MF(1, 1, b1l, b1h); MF(1, 2, b1l, b1h); MF(1, 3, b1l, b1h); \
    asm volatile("s_waitcnt lgkmcnt(0)\n\ts_barrier" ::: "memory"); SB();     \
    HOOK \
    MF(2, 0, b1l, b1h); RA(af[0], T, 0, 0); MF(2, 1, b1l, b1h); RL(b0l, T, 0, 0); MF(2, 2, b1l, b1h); RH(b0h, T, 0, 0); MF(2, 3, b1l, b1h); RA(af[1], T, 0, 1); \
    _Pragma("unroll") for (int g = 3; g < 8; ++g) { \
      MF(g, 0, b1l, b1h); RA(af[g - 1], T, 0, g - 1); MF(g, 1, b1l, b1h); if (g < 6) RL(b0l, T, 0, g - 2); MF(g, 2, b1l, b1h); if (g < 6) RH(b0h, T, 0, g - 2); MF(g, 3, b1l, b1h); } \
  } while (0)
  for (int u = 0; u < 21; ++u) {
    TILE(0, 1, );
    TILE(1, 2, if (u == ulast) tmov = ld_agent(myws + WS_TMO); SB(););
    TILE(2, 0, );
  }
  bf16x8 ag[8];
  #pragma unroll
  for (int n = 0; n < 4; ++n) { RL(b1l, 0, 1, n); RH(b1h, 0, 1, n); }
  RA(ag[0], 0, 1, 0); RA(ag[1], 0, 1, 1); RA(af[7], 0, 0, 7);
  #pragma unroll
  for (int m = 2; m < 8; ++m) RA(ag[m], 0, 1, m);
  const bool bad = rfl(tmov) == call;
  char* orow = P.out + (size_t)(tm * 256 + wm * 128 + (fr & 7)) * NTOT * 2 + colb + wn * 512 + (fr & 8) * 8 + fq * 16;
  const bool odd = fq & 1;
  auto epi = [&](const f32x4 (&ac)[4], int m) {
    u32x4 v[2], lo, hi;
    #pragma unroll
    for (int pr = 0; pr < 2; ++pr) {
      const f32x4 a = ac[2 * pr], c = ac[2 * pr + 1];
      const unsigned a0 = relu2(cvtpk(a[0], a[1])), a1 = relu2(cvtpk(a[2], a[3])), c0 = relu2(cvtpk(c[0], c[1])), c1 = relu2(cvtpk(c[2], c[3]));
      v[pr][0] = odd ? c0 : a0; v[pr][1] = odd ? c1 : a1; v[pr][2] = odd ? a0 : c0; v[pr][3] = odd ? a1 : c1;
    }
    #pragma unroll
    for (int e = 0; e < 4; ++e) {
      lo[e] = __builtin_amdgcn_update_dpp(v[0][e], v[1][e], 0x128, 0xf, 0xc, false);
      hi[e] = __builtin_amdgcn_update_dpp(v[1][e], v[0][e], 0x128, 0xf, 0x3, false);
    }
    char* pp = orow + (size_t)m * 16 * NTOT * 2;
    asm volatile("global_store_dwordx4 %0, %1, off sc1\n\ts_nop 1" :: "v"(pp), "v"(lo) : "memory");
    asm volatile("global_store_dwordx4 %0, %1, off sc1\n\ts_nop 1" :: "v"(pp + (size_t)8 * NTOT * 2), "v"(hi) : "memory");
  };
  if (bad) {
    const u32x4 qn = {0x7fc07fc0u, 0x7fc07fc0u, 0x7fc07fc0u, 0x7fc07fc0u};
    for (int i = 0; i < 16; ++i) *(u32x4*)(orow + (size_t)i * 8 * NTOT * 2) = qn;
  } else {
    SB();
    #pragma unroll
    for (int m = 0; m < 8; ++m) {
      #pragma unroll
      for (int n = 0; n < 4; ++n) {
        acc[m][n] = __builtin_amdgcn_mfma_f32_16x16x32_bf16(__builtin_bit_cast(bf16x8, __builtin_shufflevector(b0l[n], b0h[n], 0, 1, 2, 3, 4, 5, 6, 7)), af[m], acc[m][n], 0, 0, 0);
        asm volatile("" : "+v"(acc[m][n]));
      }
      #pragma unroll
      for (int n = 0; n < 4; ++n) {
        acc[m][n] = __builtin_amdgcn_mfma_f32_16x16x32_bf16(__builtin_bit_cast(bf16x8, __builtin_shufflevector(b1l[n], b1h[n], 0, 1, 2, 3, 4, 5, 6, 7)), ag[m], acc[m][n], 0, 0, 0);
        asm volatile("" : "+v"(acc[m][n]));
      }
      if (m > 0) {
        epi(acc[m - 1], m - 1);
        #pragma unroll
        for (int k = 0; k < 8; ++k) { __builtin_amdgcn_sched_group_barrier(0x8, 1, 0); __builtin_amdgcn_sched_group_barrier(0x2, 5, 0); }
      }
      SB();
    }
    epi(acc[7], 7);
  }
  if (tid == 0) __hip_atomic_store((unsigned*)(myws + WS_CTR + b * 128), call, __ATOMIC_RELAXED, __HIP_MEMORY_SCOPE_AGENT);
}

extern "C" void kernel_launch(int rank, int nranks, ncclComm_t comm, hipStream_t stream,
                              void* const* d_in, const int* in_sizes, int n_in,
                              void* d_out, int out_size, const int* in_dtypes, int out_dtype,
                              void* const* d_out_all, void* const* d_ws_all, size_t ws_bytes) {
  (void)comm; (void)in_sizes; (void)n_in; (void)out_size; (void)in_dtypes; (void)out_dtype; (void)d_out_all;
  if (nranks != NR || (unsigned)rank >= (unsigned)NR || ws_bytes < WS_NEED) { fprintf(stderr, "unexpected launch geometry\n"); return; }
  static bool init = false;
  if (!init) {
    hipError_t e = hipFuncSetAttribute((const void*)a2a_gemm_relu, hipFuncAttributeMaxDynamicSharedMemorySize, (int)LDS_BYTES);
    if (e != hipSuccess) { fprintf(stderr, "hipFuncSetAttribute: %s\n", hipGetErrorString(e)); return; }
    init = true;
  }
  Params p;
  p.x = (const char*)d_in[0]; p.w = (const char*)d_in[1]; p.out = (char*)d_out;
  for (int i = 0; i < NR; ++i) p.ws[i] = (char*)d_ws_all[i];
  p.rank = rank; p.pad = 0;
  hipLaunchKernelGGL(a2a_gemm_relu, dim3(256), dim3(512), LDS_BYTES, stream, p);
  hipError_t e = hipGetLastError();
  if (e != hipSuccess) fprintf(stderr, "launch: %s\n", hipGetErrorString(e));
}
```

```cpp
#include <hip/hip_runtime.h>
#include <rccl/rccl.h>
#include <cstdio>
#include <cstdint>

typedef __bf16 bf16x8 __attribute__((ext_vector_type(8)));
typedef __bf16 bf16x2 __attribute__((ext_vector_type(2)));
typedef short s16x4 __attribute__((ext_vector_type(4)));
typedef short s16x2 __attribute__((ext_vector_type(2)));
typedef float f32x4 __attribute__((ext_vector_type(4)));
typedef float f32x2 __attribute__((ext_vector_type(2)));
typedef unsigned u32x4 __attribute__((ext_vector_type(4)));
typedef int i32x4 __attribute__((ext_vector_type(4)));
#define LAS __attribute__((address_space(3)))
#define DI __device__ __forceinline__
#define SB() __builtin_amdgcn_sched_barrier(0)
#define BAR() asm volatile("s_barrier" ::: "memory")

constexpr int PACE = 86;
constexpr int PACEN = 34;
constexpr int PACE0 = 200;
constexpr int BSL = 9;
constexpr int PT0 = 120;
constexpr int PT1 = 30;
constexpr int LOOK = 3;
constexpr int NR = 8, KTOT = 4096, NTOT = 16384;
constexpr unsigned A_RING = 0, A_ST = 32768, B_RING = 3 * 32768, B_ST = 16384, LDS_BYTES = 3 * (A_ST + B_ST);
constexpr unsigned WS_FLAGS = 0;
constexpr unsigned WS_TMO = 4096;
constexpr unsigned WS_CTR = 8192;
constexpr unsigned WS_X = 1u << 20;
constexpr unsigned BLK = 524288;
constexpr unsigned XPAR = 8 * BLK;
constexpr size_t WS_NEED = (size_t)WS_X + 2 * (size_t)XPAR;
constexpr unsigned long long DEADLINE_TICKS = 100000000ull;

struct Params { const char* x; const char* w; char* out; char* ws[NR]; int rank; int pad; };

DI unsigned rfl(unsigned x) { return __builtin_amdgcn_readfirstlane(x); }
DI unsigned cvtpk(float lo, float hi) { f32x2 v = {lo, hi}; bf16x2 b = __builtin_convertvector(v, bf16x2); return __builtin_bit_cast(unsigned, b); }
DI unsigned relu2(unsigned x) { const s16x2 v = __builtin_bit_cast(s16x2, x), z = {0, 0}; return __builtin_bit_cast(unsigned, __builtin_elementwise_max(v, z)); }
DI i32x4 make_srd(const void* p, unsigned bytes) {
  unsigned long long a = (unsigned long long)p;
  i32x4 r; r[0] = rfl((unsigned)a); r[1] = rfl((unsigned)(a >> 32)) & 0xffff; r[2] = rfl(bytes); r[3] = 0x00020000; return r;
}
#define DMA1(v, off, pol) "buffer_load_dwordx4 " v ", %5, %6 offen offset:" off pol " lds\n\t"
#define DMA4_ASM(pol) asm volatile("s_mov_b32 m0, %0\n\ts_nop %7\n\t" DMA1("%1", "0", pol) DMA1("%2", "1024", pol) DMA1("%3", "2048", pol) DMA1("%4", "3072", pol) \
    :: "s"(m0v), "v"(v0), "v"(v1), "v"(v2), "v"(v3), "s"(srd), "s"(so), "i"(PAD) : "memory")
template <int SC, int PAD> DI void dma4(unsigned m0v, unsigned v0, unsigned v1, unsigned v2, unsigned v3, i32x4 srd, unsigned so) {
  if constexpr (SC) DMA4_ASM(" sc0 sc1"); else DMA4_ASM("");
}
DI s16x4 trd(unsigned a) { return __builtin_amdgcn_ds_read_tr16_b64_v4i16((LAS s16x4*)(size_t)a); }
DI bf16x8 rda(unsigned a) { return *(const LAS bf16x8*)(size_t)a; }
DI unsigned ld_agent(const char* p) { return __hip_atomic_load((const unsigned*)p, __ATOMIC_RELAXED, __HIP_MEMORY_SCOPE_AGENT); }

__global__ __launch_bounds__(512, 2) void a2a_gemm_relu(const Params P) {
  const int tid = threadIdx.x, lane = tid & 63;
  const unsigned wid = rfl(tid >> 6);
  const unsigned b = blockIdx.x, xcd = b & 7, idx = b >> 3, tn = xcd * 16 + (idx >> 1), tm = idx & 1;
  const unsigned rank = (unsigned)P.rank & 7u;
  char* const myws = P.ws[rank];
  const unsigned colb = (tn >> 2) * 1024 + (tn & 3) * 128;

  if (wid >= 4) {
    unsigned pos = lane, tabA, tabB;
    if (tm) {
      if (lane & 1) { if (lane >= (lane < 8 ? 3 : 11)) pos = lane - 3; }
      else if (lane + 3 < (lane < 8 ? 8 : 64)) pos = lane + 3;
    }
    { const unsigned u = pos; unsigned src, kt;
      if (u < 8) { src = rank; kt = u; tabA = kt * 128; }
      else { const unsigned j = u - 8; kt = (j * 37) >> 8; src = (rank + (j - kt * 7 + 1)) & 7u; tabA = src * BLK + kt * 65536u; }
      tabB = (src * 512u + kt * 64u) * 32768u; }
    if (wid < 6) {
      const unsigned h = wid - 4;
      unsigned callv;
      asm volatile("global_load_dword %0, %1, off sc1" : "=v"(callv) : "v"(myws + WS_CTR + b * 128) : "memory");
      const i32x4 srdL = make_srd(P.x + (size_t)rank * BLK, BLK);
      const unsigned arow = tm * 256 + h * 128 + (lane >> 3), ach = ((lane & 7) ^ (lane >> 3)) * 16;
      unsigned vL[4];
      #pragma unroll
      for (int i = 0; i < 4; ++i) vL[i] = (arow + 8 * i) * 1024 + ach - i * 1024;
      const unsigned vR = arow * 128 + ach, m0A = A_RING + h * 16384;
      auto issueL = [&](int t, unsigned slot) {
        const unsigned so = __builtin_amdgcn_readlane(tabA, t), m = m0A + slot * A_ST;
        dma4<1, 3>(m, vL[0], vL[1], vL[2], vL[3], srdL, so);
        #pragma unroll
        for (int j = 1; j < 4; ++j) dma4<1, 0>(m + j * 4096, vL[0], vL[1], vL[2], vL[3], srdL, so + j * 32768);
      };
      issueL(0, 0);
      issueL(1, 1);
      asm volatile("s_waitcnt vmcnt(16)" : "+v"(callv) :: "memory");
      BAR();
      issueL(2, 2);
      const unsigned call = rfl(callv) + 1;
      i32x4 srdR = make_srd(myws + WS_X + (call & 1) * XPAR, XPAR);
      const char* fl = myws + WS_FLAGS + (tm * 2 + h) * 256 + pos * 4;
      const char* tmo = myws + WS_TMO;
      #define SAMPLE() asm volatile("global_load_dword v200, %0, off sc0 sc1" :: "v"(fl) : "memory", "v200")
      unsigned long long ready = 0xffull;
      bool pend = true;
      unsigned slot = 0;
      for (int u = 0; u < 5; ++u) {
        asm volatile("s_waitcnt vmcnt(16)" ::: "memory"); BAR();
        if (u == 4) SAMPLE();
        issueL(u + 3, slot);
        slot = slot == 2 ? 0 : slot + 1;
      }
      for (int u = 5; u < 63; ++u) {
        const int t = u + 3;
        if (ready != ~0ull) {
          if (pend) {
            unsigned long long mk;
            asm volatile("s_waitcnt vmcnt(16)\n\ts_barrier\n\tv_cmp_ge_u32 %0, v200, %1" : "=s"(mk) : "v"(call * 8u) : "memory", "v200");
            ready |= mk;
          } else {
            asm volatile("s_waitcnt vmcnt(16)\n\ts_barrier" ::: "memory");
          }
          if (!((ready >> t) & 1)) {
            const unsigned long long t0 = __builtin_amdgcn_s_memrealtime();
            for (;;) {
              unsigned fv, tv;
              asm volatile("global_load_dword %0, %2, off sc0 sc1\n\tglobal_load_dword %1, %3, off sc1\n\ts_waitcnt vmcnt(0)"
                           : "=&v"(fv), "=&v"(tv) : "v"(fl), "v"(tmo) : "memory");
              ready |= __ballot(fv >= call * 8u);
              if ((ready >> t) & 1) break;
              if (rfl(tv) == call || __builtin_amdgcn_s_memrealtime() - t0 > DEADLINE_TICKS) {
                if (lane == 0) __hip_atomic_store((unsigned*)tmo, call, __ATOMIC_RELAXED, __HIP_MEMORY_SCOPE_AGENT);
                asm volatile("s_waitcnt vmcnt(0)" ::: "memory");
                ready = ~0ull; srdR[2] = 0;
                break;
              }
              __builtin_amdgcn_s_sleep(2);
            }
          }
          ready |= (2ull << t) - 1;
          pend = ((~ready >> t) & ((2ull << LOOK) - 1)) != 0;
          if (pend) SAMPLE();
        } else {
          asm volatile("s_waitcnt vmcnt(16)\n\ts_barrier" ::: "memory");
        }
        unsigned so = __builtin_amdgcn_readlane(tabA, t & 63);
        if (t >= 64) { srdR[2] = 0; so = 0; }
        const unsigned m = m0A + slot * A_ST;
        dma4<1, 3>(m, vR, vR, vR, vR, srdR, so);
        #pragma unroll
        for (int j = 1; j < 4; ++j) dma4<1, 0>(m + j * 4096, vR, vR, vR, vR, srdR, so + j * 4096);
        slot = slot == 2 ? 0 : slot + 1;
      }
    } else if (wid == 6) {
      const unsigned tp0 = (unsigned)__builtin_amdgcn_s_memrealtime();
      i32x4 srdW = make_srd(P.w, (unsigned)KTOT * NTOT * 2u);
      const i32x4 srdXL = make_srd(P.x + (size_t)rank * BLK, BLK);
      unsigned vB[4], dummy = 0;
      #pragma unroll
      for (int i = 0; i < 4; ++i) {
        const unsigned ch = (lane & 15) ^ (((lane >> 4) << 2) | (i >> 1));
        vB[i] = (4 * i + (lane >> 4)) * 32768 + colb + (ch >> 3) * 512 + (ch & 7) * 16 - i * 1024;
      }
      const unsigned vXL = (tm * 256 + (idx >> 1) * 16 + (lane & 15)) * 1024 + (lane >> 4) * 128;
      auto issue = [&](int t, unsigned slot) {
        const unsigned so = __builtin_amdgcn_readlane(tabB, t & 63), m = B_RING + slot * B_ST;
        dma4<0, 3>(m, vB[0], vB[1], vB[2], vB[3], srdW, so);
        #pragma unroll
        for (int j = 1; j < 4; ++j) dma4<0, 0>(m + j * 4096, vB[0], vB[1], vB[2], vB[3], srdW, so + j * 16 * 32768);
      };
      issue(0, 0);
      issue(1, 1);
      asm volatile("s_waitcnt vmcnt(16)" ::: "memory");
      BAR();
      issue(2, 2);
      asm volatile("buffer_load_dword %0, %1, %2, 0 offen offset:512" : "+v"(dummy) : "v"(vXL), "s"(srdXL) : "memory");
      unsigned tp = tp0 + PACE0, slot = 0;
      for (int u = 0; u < 63; ++u) {
        asm volatile("s_waitcnt vmcnt(16)" ::: "memory");
        if (u <= PACEN) { while ((int)((unsigned)__builtin_amdgcn_s_memrealtime() - tp) < 0) __builtin_amdgcn_s_sleep(1); tp += PACE; }
        BAR();
        if (u >= 61) srdW[2] = 0;
        __builtin_amdgcn_s_sleep(BSL);
        issue(u + 3, slot);
        slot = slot == 2 ? 0 : slot + 1;
      }
      asm volatile("s_waitcnt vmcnt(0)" :: "v"(dummy) : "memory");
    } else {
      int nbar = 0;
      unsigned long long tb = __builtin_amdgcn_s_memrealtime() + (PT0 - PT1);
      auto bar = [&]() { BAR(); ++nbar; tb = __builtin_amdgcn_s_memrealtime(); };
      auto drain = [&](int lim) {
        while ((__builtin_amdgcn_s_getreg(7 | (31 << 11)) & 0xC0000Fu) != 0 && nbar < lim) {
          if ((long long)(__builtin_amdgcn_s_memrealtime() - tb) > PT1) bar();
          else __builtin_amdgcn_s_sleep(1);
        }
        asm volatile("s_waitcnt vmcnt(0)" ::: "memory");
      };
      if (b < 224) {
        unsigned callv;
        u32x4 d[16];
        {
          const unsigned s = (b >> 5) + 1, row = (b & 31) * 16, dest = (rank - s) & 7u;
          const char* src = P.x + (size_t)dest * BLK + (size_t)(row + (lane >> 3)) * 1024 + (lane & 7) * 16;
          asm volatile("global_load_dword %0, %3, off sc1\n\tglobal_load_dwordx4 %1, %4, off\n\tglobal_load_dwordx4 %2, %5, off"
                       : "=&v"(callv), "=&v"(d[0]), "=&v"(d[1]) : "v"(myws + WS_CTR + b * 128), "v"(src), "v"(src + 8192) : "memory");
          char* const pws = P.ws[dest];
          drain(7);
          asm volatile("" : "+v"(callv), "+v"(d[0]), "+v"(d[1]));
          char* dst = pws + WS_X + ((rfl(callv) + 1) & 1) * XPAR + rank * BLK + row * 128 + lane * 16;
          asm volatile("global_store_dwordx4 %0, %1, off sc0 sc1\n\tglobal_store_dwordx4 %0, %2, off offset:1024 sc0 sc1\n\ts_nop 1" :: "v"(dst), "v"(d[0]), "v"(d[1]) : "memory");
          drain(7);
          if (lane == 0) __hip_atomic_fetch_add((unsigned*)(pws + WS_FLAGS + (row >> 7) * 256 + (8 + s - 1) * 4), 1u, __ATOMIC_RELAXED, __HIP_MEMORY_SCOPE_SYSTEM);
        }
        if (b < 196) {
          const unsigned call = rfl(callv) + 1;
          const unsigned kt = b / 28 + 1, r = b % 28, s = (r >> 2) + 1, qd = r & 3, dest = (rank - s) & 7u;
          const char* src = P.x + (size_t)dest * BLK + (size_t)(qd * 128 + (lane >> 3)) * 1024 + kt * 128 + (lane & 7) * 16;
          const int su = (int)(4 * kt + qd);
          while (nbar < su - 2) bar();
          #pragma unroll
          for (int i = 0; i < 16; i += 4)
            asm volatile("global_load_dwordx4 %0, %4, off\n\tglobal_load_dwordx4 %1, %5, off\n\tglobal_load_dwordx4 %2, %6, off\n\tglobal_load_dwordx4 %3, %7, off"
                         : "=&v"(d[i]), "=&v"(d[i + 1]), "=&v"(d[i + 2]), "=&v"(d[i + 3])
                         : "v"(src + i * 8192), "v"(src + (i + 1) * 8192), "v"(src + (i + 2) * 8192), "v"(src + (i + 3) * 8192) : "memory");
          char* const pws = P.ws[dest];
          char* dst = pws + WS_X + (call & 1) * XPAR + rank * BLK + kt * 65536 + qd * 16384 + lane * 16;
          drain((int)(7 * kt + 4));
          #pragma unroll
          for (int i = 0; i < 16; i += 4) asm volatile("" : "+v"(d[i]), "+v"(d[i + 1]), "+v"(d[i + 2]), "+v"(d[i + 3]));
          while (nbar < su) bar();
          #pragma unroll
          for (int i = 0; i < 16; ++i)
            asm volatile("global_store_dwordx4 %0, %1, off sc0 sc1\n\ts_nop 1" :: "v"(dst + i * 1024), "v"(d[i]) : "memory");
          tb -= PT1;
          drain((int)(7 * kt + 4));
          if (lane == 0)
            __hip_atomic_store((unsigned*)(pws + WS_FLAGS + qd * 256 + (8 + kt * 7 + s - 1) * 4), call * 8u, __ATOMIC_RELAXED, __HIP_MEMORY_SCOPE_SYSTEM);
        }
      }
    }
    return;
  }
  const unsigned wm = wid >> 1, wn = wid & 1, fr = lane & 15, fq = lane >> 4, q = (lane & 15) >> 2, p = lane & 3;
  const unsigned ra0 = A_RING + (wm * 128 + fr) * 128 + ((fq ^ (fr & 7)) * 16);
  const unsigned rb00 = B_RING + (8 * fq + q) * 256 + (((wn * 8 + p) ^ ((q << 2) | (fq & 1))) * 16) + (p & 1) * 8;
  const unsigned call = rfl(ld_agent(myws + WS_CTR + b * 128)) + 1;
  unsigned tmov = 0;
  int ulast = 20; asm volatile("" : "+s"(ulast));
  f32x4 acc[8][4];
  #pragma unroll
  for (int m = 0; m < 8; ++m)
    #pragma unroll
    for (int n = 0; n < 4; ++n) acc[m][n] = (f32x4){0.f, 0.f, 0.f, 0.f};
  bf16x8 af[8]; s16x4 b0l[4], b0h[4], b1l[4], b1h[4];
  #define MF(m, n_, L, H) MF2(m, (((m) & 1) ? 3 - (n_) : (n_)), L, H)
  #define MF2(m, n, L, H) do { asm volatile("v_mfma_f32_16x16x32_bf16 %0, %1, %2, %0" : "+v"(acc[m][n]) \
                                : "v"(__builtin_shufflevector(L[n], H[n], 0, 1, 2, 3, 4, 5, 6, 7)), "v"(af[m])); SB(); } while (0)
  #define RA(dst, S, KS, m) do { dst = rda(ab[(S) >> 1][KS] + ((S) & 1) * A_ST + (m) * 2048); SB(); } while (0)
  #define RL(L, S, KS, n) do { L[n] = trd(bb[n] + (S) * B_ST + (KS) * 8192); SB(); } while (0)
  #define RH(H, S, KS, n) do { H[n] = trd(bb[n] + (S) * B_ST + (KS) * 8192 + 1024); SB(); } while (0)
  unsigned ab[2][2], bb[4];
  #pragma unroll
  for (int i = 0; i < 4; ++i) {
    ab[i >> 1][i & 1] = (ra0 ^ ((i & 1) * 64)) + (i >> 1) * 2 * A_ST; asm volatile("" : "+v"(ab[i >> 1][i & 1]));
    bb[i] = rb00 ^ ((i >> 1) * 64 + (i & 1) * 8); asm volatile("" : "+v"(bb[i]));
  }
  BAR();
  #pragma unroll
  for (int n = 0; n < 4; ++n) { RL(b0l, 0, 0, n); RH(b0h, 0, 0, n); }
  #pragma unroll
  for (int m = 0; m < 7; ++m) RA(af[m], 0, 0, m);
  #define TILE(S, T, HOOK) do { \
    MF(0, 0, b0l, b0h); RA(af[7], S, 0, 7); MF(0, 1, b0l, b0h); RL(b1l, S, 1, 0); MF(0, 2, b0l, b0h); RH(b1h, S, 1, 0); MF(0, 3, b0l, b0h); \
    _Pragma("unroll") for (int g = 1; g < 8; ++g) { \
      MF(g, 0, b0l, b0h); RA(af[g - 1], S, 1, g - 1); MF(g, 1, b0l, b0h); if (g < 4) RL(b1l, S, 1, g); MF(g, 2, b0l, b0h); if (g < 4) RH(b1h, S, 1, g); MF(g, 3, b0l, b0h); } \
    MF(0, 0, b1l, b1h); RA(af[7], S, 1, 7); MF(0, 1, b1l, b1h); MF(0, 2, b1l, b1h); MF(0, 3, b1l, b1h); \
    MF(1, 0, b1l, b1h); MF(1, 1, b1l, b1h); MF(1, 2, b1l, b1h); MF(1, 3, b1l, b1h); \
    asm volatile("s_waitcnt lgkmcnt(0)\n\ts_barrier" ::: "memory"); SB();     \
    HOOK \
    MF(2, 0, b1l, b1h); RA(af[0], T, 0, 0); MF(2, 1, b1l, b1h); RL(b0l, T, 0, 0); MF(2, 2, b1l, b1h); RH(b0h, T, 0, 0); MF(2, 3, b1l, b1h); RA(af[1], T, 0, 1); \
    _Pragma("unroll") for (int g = 3; g < 8; ++g) { \
      MF(g, 0, b1l, b1h); RA(af[g - 1], T, 0, g - 1); MF(g, 1, b1l, b1h); if (g < 6) RL(b0l, T, 0, g - 2); MF(g, 2, b1l, b1h); if (g < 6) RH(b0h, T, 0, g - 2); MF(g, 3, b1l, b1h); } \
  } while (0)
  for (int u = 0; u < 21; ++u) {
    TILE(0, 1, );
    TILE(1, 2, if (u == ulast) tmov = ld_agent(myws + WS_TMO); SB(););
    TILE(2, 0, );
  }
  bf16x8 ag[8];
  #pragma unroll
  for (int n = 0; n < 4; ++n) { RL(b1l, 0, 1, n); RH(b1h, 0, 1, n); }
  RA(ag[0], 0, 1, 0); RA(ag[1], 0, 1, 1); RA(af[7], 0, 0, 7);
  #pragma unroll
  for (int m = 2; m < 8; ++m) RA(ag[m], 0, 1, m);
  const bool bad = rfl(tmov) == call;
  char* orow = P.out + (size_t)(tm * 256 + wm * 128 + (fr & 7)) * NTOT * 2 + colb + wn * 512 + (fr & 8) * 8 + fq * 16;
  const bool odd = fq & 1;
  auto epi = [&](const f32x4 (&ac)[4], int m) {
    u32x4 v[2], lo, hi;
    #pragma unroll
    for (int pr = 0; pr < 2; ++pr) {
      const f32x4 a = ac[2 * pr], c = ac[2 * pr + 1];
      const unsigned a0 = relu2(cvtpk(a[0], a[1])), a1 = relu2(cvtpk(a[2], a[3])), c0 = relu2(cvtpk(c[0], c[1])), c1 = relu2(cvtpk(c[2], c[3]));
      v[pr][0] = odd ? c0 : a0; v[pr][1] = odd ? c1 : a1; v[pr][2] = odd ? a0 : c0; v[pr][3] = odd ? a1 : c1;
    }
    #pragma unroll
    for (int e = 0; e < 4; ++e) {
      lo[e] = __builtin_amdgcn_update_dpp(v[0][e], v[1][e], 0x128, 0xf, 0xc, false);
      hi[e] = __builtin_amdgcn_update_dpp(v[1][e], v[0][e], 0x128, 0xf, 0x3, false);
    }
    char* pp = orow + (size_t)m * 16 * NTOT * 2;
    asm volatile("global_store_dwordx4 %0, %1, off sc1\n\ts_nop 1" :: "v"(pp), "v"(lo) : "memory");
    asm volatile("global_store_dwordx4 %0, %1, off sc1\n\ts_nop 1" :: "v"(pp + (size_t)8 * NTOT * 2), "v"(hi) : "memory");
  };
  if (bad) {
    const u32x4 qn = {0x7fc07fc0u, 0x7fc07fc0u, 0x7fc07fc0u, 0x7fc07fc0u};
    for (int i = 0; i < 16; ++i) *(u32x4*)(orow + (size_t)i * 8 * NTOT * 2) = qn;
  } else {
    SB();
    #pragma unroll
    for (int m = 0; m < 8; ++m) {
      #pragma unroll
      for (int n = 0; n < 4; ++n) {
        acc[m][n] = __builtin_amdgcn_mfma_f32_16x16x32_bf16(__builtin_bit_cast(bf16x8, __builtin_shufflevector(b0l[n], b0h[n], 0, 1, 2, 3, 4, 5, 6, 7)), af[m], acc[m][n], 0, 0, 0);
        asm volatile("" : "+v"(acc[m][n]));
      }
      #pragma unroll
      for (int n = 0; n < 4; ++n) {
        acc[m][n] = __builtin_amdgcn_mfma_f32_16x16x32_bf16(__builtin_bit_cast(bf16x8, __builtin_shufflevector(b1l[n], b1h[n], 0, 1, 2, 3, 4, 5, 6, 7)), ag[m], acc[m][n], 0, 0, 0);
        asm volatile("" : "+v"(acc[m][n]));
      }
      if (m > 0) {
        epi(acc[m - 1], m - 1);
        #pragma unroll
        for (int k = 0; k < 8; ++k) { __builtin_amdgcn_sched_group_barrier(0x8, 1, 0); __builtin_amdgcn_sched_group_barrier(0x2, 5, 0); }
      }
      SB();
    }
    epi(acc[7], 7);
  }
  if (tid == 0) __hip_atomic_store((unsigned*)(myws + WS_CTR + b * 128), call, __ATOMIC_RELAXED, __HIP_MEMORY_SCOPE_AGENT);
}

extern "C" void kernel_launch(int rank, int nranks, ncclComm_t comm, hipStream_t stream,
                              void* const* d_in, const int* in_sizes, int n_in,
                              void* d_out, int out_size, const int* in_dtypes, int out_dtype,
                              void* const* d_out_all, void* const* d_ws_all, size_t ws_bytes) {
  (void)comm; (void)d_out_all;
  if (nranks != NR || (unsigned)rank >= (unsigned)NR || ws_bytes < WS_NEED || n_in != 2 || in_sizes[0] != KTOT * 512 || in_sizes[1] != KTOT * NTOT ||
      out_size != 512 * NTOT || in_dtypes[0] != 1 || in_dtypes[1] != 1 || out_dtype != 1) { fprintf(stderr, "unexpected problem geometry\n"); return; }
  static bool init = false;
  if (!init) {
    hipError_t e = hipFuncSetAttribute((const void*)a2a_gemm_relu, hipFuncAttributeMaxDynamicSharedMemorySize, (int)LDS_BYTES);
    if (e != hipSuccess) { fprintf(stderr, "hipFuncSetAttribute: %s\n", hipGetErrorString(e)); return; }
    init = true;
  }
  Params p;
  p.x = (const char*)d_in[0]; p.w = (const char*)d_in[1]; p.out = (char*)d_out;
  for (int i = 0; i < NR; ++i) p.ws[i] = (char*)d_ws_all[i];
  p.rank = rank; p.pad = 0;
  hipLaunchKernelGGL(a2a_gemm_relu, dim3(256), dim3(512), LDS_BYTES, stream, p);
  hipError_t e = hipGetLastError();
  if (e != hipSuccess) fprintf(stderr, "launch: %s\n", hipGetErrorString(e));
}
```
